```python
import numpy as np

kernel_name = "gpu_ag_gemm_m4096_k4096_n16384_e4m3e5m2_dqrelu"


ABSMAX_RTOL = 0.05

COMPUTE_DTYPE = "float32"
EPILOGUE = "dequant_relu"
IN_QUANT = None
OUT_QUANT = None
IN_FP8 = "e4m3e5m2"
IN_DTYPES = {"x": "e4m3", "w_mat": "e5m2",
             "scale_x": "float32", "scale_w": "float32"}
WORLD_SIZE = 8

def _snap_fp8(x, mant_bits, max_abs, min_normal):
    a = np.abs(x.astype(np.float64))
    _m, _e = np.frexp(a)
    q = np.where(a >= min_normal, np.ldexp(1.0, _e - 1 - mant_bits),
                 min_normal * 2.0 ** -mant_bits)
    snapped = np.minimum(np.round(a / q) * q, max_abs)
    return (np.sign(x) * snapped).astype(np.float32)

def setup_inputs():

    x = _snap_fp8(np.random.randn(4096, 4096).astype(np.float32), 3, 448.0, 2.0 ** -6)
    w_mat = _snap_fp8(np.random.randn(4096, 16384).astype(np.float32), 2, 57344.0, 2.0 ** -14)
    scale_x = (0.02 * (1.0 + np.random.rand(1))).astype(np.float32)
    scale_w = (0.02 * (1.0 + np.random.rand(1))).astype(np.float32)
    return {"x": x, "w_mat": w_mat, "scale_x": scale_x, "scale_w": scale_w}

def reference(x, w_mat, scale_x, scale_w):
    acc = (x @ w_mat).astype(np.float32)
    y = (acc * (scale_x * scale_w)).astype(np.float32)
    return np.maximum(y, 0.0).astype(np.float32)
```

```cpp
#include <hip/hip_runtime.h>
#include <rccl/rccl.h>
#include <cstdio>
#include <cstdint>
#include <type_traits>

#define HIP_CHECK(expr) do { hipError_t _e = (expr); if (_e != hipSuccess) { \
    fprintf(stderr, "HIP error %s at %s:%d\n", hipGetErrorString(_e), __FILE__, __LINE__); \
    return; } } while (0)

typedef int v2i __attribute__((ext_vector_type(2)));
typedef int v4i __attribute__((ext_vector_type(4)));
typedef unsigned v4u __attribute__((ext_vector_type(4)));
typedef int v8i __attribute__((ext_vector_type(8)));
typedef float v4f __attribute__((ext_vector_type(4)));
#define LDSP(T) __attribute__((address_space(3))) T*

constexpr int NR = 8;
constexpr int M = 4096, N = 2048, K = 4096;
constexpr int MS = M / NR;
constexpr int BM = 256, BN = 128, BK = 128;
constexpr int A_BYTES = BM * BK, B_BYTES = BK * BN, STAGE = A_BYTES + B_BYTES, NST = 3;
constexpr int NKS = K / BK;
constexpr int KC = 256;
constexpr int NCH = K / KC;
constexpr int KPC = KC / BK;
constexpr int PARTS = 64;
constexpr int ARRIVALS = PARTS;
constexpr int GEMM_WGS = (M / BM) * (N / BN);
constexpr int NTHREADS = 768;
#define PFD 3
#define WLO 2u
#define WHI 14u
#define OAUX 17
constexpr unsigned NPUSH = 256 + 32;
constexpr int LDS_CTRL = NST * STAGE;
constexpr int LDS_DUMMY = LDS_CTRL + 64;
constexpr int LDS_FLAG = LDS_DUMMY + 256;
constexpr int LDS_BYTES = LDS_FLAG + 8 * 256;
constexpr size_t WS_TMO = 0;
constexpr size_t WS_CNT = 4096;
constexpr size_t WS_PTICK = 24576;
constexpr size_t WS_PCALL = 32768;
constexpr int NITEM = NCH * PARTS;
constexpr size_t WS_STG = 1u << 20;
constexpr size_t STG_BYTES = (size_t)M * K;
constexpr size_t WS_NEED = WS_STG + 2 * STG_BYTES;
constexpr unsigned long long DEADLINE = 40000000ull;

struct Params {
  const unsigned char* x; const unsigned char* w; const float* sx; const float* sw; float* out;
  unsigned char* ws[NR];
  int rank;
};

__device__ __forceinline__ void glds16(const void* sbase, unsigned voff, unsigned lds_off) {
  unsigned keep;
  asm volatile("s_mov_b32 %0, m0\n\ts_mov_b32 m0, %3\n\ts_nop 0\n\tglobal_load_lds_dwordx4 %1, %2\n\ts_mov_b32 m0, %0"
               : "=&s"(keep) : "v"(voff), "s"(sbase), "s"(lds_off) : "memory");
}

template <typename T> __device__ __forceinline__ T* uni(T* p) {
  unsigned long long v = (unsigned long long)p;
  unsigned lo = __builtin_amdgcn_readfirstlane((unsigned)v), hi = __builtin_amdgcn_readfirstlane((unsigned)(v >> 32));
  return (T*)(((unsigned long long)hi << 32) | lo);
}

__device__ __forceinline__ void glds4_sink(const void* vaddr, unsigned lds_off) {
  unsigned keep;
  asm volatile("s_mov_b32 %0, m0\n\ts_mov_b32 m0, %2\n\ts_nop 0\n\tglobal_load_lds_dword %1, off\n\ts_mov_b32 m0, %0"
               : "=&s"(keep) : "v"(vaddr), "s"(lds_off) : "memory");
}

__device__ __forceinline__ void glds4_flag(const void* sbase, unsigned lds_off) {
  unsigned keep;
  asm volatile("s_mov_b32 %0, m0\n\ts_mov_b32 m0, %3\n\ts_nop 0\n\tglobal_load_lds_dword %1, %2 sc0 sc1\n\ts_mov_b32 m0, %0"
               : "=&s"(keep) : "v"(0u), "s"(sbase), "s"(lds_off) : "memory");
}

__device__ __forceinline__ void wait_cnt(const unsigned* flag, unsigned target, unsigned* tmo, unsigned call) {
  unsigned long long t0 = __builtin_amdgcn_s_memrealtime();
  while (__hip_atomic_load(flag, __ATOMIC_RELAXED, __HIP_MEMORY_SCOPE_SYSTEM) < target) {
    if (__hip_atomic_load(tmo, __ATOMIC_RELAXED, __HIP_MEMORY_SCOPE_AGENT) == call ||
        __builtin_amdgcn_s_memrealtime() - t0 > DEADLINE) {
      __hip_atomic_store(tmo, call, __ATOMIC_RELAXED, __HIP_MEMORY_SCOPE_AGENT);
      break;
    }
    __builtin_amdgcn_s_sleep(1);
  }
  asm volatile("" ::: "memory");
}

__global__ void __launch_bounds__(NTHREADS, 3) ag_gemm_kernel(Params p) {
  extern __shared__ __attribute__((aligned(16))) unsigned char smem[];
  const int tid = threadIdx.x, l = tid & 63, w = __builtin_amdgcn_readfirstlane(tid >> 6);
  unsigned char* myws = p.ws[p.rank];
  const unsigned lds0 = (unsigned)(uintptr_t)smem;
  LDSP(unsigned) ctrl = (LDSP(unsigned))(uintptr_t)(lds0 + LDS_CTRL);

  const int bid = blockIdx.x, xcd = bid & 7, idx = bid >> 3;
  const int tm = (xcd >> 1) * 4 + (idx >> 3), tn = (xcd & 1) * 8 + (idx & 7);
  auto ldc = [&]() { return (unsigned)__builtin_amdgcn_readfirstlane(__hip_atomic_load(ctrl + 1, __ATOMIC_RELAXED, __HIP_MEMORY_SCOPE_WORKGROUP)); };
  auto sig = [&](unsigned inc) { if (l == 0) __hip_atomic_fetch_add(ctrl + 1, inc, __ATOMIC_RELAXED, __HIP_MEMORY_SCOPE_WORKGROUP); };
  const int src = tm >> 1;
  const bool own = (src == p.rank);

  if (w == 8 || (w == 11 && own)) {
    const unsigned v = (w == 8) ? (unsigned)bid : (unsigned)(GEMM_WGS + (tm & 1) * 16 + tn);
    const unsigned j = v % (NR - 1), nj = (NPUSH - j + NR - 2) / (NR - 1);
    const unsigned loff = (unsigned)(l >> 4) * K + (l & 15) * 16;
    auto item_off = [&](unsigned t) { return (t % PARTS) * 8 * K + (t / PARTS) * KC + loff; };
    unsigned* pcall = (unsigned*)(myws + WS_PCALL + ((w == 11 ? 256 : 0) + blockIdx.x) * 128);
    unsigned t0 = v / (NR - 1), t1 = nj + t0;
    unsigned o0 = item_off(t0);
    v4u cur[2], nxt[2];
    unsigned n = __hip_atomic_load(pcall, __ATOMIC_RELAXED, __HIP_MEMORY_SCOPE_AGENT);
    #pragma unroll
    for (int i = 0; i < 2; ++i) cur[i] = *(const v4u*)(p.x + o0 + i * 4 * K);
    __builtin_amdgcn_s_barrier();
    n = __builtin_amdgcn_readfirstlane(n);
    const unsigned parity = (n + 1) & 1;
    const unsigned dst = (unsigned)(p.rank + 1 + j) % NR;
    unsigned char* dws = p.ws[dst];
    __amdgpu_buffer_rsrc_t rs = __builtin_amdgcn_make_buffer_rsrc(dws + WS_STG + parity * STG_BYTES + (size_t)p.rank * MS * K, 0, MS * K, 0x00020000);
    unsigned vz; asm volatile("v_mov_b32 %0, 0" : "=v"(vz));
    unsigned* cnt = (unsigned*)(dws + WS_CNT + (size_t)p.rank * NCH * 128) + vz;
    unsigned* tick = (unsigned*)(myws + WS_PTICK + j * 128) + vz;
    const unsigned tbase = n * NITEM - 2 * nj;
    unsigned cprev = ~0u;
#define PW(n) asm volatile("s_waitcnt vmcnt(%0)" :: "i"(n) : "memory")
#define PSIG() do { if (cprev != ~0u && l == 0) __hip_atomic_fetch_add(cnt + cprev * 32, 1u, __ATOMIC_RELAXED, __HIP_MEMORY_SCOPE_SYSTEM); } while (0)
    asm volatile("" :: "v"(cur[0]), "v"(cur[1]));
    auto body = [&](auto wtag) {
      constexpr bool WIDE = decltype(wtag)::value;
      const unsigned o1 = item_off(t1 < (unsigned)NITEM ? t1 : t0);
      unsigned t2 = 0;
      if (l == 0) t2 = __hip_atomic_fetch_add(tick, 1u, __ATOMIC_RELAXED, __HIP_MEMORY_SCOPE_AGENT);
      #pragma unroll
      for (int i = 0; i < 2; ++i) nxt[i] = *(const v4u*)(p.x + o1 + i * 4 * K);
      if (WIDE) {
        PW(4);
        __builtin_amdgcn_raw_buffer_store_b128(cur[0], rs, o0, 0, 17);
        PW(4);
        PSIG();
      } else {
        PW(3);
        PSIG();
        __builtin_amdgcn_raw_buffer_store_b128(cur[0], rs, o0, 0, 17);
        PW(0);
      }
      __builtin_amdgcn_raw_buffer_store_b128(cur[1], rs, o0 + 4 * K, 0, 17);
      asm volatile("" ::: "memory");
      cprev = t0 / PARTS; t0 = t1; o0 = o1;
      t1 = __builtin_amdgcn_readfirstlane(t2) - tbase;
      #pragma unroll
      for (int i = 0; i < 2; ++i) cur[i] = nxt[i];
      asm volatile("" :: "v"(cur[0]), "v"(cur[1]));
    };
    const unsigned lo = own ? (unsigned)NITEM : WLO * PARTS, hi = WHI * PARTS;
    while (t0 < lo) body(std::true_type{});
    while (t0 < hi) body(std::false_type{});
    while (t0 < (unsigned)NITEM) body(std::true_type{});
    PW(0);
    PSIG();
    if (l == 0) __hip_atomic_store(pcall, n + 1, __ATOMIC_RELAXED, __HIP_MEMORY_SCOPE_AGENT);
    return;
  }
  if (w == 11) {
    unsigned* pcall = (unsigned*)(myws + WS_PCALL + (256 + blockIdx.x) * 128);
    unsigned n = __hip_atomic_load(pcall, __ATOMIC_RELAXED, __HIP_MEMORY_SCOPE_AGENT);
    __builtin_amdgcn_s_barrier();
    __builtin_amdgcn_fence(__ATOMIC_ACQUIRE, "");
    asm volatile("s_waitcnt vmcnt(0)" ::: "memory");
    const unsigned call = __builtin_amdgcn_readfirstlane(n) + 1;
    const unsigned* cnt = (const unsigned*)(myws + WS_CNT + (size_t)src * NCH * 128) + (l & (NCH - 1)) * 32;
    unsigned* tmo = (unsigned*)(myws + WS_TMO);
    const unsigned target = call * ARRIVALS;
    const unsigned tag = ((call & 1) + 1) << 8;
    unsigned have = 0;
    if (l == 0) __hip_atomic_store(ctrl + 3, tag, __ATOMIC_RELAXED, __HIP_MEMORY_SCOPE_WORKGROUP);
    const unsigned long long t0 = __builtin_amdgcn_s_memrealtime();
    while (have < (unsigned)NCH) {
      unsigned v = __hip_atomic_load(cnt, __ATOMIC_RELAXED, __HIP_MEMORY_SCOPE_SYSTEM);
      unsigned long long m = __builtin_amdgcn_ballot_w64(v < target) | (1ull << NCH);
      unsigned now = (unsigned)__builtin_ctzll(m);
      if (now > have) {
        have = now;
        if (l == 0) __hip_atomic_store(ctrl + 3, tag | have, __ATOMIC_RELAXED, __HIP_MEMORY_SCOPE_WORKGROUP);
      } else if (__builtin_amdgcn_s_memrealtime() - t0 > DEADLINE ||
                 ((__builtin_amdgcn_s_memrealtime() - t0 > 100000) && __hip_atomic_load(tmo, __ATOMIC_RELAXED, __HIP_MEMORY_SCOPE_AGENT) == call)) {
        if (l == 0) {
          __hip_atomic_store(tmo, call, __ATOMIC_RELAXED, __HIP_MEMORY_SCOPE_AGENT);
          __hip_atomic_store(ctrl, 1u, __ATOMIC_RELAXED, __HIP_MEMORY_SCOPE_WORKGROUP);
          asm volatile("s_waitcnt lgkmcnt(0)" ::: "memory");
          __hip_atomic_store(ctrl + 3, tag | NCH, __ATOMIC_RELAXED, __HIP_MEMORY_SCOPE_WORKGROUP);
        }
        break;
      }
    }
    if (l == 0) __hip_atomic_store(pcall, call, __ATOMIC_RELAXED, __HIP_MEMORY_SCOPE_AGENT);
    return;
  }
  if (w > 8) {
    const int h = w - 9;
    __builtin_amdgcn_s_barrier();
    auto ldr = [&]() { return (unsigned)__builtin_amdgcn_readfirstlane(__hip_atomic_load(ctrl + 3, __ATOMIC_RELAXED, __HIP_MEMORY_SCOPE_WORKGROUP)); };
    unsigned rdy = 0x100u | NCH;
    if (!own) do { rdy = ldr(); } while (rdy == 0);
    const unsigned parity = (rdy >> 8) - 1;
    rdy &= 255u;
    const unsigned char* Xb = uni(own ? (p.x + (size_t)(tm & 1) * BM * K) : (myws + WS_STG + parity * STG_BYTES + (size_t)tm * BM * K));
    unsigned aoff[2];
    #pragma unroll
    for (int i = 0; i < 2; ++i) { int r = h * 128 + i * 8 + (l >> 3); int c = (l & 7) ^ (((l >> 4) & 1) | ((i & 1) * 6)); aoff[i] = r * K + c * 16; }
    const unsigned inc = h ? (1u << 22) : (1u << 12);
    int done = 0, buf = 0;
    for (int j = 0; j < NKS; ++j) {
      const unsigned need = (unsigned)(j / KPC + 1);
      if (rdy < need) rdy = ldr() & 255u;
      if (rdy < need || (j >= 3 && (ldc() & 0xfffu) < 8u * (j - 1))) {
        if (done < j) { asm volatile("s_waitcnt vmcnt(0)" ::: "memory"); sig(inc * (j - done)); done = j; }
        while (rdy < need) rdy = ldr() & 255u;
        while (j >= 3 && (ldc() & 0xfffu) < 8u * (j - 1)) {}
      }
      asm volatile("" ::: "memory");
      const unsigned char* xa = Xb + j * BK;
      const unsigned d = lds0 + buf * STAGE + h * 128 * 128;
      #pragma unroll
      for (int i = 0; i < 16; ++i) glds16(xa + (size_t)(i >> 1) * 16 * K, aoff[i & 1], d + i * 1024);
      if (done < j) { asm volatile("s_waitcnt vmcnt(16)" ::: "memory"); sig(inc * (j - done)); done = j; }
      buf = buf + 1; if (buf >= 3) buf = 0;
    }
    asm volatile("s_waitcnt vmcnt(0)" ::: "memory");
    sig(inc * (NKS - done));
    return;
  }

  const unsigned char* Wb = uni(p.w + (size_t)tn * BN);
  unsigned boff[2];
  #pragma unroll
  for (int i = 0; i < 2; ++i) { int k = w * 16 + i * 8 + (l >> 3); int c = (l & 7) ^ (((l >> 4) & 3) | (((w >> 1) & 1) << 2)); boff[i] = k * N + c * 16; }
  #pragma unroll
  for (int s = 0; s < 3; ++s)
    #pragma unroll
    for (int i = 0; i < 2; ++i) glds16(Wb + (size_t)s * BK * N, boff[i], lds0 + s * STAGE + A_BYTES + (w * 16 + i * 8) * 128);
  if (tid == 0) { ctrl[0] = 0; ctrl[1] = 0; ctrl[3] = 0; }
  asm volatile("s_waitcnt lgkmcnt(0)" ::: "memory");
  __builtin_amdgcn_s_barrier();
  auto waitc = [&](unsigned c, unsigned la) {
    for (;;) { unsigned v = ldc(); if ((v & 0xfffu) >= c && ((v >> 12) & 0x3ffu) >= la && (v >> 22) >= la) break; }
    asm volatile("" ::: "memory");
  };
  const int wm = w >> 1, wn = w & 1, g = l >> 4, r16 = l & 15;
  const unsigned xs = ((r16 >> 1) & 1) | (((r16 >> 3) & 1) * 6);
  const unsigned xa0 = (wm * 64 + r16) * 128 + (((2 * g) ^ xs) * 16);
  const unsigned xa1 = xa0 ^ 16;
  const int q = r16 >> 1, pp = l & 1;
  const unsigned sB = ((q >> 1) & 3) | ((g & 1) << 2);
  unsigned wa[4];
  #pragma unroll
  for (int ni = 0; ni < 4; ++ni) wa[ni] = A_BYTES + (32 * g + q) * 128 + (((wn * 4 + ni) ^ sB) * 16) + 8 * pp;
  v4f acc[4][4];
  #pragma unroll
  for (int a = 0; a < 4; ++a)
    #pragma unroll
    for (int b = 0; b < 4; ++b) acc[a][b] = (v4f){0, 0, 0, 0};
  v8i xf[4], wf[4];
#define RW(ni, base) do { _Pragma("unroll") for (int i_ = 0; i_ < 4; ++i_) { \
    v2i t_ = __builtin_amdgcn_ds_read_tr8_b64_v2i32((LDSP(v2i))(uintptr_t)((base) + wa[ni] + i_ * 1024)); \
    wf[ni][2 * i_] = t_.x; wf[ni][2 * i_ + 1] = t_.y; } } while (0)
#define RX(mi, base) do { v4i t0_ = *(LDSP(v4i))(uintptr_t)((base) + xa0 + (mi) * 2048); v4i t1_ = *(LDSP(v4i))(uintptr_t)((base) + xa1 + (mi) * 2048); \
    xf[mi] = (v8i){t0_.x, t0_.y, t0_.z, t0_.w, t1_.x, t1_.y, t1_.z, t1_.w}; } while (0)
#define MM(mi, ni) acc[ni][mi] = __builtin_amdgcn_mfma_scale_f32_16x16x128_f8f6f4(wf[ni], xf[mi], acc[ni][mi], 1, 0, 0, 0, 0, 0)
#define SB __builtin_amdgcn_sched_barrier(0)
  asm volatile("s_waitcnt vmcnt(4)" ::: "memory");
  sig(1u);
  waitc(8u, 1u);
  RX(0, lds0); RX(1, lds0); RW(0, lds0); RW(1, lds0);
  int buf = 0;
  for (int s = 0; s < NKS - 1; ++s) {
    const unsigned base = lds0 + buf * STAGE;
    int nb = buf + 1; if (nb >= 3) nb = 0;
    const unsigned nbase = lds0 + nb * STAGE;
    SB; RW(2, base); RW(3, base); RX(2, base); RX(3, base); SB;
    MM(0, 0); MM(0, 1); MM(1, 0); MM(1, 1);
    MM(0, 2); MM(0, 3); MM(1, 2); MM(1, 3); SB;
    if (s + 2 < NKS) asm volatile("s_waitcnt vmcnt(2) lgkmcnt(0)" ::: "memory"); else asm volatile("s_waitcnt vmcnt(0) lgkmcnt(0)" ::: "memory");
    sig(1u);
    SB; MM(2, 0); MM(2, 1); SB;
    waitc(8u * (s + 2), s + 2);
    RX(0, nbase); RX(1, nbase);
    if (s + 3 < NKS) {
      const unsigned char* wb = Wb + (size_t)(s + 3) * BK * N;
      glds16(wb, boff[0], base + A_BYTES + (w * 16) * 128);
      glds16(wb, boff[1], base + A_BYTES + (w * 16 + 8) * 128);
    }
    SB; MM(3, 0); MM(3, 1); SB;
    RW(0, nbase); RW(1, nbase);
    SB; MM(2, 2); MM(2, 3); MM(3, 2); MM(3, 3); SB;
    buf = nb;
  }
  {
    const unsigned base = lds0 + buf * STAGE;
    RW(2, base); RW(3, base); RX(2, base); RX(3, base);
    MM(0, 0); MM(0, 1); MM(1, 0); MM(1, 1);
    MM(0, 2); MM(0, 3); MM(1, 2); MM(1, 3);
    MM(2, 0); MM(2, 1); MM(3, 0); MM(3, 1);
    MM(2, 2); MM(2, 3); MM(3, 2); MM(3, 3);
  }
  float sc = p.sx[0] * p.sw[0];
  const bool bad = __hip_atomic_load(ctrl, __ATOMIC_RELAXED, __HIP_MEMORY_SCOPE_WORKGROUP) != 0;
  __amdgpu_buffer_rsrc_t ors = __builtin_amdgcn_make_buffer_rsrc(p.out + (size_t)tm * BM * N + tn * BN, 0, BM * N * 4, 0x00020000);
  const unsigned ooff = (unsigned)((wm * 64 + r16) * N + wn * 64 + g * 4) * 4;
  #pragma unroll
  for (int mi = 0; mi < 4; ++mi) {
    #pragma unroll
    for (int ni = 0; ni < 4; ++ni) {
      v4f v = acc[ni][mi] * sc;
      v.x = fmaxf(v.x, 0.f); v.y = fmaxf(v.y, 0.f); v.z = fmaxf(v.z, 0.f); v.w = fmaxf(v.w, 0.f);
      if (bad) v = (v4f){__builtin_nanf(""), __builtin_nanf(""), __builtin_nanf(""), __builtin_nanf("")};
      __builtin_amdgcn_raw_buffer_store_b128(__builtin_bit_cast(v4u, v), ors, ooff + (mi * 16 * N + ni * 16) * 4, 0, OAUX);
    }
  }
}

extern "C" void kernel_launch(int rank, int nranks, ncclComm_t comm,
                              hipStream_t stream,
                              void* const* d_in, const int* in_sizes, int n_in,
                              void* d_out, int out_size,
                              const int* in_dtypes, int out_dtype,
                              void* const* d_out_all,
                              void* const* d_ws_all, size_t ws_bytes) {
  static bool init_a[NR];
  if (nranks != NR || ws_bytes < WS_NEED || (unsigned)rank >= (unsigned)NR) { fprintf(stderr, "bad config\n"); return; }
  if (!init_a[rank]) {
    HIP_CHECK(hipFuncSetAttribute((const void*)ag_gemm_kernel, hipFuncAttributeMaxDynamicSharedMemorySize, LDS_BYTES));
    init_a[rank] = true;
  }
  Params p;
  p.x = (const unsigned char*)d_in[0]; p.w = (const unsigned char*)d_in[1];
  p.sx = (const float*)d_in[2]; p.sw = (const float*)d_in[3]; p.out = (float*)d_out;
  for (int i = 0; i < NR; ++i) p.ws[i] = (unsigned char*)d_ws_all[i];
  p.rank = rank;
  ag_gemm_kernel<<<GEMM_WGS, NTHREADS, LDS_BYTES, stream>>>(p);
}
```

```cpp
#include <hip/hip_runtime.h>
#include <rccl/rccl.h>
#include <cstdio>
#include <cstdint>
#include <type_traits>

#define HIP_CHECK(expr) do { hipError_t _e = (expr); if (_e != hipSuccess) { \
    fprintf(stderr, "HIP error %s at %s:%d\n", hipGetErrorString(_e), __FILE__, __LINE__); \
    return; } } while (0)

typedef int v2i __attribute__((ext_vector_type(2)));
typedef int v4i __attribute__((ext_vector_type(4)));
typedef unsigned v4u __attribute__((ext_vector_type(4)));
typedef int v8i __attribute__((ext_vector_type(8)));
typedef float v4f __attribute__((ext_vector_type(4)));
#define LDSP(T) __attribute__((address_space(3))) T*

constexpr int NR = 8;
constexpr int M = 4096, N = 2048, K = 4096;
constexpr int MS = M / NR;
constexpr int BM = 256, BN = 128, BK = 128;
constexpr int A_BYTES = BM * BK, B_BYTES = BK * BN, STAGE = A_BYTES + B_BYTES, NST = 3;
constexpr int NKS = K / BK;
#define KCB 256
constexpr int KC = KCB;
constexpr int NCH = K / KC;
constexpr int KPC = KC / BK;
constexpr int LPR = KC / 16, RPS = 64 / LPR;
constexpr int PARTS = MS / (2 * RPS);
constexpr int ARRIVALS = PARTS / 2;
constexpr int GEMM_WGS = (M / BM) * (N / BN);
constexpr int NTHREADS = 768;
#define PFD 3
#define WLO 2u
#define WHI 14u
#define FLO 0
#define FHI 0
#define TAUQ 1000
#define TE0 100
#define PLW 2
#define OAUX 17
constexpr unsigned NPUSH = 256 + 32;
constexpr int LDS_CTRL = NST * STAGE;
constexpr int LDS_DUMMY = LDS_CTRL + 64;
constexpr int LDS_FLAG = LDS_DUMMY + 256;
constexpr int LDS_BYTES = LDS_FLAG + 8 * 256;
constexpr size_t WS_TMO = 0;
constexpr size_t WS_CNT = 4096;
constexpr size_t WS_PTICK = 69632;
constexpr size_t WS_PCALL = 524288;
constexpr int NITEM = NCH * PARTS;
constexpr size_t WS_STG = 1u << 20;
constexpr size_t STG_BYTES = (size_t)M * K;
constexpr size_t WS_NEED = WS_STG + 2 * STG_BYTES;
constexpr unsigned long long DEADLINE = 40000000ull;

struct Params {
  const unsigned char* x; const unsigned char* w; const float* sx; const float* sw; float* out;
  unsigned char* ws[NR];
  int rank;
  unsigned nw[NR - 1];
};

__device__ __forceinline__ void glds16(const void* sbase, unsigned voff, unsigned lds_off) {
  unsigned keep;
  asm volatile("s_mov_b32 %0, m0\n\ts_mov_b32 m0, %3\n\ts_nop 0\n\tglobal_load_lds_dwordx4 %1, %2\n\ts_mov_b32 m0, %0"
               : "=&s"(keep) : "v"(voff), "s"(sbase), "s"(lds_off) : "memory");
}

template <typename T> __device__ __forceinline__ T* uni(T* p) {
  unsigned long long v = (unsigned long long)p;
  unsigned lo = __builtin_amdgcn_readfirstlane((unsigned)v), hi = __builtin_amdgcn_readfirstlane((unsigned)(v >> 32));
  return (T*)(((unsigned long long)hi << 32) | lo);
}

__device__ __forceinline__ void glds4_sink(const void* vaddr, unsigned lds_off) {
  unsigned keep;
  asm volatile("s_mov_b32 %0, m0\n\ts_mov_b32 m0, %2\n\ts_nop 0\n\tglobal_load_lds_dword %1, off\n\ts_mov_b32 m0, %0"
               : "=&s"(keep) : "v"(vaddr), "s"(lds_off) : "memory");
}

__device__ __forceinline__ void glds4_flag(const void* sbase, unsigned lds_off) {
  unsigned keep;
  asm volatile("s_mov_b32 %0, m0\n\ts_mov_b32 m0, %3\n\ts_nop 0\n\tglobal_load_lds_dword %1, %2 sc0 sc1\n\ts_mov_b32 m0, %0"
               : "=&s"(keep) : "v"(0u), "s"(sbase), "s"(lds_off) : "memory");
}

__device__ __forceinline__ void wait_cnt(const unsigned* flag, unsigned target, unsigned* tmo, unsigned call) {
  unsigned long long t0 = __builtin_amdgcn_s_memrealtime();
  while (__hip_atomic_load(flag, __ATOMIC_RELAXED, __HIP_MEMORY_SCOPE_SYSTEM) < target) {
    if (__hip_atomic_load(tmo, __ATOMIC_RELAXED, __HIP_MEMORY_SCOPE_AGENT) == call ||
        __builtin_amdgcn_s_memrealtime() - t0 > DEADLINE) {
      __hip_atomic_store(tmo, call, __ATOMIC_RELAXED, __HIP_MEMORY_SCOPE_AGENT);
      break;
    }
    __builtin_amdgcn_s_sleep(1);
  }
  asm volatile("" ::: "memory");
}

__global__ void __launch_bounds__(NTHREADS, 3) ag_gemm_kernel(Params p) {
  extern __shared__ __attribute__((aligned(16))) unsigned char smem[];
  const int tid = threadIdx.x, l = tid & 63, w = __builtin_amdgcn_readfirstlane(tid >> 6);
  unsigned char* myws = p.ws[p.rank];
  const unsigned lds0 = (unsigned)(uintptr_t)smem;
  LDSP(unsigned) ctrl = (LDSP(unsigned))(uintptr_t)(lds0 + LDS_CTRL);

  const int bid = blockIdx.x, xcd = bid & 7, idx = bid >> 3;
  const int tm = (xcd >> 1) * 4 + (idx >> 3), tn = (xcd & 1) * 8 + (idx & 7);
  auto ldc = [&]() { return (unsigned)__builtin_amdgcn_readfirstlane(__hip_atomic_load(ctrl + 1, __ATOMIC_RELAXED, __HIP_MEMORY_SCOPE_WORKGROUP)); };
  auto sig = [&](unsigned inc) { if (l == 0) __hip_atomic_fetch_add(ctrl + 1, inc, __ATOMIC_RELAXED, __HIP_MEMORY_SCOPE_WORKGROUP); };
  const int src = tm >> 1;
  const bool own = (src == p.rank);

  if (w == 8 || (w == 11 && own)) {
    const unsigned v = (w == 8) ? (unsigned)bid : (unsigned)(GEMM_WGS + (tm & 1) * 16 + tn);
    const unsigned j = v % (NR - 1), nj = (NPUSH - j + NR - 2) / (NR - 1);
    const unsigned nw = p.nw[j];
    const unsigned loff = (unsigned)(l / LPR) * K + (l % LPR) * 16;
    auto item_off = [&](unsigned t) { return (t % PARTS) * 2 * RPS * K + (t / PARTS) * KC + loff; };
    unsigned* pcall = (unsigned*)(myws + WS_PCALL + ((w == 11 ? 256 : 0) + blockIdx.x) * 128);
    unsigned t0 = v / (NR - 1), t1;
    unsigned o0 = item_off(t0);
    v4u cur[2], nxt[2];
    unsigned n = __hip_atomic_load(pcall, __ATOMIC_RELAXED, __HIP_MEMORY_SCOPE_AGENT);
    #pragma unroll
    for (int i = 0; i < 2; ++i) cur[i] = *(const v4u*)(p.x + o0 + i * RPS * K);
    const unsigned long long tk0 = __builtin_amdgcn_s_memrealtime();
    __builtin_amdgcn_s_barrier();
    n = __builtin_amdgcn_readfirstlane(n);
    const unsigned parity = (n + 1) & 1;
    const unsigned dst = (unsigned)(p.rank + 1 + j) % NR;
    unsigned char* dws = p.ws[dst];
    __amdgpu_buffer_rsrc_t rs = __builtin_amdgcn_make_buffer_rsrc(dws + WS_STG + parity * STG_BYTES + (size_t)p.rank * MS * K, 0, MS * K, 0x00020000);
    unsigned vz; asm volatile("v_mov_b32 %0, 0" : "=v"(vz));
    unsigned* cnt = (unsigned*)(dws + WS_CNT + (size_t)p.rank * 2 * NCH * 128) + vz;
    unsigned* tick = (unsigned*)(myws + WS_PTICK + j * 128) + vz;
    unsigned cprev = ~0u;
    auto cidx = [&](unsigned t) { return ((t % PARTS) / (PARTS / 2)) * NCH + t / PARTS; };
#define PW(n) asm volatile("s_waitcnt vmcnt(%0)" :: "i"(n) : "memory")
#define PSIG() do { if (cprev != ~0u && l == 0) __hip_atomic_fetch_add(cnt + cprev * 32, 1u, __ATOMIC_RELAXED, __HIP_MEMORY_SCOPE_SYSTEM); } while (0)
    asm volatile("" :: "v"(cur[0]), "v"(cur[1]));
    const unsigned tbase = n * (NITEM + nw) - nj;
    if (own) {
      unsigned t2 = 0;
      if (l == 0) t2 = __hip_atomic_fetch_add(tick, 1u, __ATOMIC_RELAXED, __HIP_MEMORY_SCOPE_AGENT);
      t1 = __builtin_amdgcn_readfirstlane(t2) - tbase;
      while (t0 < (unsigned)NITEM) {
        const unsigned o1 = item_off(t1 < (unsigned)NITEM ? t1 : t0);
        if (l == 0) t2 = __hip_atomic_fetch_add(tick, 1u, __ATOMIC_RELAXED, __HIP_MEMORY_SCOPE_AGENT);
        #pragma unroll
        for (int i = 0; i < 2; ++i) nxt[i] = *(const v4u*)(p.x + o1 + i * RPS * K);
        PW(4);
        __builtin_amdgcn_raw_buffer_store_b128(cur[0], rs, o0, 0, 17);
        PW(4);
        PSIG();
        __builtin_amdgcn_raw_buffer_store_b128(cur[1], rs, o0 + RPS * K, 0, 17);
        asm volatile("" ::: "memory");
        cprev = cidx(t0); t0 = t1; o0 = o1;
        t1 = __builtin_amdgcn_readfirstlane(t2) - tbase;
        #pragma unroll
        for (int i = 0; i < 2; ++i) cur[i] = nxt[i];
        asm volatile("" :: "v"(cur[0]), "v"(cur[1]));
      }
    } else {
      unsigned t2 = 0;
      if (l == 0) t2 = __hip_atomic_fetch_add(tick, 1u, __ATOMIC_RELAXED, __HIP_MEMORY_SCOPE_AGENT);
      __builtin_amdgcn_raw_buffer_store_b128(cur[0], rs, o0, 0, 17);
      __builtin_amdgcn_raw_buffer_store_b128(cur[1], rs, o0 + RPS * K, 0, 17);
      cprev = cidx(t0);
      asm volatile("" : "+v"(t2) :: "memory");
      t0 = __builtin_amdgcn_readfirstlane(t2) - tbase;
      o0 = item_off(t0 < (unsigned)NITEM ? t0 : 0u);
      #pragma unroll
      for (int i = 0; i < 2; ++i) cur[i] = *(const v4u*)(p.x + o0 + i * RPS * K);
      if (t0 < (unsigned)NITEM && l == 0) t2 = __hip_atomic_fetch_add(tick, 1u, __ATOMIC_RELAXED, __HIP_MEMORY_SCOPE_AGENT);
      while (t0 < (unsigned)NITEM) {
        PW(0);
        asm volatile("" : "+v"(t2), "+v"(cur[0]), "+v"(cur[1]));
        t1 = __builtin_amdgcn_readfirstlane(t2) - tbase;
        const unsigned el = (unsigned)(__builtin_amdgcn_s_memrealtime() - tk0);
        const bool fast = (t1 * (TAUQ) / 16 + (TE0) < el) || t0 < (unsigned)(FLO) || t0 >= (unsigned)(NITEM - (FHI));
        const unsigned o1 = item_off(t1 < (unsigned)NITEM ? t1 : t0);
        PSIG();
        __builtin_amdgcn_raw_buffer_store_b128(cur[0], rs, o0, 0, 17);
        #pragma unroll
        for (int i = 0; i < 2; ++i) nxt[i] = *(const v4u*)(p.x + o1 + i * RPS * K);
        if (!fast) PW(2);
        __builtin_amdgcn_raw_buffer_store_b128(cur[1], rs, o0 + RPS * K, 0, 17);
        if (t1 < (unsigned)NITEM && l == 0) t2 = __hip_atomic_fetch_add(tick, 1u, __ATOMIC_RELAXED, __HIP_MEMORY_SCOPE_AGENT);
        asm volatile("" ::: "memory");
        cprev = cidx(t0); t0 = t1; o0 = o1;
        cur[0] = nxt[0]; cur[1] = nxt[1];
      }
    }
    PW(0);
    PSIG();
    if (l == 0) __hip_atomic_fetch_add(ctrl + 2, 1u, __ATOMIC_RELAXED, __HIP_MEMORY_SCOPE_WORKGROUP);
    if (l == 0) __hip_atomic_store(pcall, n + 1, __ATOMIC_RELAXED, __HIP_MEMORY_SCOPE_AGENT);
    return;
  }
  if (w == 11) {
    unsigned* pcall = (unsigned*)(myws + WS_PCALL + (256 + blockIdx.x) * 128);
    unsigned n = __hip_atomic_load(pcall, __ATOMIC_RELAXED, __HIP_MEMORY_SCOPE_AGENT);
    __builtin_amdgcn_s_barrier();
    __builtin_amdgcn_fence(__ATOMIC_ACQUIRE, "");
    asm volatile("s_waitcnt vmcnt(0)" ::: "memory");
    const unsigned call = __builtin_amdgcn_readfirstlane(n) + 1;
    const unsigned* cnt = (const unsigned*)(myws + WS_CNT + (size_t)tm * NCH * 128) + (l & (NCH - 1)) * 32;
    unsigned* tmo = (unsigned*)(myws + WS_TMO);
    const unsigned target = call * ARRIVALS;
    const unsigned tag = ((call & 31) + 1) << 8;
    unsigned have = 0;
    if (l == 0) __hip_atomic_store(ctrl + 3, tag, __ATOMIC_RELAXED, __HIP_MEMORY_SCOPE_WORKGROUP);
    const unsigned long long t0 = __builtin_amdgcn_s_memrealtime();
    while (have < (unsigned)NCH) {
      unsigned v = 0;
      if ((unsigned)l - have < (unsigned)(PLW) && l < NCH) v = __hip_atomic_load(cnt, __ATOMIC_RELAXED, __HIP_MEMORY_SCOPE_SYSTEM);
      unsigned long long m = __builtin_amdgcn_ballot_w64(v < target && (unsigned)l >= have) | (1ull << NCH);
      unsigned now = (unsigned)__builtin_ctzll(m);
      if (now > have) {
        have = now;
        if (l == 0) __hip_atomic_store(ctrl + 3, tag | have, __ATOMIC_RELAXED, __HIP_MEMORY_SCOPE_WORKGROUP);
      } else if (__builtin_amdgcn_s_memrealtime() - t0 > DEADLINE ||
                 ((__builtin_amdgcn_s_memrealtime() - t0 > 100000) && __hip_atomic_load(tmo, __ATOMIC_RELAXED, __HIP_MEMORY_SCOPE_AGENT) == call)) {
        if (l == 0) {
          __hip_atomic_store(tmo, call, __ATOMIC_RELAXED, __HIP_MEMORY_SCOPE_AGENT);
          __hip_atomic_store(ctrl, 1u, __ATOMIC_RELAXED, __HIP_MEMORY_SCOPE_WORKGROUP);
          asm volatile("s_waitcnt lgkmcnt(0)" ::: "memory");
          __hip_atomic_store(ctrl + 3, tag | NCH, __ATOMIC_RELAXED, __HIP_MEMORY_SCOPE_WORKGROUP);
        }
        break;
      }
    }
    if (l == 0) __hip_atomic_store(pcall, call, __ATOMIC_RELAXED, __HIP_MEMORY_SCOPE_AGENT);
    return;
  }
  if (w > 8) {
    const int h = w - 9;
    __builtin_amdgcn_s_barrier();
    auto ldr = [&]() { return (unsigned)__builtin_amdgcn_readfirstlane(__hip_atomic_load(ctrl + 3, __ATOMIC_RELAXED, __HIP_MEMORY_SCOPE_WORKGROUP)); };
    unsigned rdy = 0x100u | NCH;
    if (!own) do { rdy = ldr(); } while (rdy == 0);
    const unsigned parity = ((rdy >> 8) - 1) & 1;
    rdy &= 255u;
    const unsigned char* Xb = uni(own ? (p.x + (size_t)(tm & 1) * BM * K) : (myws + WS_STG + parity * STG_BYTES + (size_t)tm * BM * K));
    unsigned aoff[2];
    #pragma unroll
    for (int i = 0; i < 2; ++i) { int r = h * 128 + i * 8 + (l >> 3); int c = (l & 7) ^ (((l >> 4) & 1) | ((i & 1) * 6)); aoff[i] = r * K + c * 16; }
    const unsigned inc = h ? (1u << 22) : (1u << 12);
    int done = 0, buf = 0;
    for (int j = 0; j < NKS; ++j) {
      const unsigned need = (unsigned)(j / KPC + 1);
      if (rdy < need) rdy = ldr() & 255u;
      if (rdy < need || (j >= 3 && (ldc() & 0xfffu) < 8u * (j - 1))) {
        if (done < j) { asm volatile("s_waitcnt vmcnt(0)" ::: "memory"); sig(inc * (j - done)); done = j; }
        while (rdy < need) rdy = ldr() & 255u;
        while (j >= 3 && (ldc() & 0xfffu) < 8u * (j - 1)) {}
      }
      asm volatile("" ::: "memory");
      const unsigned char* xa = Xb + j * BK;
      const unsigned d = lds0 + buf * STAGE + h * 128 * 128;
      #pragma unroll
      for (int i = 0; i < 16; ++i) glds16(xa + (size_t)(i >> 1) * 16 * K, aoff[i & 1], d + i * 1024);
      if (done < j) { asm volatile("s_waitcnt vmcnt(16)" ::: "memory"); sig(inc * (j - done)); done = j; }
      buf = buf + 1; if (buf >= 3) buf = 0;
    }
    asm volatile("s_waitcnt vmcnt(0)" ::: "memory");
    sig(inc * (NKS - done));
    return;
  }

  const unsigned char* Wb = uni(p.w + (size_t)tn * BN);
  unsigned boff[2];
  #pragma unroll
  for (int i = 0; i < 2; ++i) { int k = w * 16 + i * 8 + (l >> 3); int c = (l & 7) ^ (((l >> 4) & 3) | (((w >> 1) & 1) << 2)); boff[i] = k * N + c * 16; }
  #pragma unroll
  for (int s = 0; s < 3; ++s)
    #pragma unroll
    for (int i = 0; i < 2; ++i) glds16(Wb + (size_t)s * BK * N, boff[i], lds0 + s * STAGE + A_BYTES + (w * 16 + i * 8) * 128);
  if (tid == 0) { ctrl[0] = 0; ctrl[1] = 0; ctrl[2] = 0; ctrl[3] = 0; }
  asm volatile("s_waitcnt lgkmcnt(0)" ::: "memory");
  __builtin_amdgcn_s_barrier();
  auto waitc = [&](unsigned c, unsigned la) {
    for (;;) { unsigned v = ldc(); if ((v & 0xfffu) >= c && ((v >> 12) & 0x3ffu) >= la && (v >> 22) >= la) break; }
    asm volatile("" ::: "memory");
  };
  const int wm = w >> 1, wn = w & 1, g = l >> 4, r16 = l & 15;
  const unsigned xs = ((r16 >> 1) & 1) | (((r16 >> 3) & 1) * 6);
  const unsigned xa0 = (wm * 64 + r16) * 128 + (((2 * g) ^ xs) * 16);
  const unsigned xa1 = xa0 ^ 16;
  const int q = r16 >> 1, pp = l & 1;
  const unsigned sB = ((q >> 1) & 3) | ((g & 1) << 2);
  unsigned wa[4];
  #pragma unroll
  for (int ni = 0; ni < 4; ++ni) wa[ni] = A_BYTES + (32 * g + q) * 128 + (((wn * 4 + ni) ^ sB) * 16) + 8 * pp;
  v4f acc[4][4];
  #pragma unroll
  for (int a = 0; a < 4; ++a)
    #pragma unroll
    for (int b = 0; b < 4; ++b) acc[a][b] = (v4f){0, 0, 0, 0};
  v8i xf[4], wf[4];
#define RW(ni, base) do { _Pragma("unroll") for (int i_ = 0; i_ < 4; ++i_) { \
    v2i t_ = __builtin_amdgcn_ds_read_tr8_b64_v2i32((LDSP(v2i))(uintptr_t)((base) + wa[ni] + i_ * 1024)); \
    wf[ni][2 * i_] = t_.x; wf[ni][2 * i_ + 1] = t_.y; } } while (0)
#define RX(mi, base) do { v4i t0_ = *(LDSP(v4i))(uintptr_t)((base) + xa0 + (mi) * 2048); v4i t1_ = *(LDSP(v4i))(uintptr_t)((base) + xa1 + (mi) * 2048); \
    xf[mi] = (v8i){t0_.x, t0_.y, t0_.z, t0_.w, t1_.x, t1_.y, t1_.z, t1_.w}; } while (0)
#define MM(mi, ni) acc[ni][mi] = __builtin_amdgcn_mfma_scale_f32_16x16x128_f8f6f4(wf[ni], xf[mi], acc[ni][mi], 1, 0, 0, 0, 0, 0)
#define SB __builtin_amdgcn_sched_barrier(0)
  asm volatile("s_waitcnt vmcnt(4)" ::: "memory");
  sig(1u);
  waitc(8u, 1u);
  RX(0, lds0); RX(1, lds0); RW(0, lds0); RW(1, lds0);
  int buf = 0;
  for (int s = 0; s < NKS - 1; ++s) {
    const unsigned base = lds0 + buf * STAGE;
    int nb = buf + 1; if (nb >= 3) nb = 0;
    const unsigned nbase = lds0 + nb * STAGE;
    SB; RW(2, base); RW(3, base); RX(2, base); RX(3, base); SB;
    MM(0, 0); MM(0, 1); MM(1, 0); MM(1, 1);
    MM(0, 2); MM(0, 3); MM(1, 2); MM(1, 3); SB;
    if (s + 2 < NKS) asm volatile("s_waitcnt vmcnt(2) lgkmcnt(0)" ::: "memory"); else asm volatile("s_waitcnt vmcnt(0) lgkmcnt(0)" ::: "memory");
    sig(1u);
    SB; MM(2, 0); MM(2, 1); SB;
    waitc(8u * (s + 2), s + 2);
    RX(0, nbase); RX(1, nbase);
    if (s + 3 < NKS) {
      const unsigned char* wb = Wb + (size_t)(s + 3) * BK * N;
      glds16(wb, boff[0], base + A_BYTES + (w * 16) * 128);
      glds16(wb, boff[1], base + A_BYTES + (w * 16 + 8) * 128);
    }
    SB; MM(3, 0); MM(3, 1); SB;
    RW(0, nbase); RW(1, nbase);
    SB; MM(2, 2); MM(2, 3); MM(3, 2); MM(3, 3); SB;
    buf = nb;
  }
  {
    const unsigned base = lds0 + buf * STAGE;
    RW(2, base); RW(3, base); RX(2, base); RX(3, base);
    MM(0, 0); MM(0, 1); MM(1, 0); MM(1, 1);
    MM(0, 2); MM(0, 3); MM(1, 2); MM(1, 3);
    MM(2, 0); MM(2, 1); MM(3, 0); MM(3, 1);
    MM(2, 2); MM(2, 3); MM(3, 2); MM(3, 3);
  }
  float sc = p.sx[0] * p.sw[0];
  while (__builtin_amdgcn_readfirstlane(__hip_atomic_load(ctrl + 2, __ATOMIC_RELAXED, __HIP_MEMORY_SCOPE_WORKGROUP)) < (own ? 2 : 1)) __builtin_amdgcn_s_sleep(2);
  const bool bad = __hip_atomic_load(ctrl, __ATOMIC_RELAXED, __HIP_MEMORY_SCOPE_WORKGROUP) != 0;
  __amdgpu_buffer_rsrc_t ors = __builtin_amdgcn_make_buffer_rsrc(p.out + (size_t)tm * BM * N + tn * BN, 0, BM * N * 4, 0x00020000);
  const unsigned ooff = (unsigned)((wm * 64 + r16) * N + wn * 64 + g * 4) * 4;
  #pragma unroll
  for (int mi = 0; mi < 4; ++mi) {
    #pragma unroll
    for (int ni = 0; ni < 4; ++ni) {
      v4f v = acc[ni][mi] * sc;
      v.x = fmaxf(v.x, 0.f); v.y = fmaxf(v.y, 0.f); v.z = fmaxf(v.z, 0.f); v.w = fmaxf(v.w, 0.f);
      if (bad) v = (v4f){__builtin_nanf(""), __builtin_nanf(""), __builtin_nanf(""), __builtin_nanf("")};
      __builtin_amdgcn_raw_buffer_store_b128(__builtin_bit_cast(v4u, v), ors, ooff + (mi * 16 * N + ni * 16) * 4, 0, OAUX);
    }
  }
}

extern "C" void kernel_launch(int rank, int nranks, ncclComm_t comm,
                              hipStream_t stream,
                              void* const* d_in, const int* in_sizes, int n_in,
                              void* d_out, int out_size,
                              const int* in_dtypes, int out_dtype,
                              void* const* d_out_all,
                              void* const* d_ws_all, size_t ws_bytes) {
  static bool init_a[NR];
  if (nranks != NR || ws_bytes < WS_NEED || (unsigned)rank >= (unsigned)NR) { fprintf(stderr, "bad config\n"); return; }
  if (!init_a[rank]) {
    HIP_CHECK(hipFuncSetAttribute((const void*)ag_gemm_kernel, hipFuncAttributeMaxDynamicSharedMemorySize, LDS_BYTES));
    init_a[rank] = true;
  }
  Params p;
  p.x = (const unsigned char*)d_in[0]; p.w = (const unsigned char*)d_in[1];
  p.sx = (const float*)d_in[2]; p.sw = (const float*)d_in[3]; p.out = (float*)d_out;
  for (int i = 0; i < NR; ++i) p.ws[i] = (unsigned char*)d_ws_all[i];
  p.rank = rank;
  for (int j = 0; j < NR - 1; ++j) p.nw[j] = 0;
  for (int i = 0; i < 32; ++i) {
    const int otm = rank * 2 + (i >> 4), otn = i & 15;
    const int ob = (((otm & 3) << 3) | (otn & 7)) * 8 + (otm >> 2) * 2 + (otn >> 3);
    p.nw[ob % (NR - 1)]++; p.nw[(GEMM_WGS + i) % (NR - 1)]++;
  }
  ag_gemm_kernel<<<GEMM_WGS, NTHREADS, LDS_BYTES, stream>>>(p);
}
```

```cpp
#include <hip/hip_runtime.h>
#include <rccl/rccl.h>
#include <cstdio>
#include <cstdint>

#define HIP_CHECK(expr) do { hipError_t _e = (expr); if (_e != hipSuccess) { \
    fprintf(stderr, "HIP error %s at %s:%d\n", hipGetErrorString(_e), __FILE__, __LINE__); \
    return; } } while (0)

typedef int v2i __attribute__((ext_vector_type(2)));
typedef int v4i __attribute__((ext_vector_type(4)));
typedef unsigned v4u __attribute__((ext_vector_type(4)));
typedef int v8i __attribute__((ext_vector_type(8)));
typedef float v4f __attribute__((ext_vector_type(4)));
#define LDSP(T) __attribute__((address_space(3))) T*

constexpr int NR = 8;
constexpr int M = 4096, N = 2048, K = 4096;
constexpr int MS = M / NR;
constexpr int BM = 256, BN = 128, BK = 128;
constexpr int X_BYTES = BM * BK, W_BYTES = BK * BN;
#define NSWM 5
constexpr int NSX = 2, NSW = NSWM;
constexpr int NKS = K / BK;
constexpr int KC = 256;
constexpr int NCH = K / KC;
#define SPL 1
constexpr int NCT = NCH + SPL;
constexpr int PARTS = 64;
constexpr int ARRIVALS = 4;
constexpr int GEMM_WGS = (M / BM) * (N / BN);
constexpr int NTHREADS = 768;
#define PWN 3
#define PAUX 17
#define PLW 3
#define OAUX 17
#define TL 1
#define HL TL
#define HMAJ 0
#define PL 1
#define WP 0
#define WPRE 2
constexpr unsigned NPUSH = 256 + 32;
constexpr int LDS_W = NSX * X_BYTES;
constexpr int LDS_CTRL = LDS_W + NSW * W_BYTES;
constexpr int LDS_DBG = LDS_CTRL + 128;
constexpr int LDS_BYTES = LDS_DBG + 512;
constexpr size_t WS_TMO = 0;
constexpr size_t WS_CNT = 4096;
constexpr size_t WS_PTICK = 69632;
constexpr size_t WS_PCALL = 524288;
constexpr int NITEM = NCH * PARTS;
constexpr size_t WS_STG = 1u << 20;
constexpr size_t STG_BYTES = (size_t)M * K;
constexpr size_t WS_NEED = WS_STG + 2 * STG_BYTES;
constexpr unsigned long long DEADLINE = 40000000ull;

struct Params {
  const unsigned char* x; const unsigned char* w; const float* sx; const float* sw; float* out;
  unsigned char* ws[NR];
  int rank;
};

__device__ __forceinline__ void glds16x4(const unsigned char* s0, const unsigned char* s1, const unsigned char* s2, const unsigned char* s3,
                                         unsigned v0, unsigned v1, unsigned v2, unsigned v3, unsigned lds_off) {
  asm volatile("s_mov_b32 m0, %8\n\ts_nop 0\n\tglobal_load_lds_dwordx4 %0, %4\n\tglobal_load_lds_dwordx4 %1, %5 offset:1024\n\t"
               "global_load_lds_dwordx4 %2, %6 offset:2048\n\tglobal_load_lds_dwordx4 %3, %7 offset:3072"
               :: "v"(v0), "v"(v1), "v"(v2), "v"(v3), "s"(s0), "s"(s1 - 1024), "s"(s2 - 2048), "s"(s3 - 3072), "s"(lds_off) : "memory", "m0");
}

template <typename T> __device__ __forceinline__ T* uni(T* p) {
  unsigned long long v = (unsigned long long)p;
  unsigned lo = __builtin_amdgcn_readfirstlane((unsigned)v), hi = __builtin_amdgcn_readfirstlane((unsigned)(v >> 32));
  return (T*)(((unsigned long long)hi << 32) | lo);
}

__global__ void __launch_bounds__(NTHREADS, 3) ag_gemm_kernel(Params p) {
  extern __shared__ __attribute__((aligned(128))) unsigned char smem[];
  const int tid = threadIdx.x, l = tid & 63, w = __builtin_amdgcn_readfirstlane(tid >> 6);
  unsigned char* myws = p.ws[p.rank];
  const unsigned lds0 = (unsigned)(uintptr_t)smem;
  LDSP(unsigned) ctrl = (LDSP(unsigned))(uintptr_t)(lds0 + LDS_CTRL);

  const int bid = blockIdx.x, xcd = bid & 7, idx = bid >> 3;
  const int tm = (xcd >> 1) * 4 + (idx >> 3), tn = (xcd & 1) * 8 + (idx & 7);
  const int src = tm >> 1;
  const bool own = (src == p.rank);

  if (w == 8 || (w == 11 && own)) {
    const unsigned v = (w == 8) ? (unsigned)bid : (unsigned)(GEMM_WGS + (tm & 1) * 16 + tn);
    const unsigned j = v % (NR - 1), nj = (NPUSH - j + NR - 2) / (NR - 1);
    const unsigned tfirst = v / (NR - 1);
    const unsigned loff = (unsigned)(l >> 4) * K + (l & 15) * 16;
    const unsigned loff2 = (unsigned)(l >> 3) * K + (l & 7) * 16;
    constexpr unsigned TB = NITEM - HL * PARTS;
    auto canon = [&](unsigned t) {
      if (t < TB) return t;
      const unsigned u = t - TB, b = u / (8 * TL), r = u % (8 * TL);
      return TB + (r >> 3) * 64 + b * 8 + (r & 7);
    };
    auto spl = [&](unsigned c) { return c >= (unsigned)(NITEM - PARTS); };
    auto item_off = [&](unsigned c) {
      return spl(c) ? (((c >> 1) & 1) * 256 + ((c >> 3) & 7) * 32 + (c & 1) * 16) * K + (NCH - 1) * KC + ((c >> 2) & 1) * 128 + loff2
                    : (((c >> 2) & 1) * 256 + ((c >> 3) & 7) * 32 + (c & 3) * 8) * K + (c / PARTS) * KC + loff; };
    auto item_st = [&](unsigned c) { return spl(c) ? 8u * K : 4u * K; };
    auto cidx = [&](unsigned c) {
      return spl(c) ? ((((c >> 1) & 1) * NCT + NCH - 1 + ((c >> 2) & 1)) * 32 + ((c >> 3) & 7)) | 0x20000u
                    : ((((c >> 2) & 1) * NCT + c / PARTS) * 32 + ((c >> 3) & 7)) | 0x10000u; };
    auto tk = [&](unsigned t) { t = __builtin_amdgcn_readfirstlane(t) + nj; return t < (unsigned)NITEM ? canon(t) : t; };
    unsigned* pcall = (unsigned*)(myws + WS_PCALL + ((w == 11 ? 256 : 0) + blockIdx.x) * 128);
    unsigned t0 = tfirst, t1;
    unsigned o0 = item_off(t0), o1;
    v4u cur[2], nxt[2];
    unsigned n = __hip_atomic_load(pcall, __ATOMIC_RELAXED, __HIP_MEMORY_SCOPE_AGENT);
    #pragma unroll
    for (int i = 0; i < 2; ++i) cur[i] = *(const v4u*)(p.x + o0 + i * 4 * K);
    __builtin_amdgcn_s_barrier();
    n = __builtin_amdgcn_readfirstlane(n);
    const unsigned parity = (n + 1) & 1;
    unsigned vz; asm volatile("v_mov_b32 %0, 0" : "=v"(vz));
    unsigned char* dws = p.ws[(unsigned)(p.rank + 1 + j) % NR];
    __amdgpu_buffer_rsrc_t rs = __builtin_amdgcn_make_buffer_rsrc(dws + WS_STG + parity * STG_BYTES + (size_t)p.rank * MS * K, 0, MS * K, 0x00020000);
    unsigned* cnt = (unsigned*)(dws + WS_CNT + (size_t)p.rank * 2 * NCT * 128) + vz;
    unsigned* tick = (unsigned*)(myws + WS_PTICK + parity * 1024 + j * 128) + vz;
    unsigned cprev;
#define PW(n) asm volatile("s_waitcnt vmcnt(%0)" :: "i"(n) : "memory")
#define PSIG() do { if (l == 0) __hip_atomic_fetch_add(cnt + (cprev & 0xffffu), cprev >> 16, __ATOMIC_RELAXED, __HIP_MEMORY_SCOPE_SYSTEM); } while (0)
#define DRAW(t) do { if (l == 0) t = __hip_atomic_fetch_add(tick, 1u, __ATOMIC_RELAXED, __HIP_MEMORY_SCOPE_AGENT); } while (0)
    unsigned t2 = 0, t3 = 0;
    asm volatile("" :: "v"(cur[0]), "v"(cur[1]));
    DRAW(t2);
    if (own) DRAW(t3);
    __builtin_amdgcn_raw_buffer_store_b128(cur[0], rs, o0, 0, PAUX);
    __builtin_amdgcn_raw_buffer_store_b128(cur[1], rs, o0 + 4 * K, 0, PAUX);
    if (v == 0 && l < 8)
      __hip_atomic_store((unsigned*)(myws + WS_PTICK + (parity ^ 1) * 1024) + l * 32, 0u, __ATOMIC_RELAXED, __HIP_MEMORY_SCOPE_SYSTEM);
    cprev = cidx(t0);
    asm volatile("" : "+v"(t2), "+v"(t3) :: "memory");
    t0 = tk(t2);
    o0 = item_off(t0 < (unsigned)NITEM ? t0 : 0u);
    unsigned s0 = item_st(t0 < (unsigned)NITEM ? t0 : 0u), s1;
    #pragma unroll
    for (int i = 0; i < 2; ++i) cur[i] = *(const v4u*)(p.x + o0 + i * s0);
    if (own) {
      t1 = tk(t3);
      asm volatile("" :: "v"(cur[0]), "v"(cur[1]));
      while (t0 < (unsigned)NITEM) {
        o1 = item_off(t1 < (unsigned)NITEM ? t1 : t0); s1 = item_st(t1 < (unsigned)NITEM ? t1 : t0);
        DRAW(t2);
        #pragma unroll
        for (int i = 0; i < 2; ++i) nxt[i] = *(const v4u*)(p.x + o1 + i * s1);
        PW(4);
        __builtin_amdgcn_raw_buffer_store_b128(cur[0], rs, o0, 0, PAUX);
        PW(4);
        PSIG();
        __builtin_amdgcn_raw_buffer_store_b128(cur[1], rs, o0, s0, PAUX);
        asm volatile("" ::: "memory");
        cprev = cidx(t0); t0 = t1; o0 = o1; s0 = s1;
        t1 = tk(t2);
        #pragma unroll
        for (int i = 0; i < 2; ++i) cur[i] = nxt[i];
        asm volatile("" :: "v"(cur[0]), "v"(cur[1]));
      }
    } else {
      DRAW(t2);
      while (t0 < (unsigned)NITEM) {
        PW(0);
        asm volatile("" : "+v"(t2), "+v"(cur[0]), "+v"(cur[1]));
        __builtin_amdgcn_raw_buffer_store_b128(cur[0], rs, o0, 0, PAUX);
        asm volatile("" ::: "memory");
        PSIG();
        t1 = tk(t2);
        o1 = item_off(t1 < (unsigned)NITEM ? t1 : t0); s1 = item_st(t1 < (unsigned)NITEM ? t1 : t0);
        #pragma unroll
        for (int i = 0; i < 2; ++i) nxt[i] = *(const v4u*)(p.x + o1 + i * s1);
        cprev = cidx(t0);
        PW(PWN);
        __builtin_amdgcn_raw_buffer_store_b128(cur[1], rs, o0, s0, PAUX);
        if (t1 < (unsigned)NITEM) DRAW(t2);
        asm volatile("" ::: "memory");
        t0 = t1; o0 = o1; s0 = s1;
        cur[0] = nxt[0]; cur[1] = nxt[1];
      }
    }
    PW(0);
    PSIG();
    if (l == 0) __hip_atomic_fetch_add(ctrl + 2, 1u, __ATOMIC_RELAXED, __HIP_MEMORY_SCOPE_WORKGROUP);
    if (l == 0) __hip_atomic_store(pcall, n + 1, __ATOMIC_RELAXED, __HIP_MEMORY_SCOPE_AGENT);
    return;
  }
  if (w == 11) {
    unsigned* pcall = (unsigned*)(myws + WS_PCALL + (256 + blockIdx.x) * 128);
    unsigned n = __hip_atomic_load(pcall, __ATOMIC_RELAXED, __HIP_MEMORY_SCOPE_AGENT);
    __builtin_amdgcn_s_barrier();
    __builtin_amdgcn_fence(__ATOMIC_ACQUIRE, "");
    asm volatile("s_waitcnt vmcnt(0)" ::: "memory");
    const unsigned call = __builtin_amdgcn_readfirstlane(n) + 1;
    const unsigned* cnt = (const unsigned*)(myws + WS_CNT + (size_t)tm * NCT * 128) + (l >> 3) * 32 + (l & 7);
    unsigned* tmo = (unsigned*)(myws + WS_TMO);
    const unsigned target = call * ARRIVALS;
    const unsigned tag = 0x100u | ((call & 1) << 9);
    LDSP(unsigned) pub = ctrl + 17 + 2 * (l & 7);
    if (l < 8) __hip_atomic_store(pub, tag, __ATOMIC_RELAXED, __HIP_MEMORY_SCOPE_WORKGROUP);
    unsigned base = 0, hv = 0;
    const unsigned long long t0 = __builtin_amdgcn_s_memrealtime();
    while (base < (unsigned)NCT) {
      bool ok = false;
      if (l < 8 * PLW && base + (l >> 3) < (unsigned)NCT) ok = __hip_atomic_load(cnt + base * 32, __ATOMIC_RELAXED, __HIP_MEMORY_SCOPE_SYSTEM) >= target;
      const unsigned long long m = __builtin_amdgcn_ballot_w64(ok);
      const unsigned now = base + ((unsigned)__builtin_ctzll(~(m >> (l & 7)) & 0x0101010101010101ull) >> 3);
      if (__builtin_amdgcn_ballot_w64(now > hv)) {
        hv = now > hv ? now : hv;
        if (l < 8) __hip_atomic_store(pub, tag | (hv < (unsigned)NCH ? 2 * hv : NCH - 1 + hv), __ATOMIC_RELAXED, __HIP_MEMORY_SCOPE_WORKGROUP);
        unsigned k = 0;
        while (k < (unsigned)PLW && ((m >> (8 * k)) & 255u) == 255u) ++k;
        base += k;
      } else if (__builtin_amdgcn_s_memrealtime() - t0 > DEADLINE ||
                 ((__builtin_amdgcn_s_memrealtime() - t0 > 100000) && __hip_atomic_load(tmo, __ATOMIC_RELAXED, __HIP_MEMORY_SCOPE_AGENT) == call)) {
        if (l == 0) {
          __hip_atomic_store(tmo, call, __ATOMIC_RELAXED, __HIP_MEMORY_SCOPE_AGENT);
          __hip_atomic_store(ctrl, 1u, __ATOMIC_RELAXED, __HIP_MEMORY_SCOPE_WORKGROUP);
        }
        asm volatile("s_waitcnt lgkmcnt(0)" ::: "memory");
        if (l < 8) __hip_atomic_store(pub, tag | NKS, __ATOMIC_RELAXED, __HIP_MEMORY_SCOPE_WORKGROUP);
        break;
      }
    }
    if (l == 0) __hip_atomic_store(pcall, call, __ATOMIC_RELAXED, __HIP_MEMORY_SCOPE_AGENT);
    return;
  }
  if (w > 8) {
    const int h = w - 9;
    const unsigned char* Wb = uni(p.w + (size_t)tn * BN + (size_t)h * 64 * N);
    unsigned boff[2];
    #pragma unroll
    for (int i = 0; i < 2; ++i) boff[i] = (l >> 3) * N + (((l & 7) ^ (((l >> 4) & 3) | (i << 2))) * 16);
    auto ldw = [&](int j, int b) {
      const unsigned char* wb = Wb + (size_t)j * BK * N;
      #pragma unroll
      for (int i = 0; i < 8; i += 4) glds16x4(wb + (size_t)i * 8 * N, wb + (size_t)(i + 1) * 8 * N, wb + (size_t)(i + 2) * 8 * N, wb + (size_t)(i + 3) * 8 * N,
                                              boff[i >> 2], boff[i >> 2], boff[i >> 2], boff[i >> 2], lds0 + LDS_W + b * W_BYTES + h * 8192 + i * 1024);
    };
    const unsigned inc = h ? (1u << 16) : 1u;
    auto sig = [&](unsigned k) { if (l < 8) __hip_atomic_fetch_add(ctrl + 16 + 2 * l, inc * k, __ATOMIC_RELAXED, __HIP_MEMORY_SCOPE_WORKGROUP); };
    auto ldf = [&](int b) { return (unsigned)__builtin_amdgcn_readfirstlane(__hip_atomic_load(ctrl + 8 + b, __ATOMIC_RELAXED, __HIP_MEMORY_SCOPE_WORKGROUP)); };
    #pragma unroll
    for (int j = 0; j < WPRE; ++j) ldw(j, j);
    __builtin_amdgcn_s_barrier();
    int done = 0, buf = WPRE;
    unsigned fr = 0;
    if (WPRE > 1) { asm volatile("s_waitcnt vmcnt(%0)" :: "i"(8 * (WPRE - 1)) : "memory"); sig(1); done = 1; }
    for (int j = WPRE; j < NKS; ++j) {
      if (buf >= NSW) { buf = 0; fr += 8; }
      if (fr && ldf(buf) < fr) {
        if (done < j) { asm volatile("s_waitcnt vmcnt(0)" ::: "memory"); sig(j - done); done = j; }
        while (ldf(buf) < fr) {}
      }
      asm volatile("" ::: "memory");
      ldw(j, buf);
      if (done < j) { asm volatile("s_waitcnt vmcnt(8)" ::: "memory"); sig(j - done); done = j; }
      ++buf;
    }
    asm volatile("s_waitcnt vmcnt(0)" ::: "memory");
    sig(NKS - done);
    return;
  }

  if (tid < 32) ctrl[tid] = (own && tid > 16 && (tid & 1)) ? (0x100u | NKS) : 0u;
  asm volatile("s_waitcnt lgkmcnt(0)" ::: "memory");
  __builtin_amdgcn_s_barrier();
  const int g = l >> 4, r16 = l & 15;
  const unsigned xs = ((r16 >> 1) & 1) | (((r16 >> 3) & 1) * 6);
  const unsigned xa0 = lds0 + (w * 32 + r16) * 128 + (((2 * g) ^ xs) * 16);
  const int q = r16 >> 1, pp = l & 1;
  const unsigned sB = ((q >> 1) & 3) | ((g & 1) << 2);
  const unsigned wa0 = lds0 + LDS_W + (32 * g + q) * 128 + sB * 16 + 8 * pp;
  LDSP(unsigned long long) flg = (LDSP(unsigned long long))(uintptr_t)(lds0 + LDS_CTRL + 64 + w * 8);
  auto ldflg = [&]() { return __hip_atomic_load(flg, __ATOMIC_RELAXED, __HIP_MEMORY_SCOPE_WORKGROUP); };
  v4f acc[8][2];
  #pragma unroll
  for (int a = 0; a < 8; ++a)
    #pragma unroll
    for (int b = 0; b < 2; ++b) acc[a][b] = (v4f){0, 0, 0, 0};
  v8i xf[2], wf[4];
#define RW(sl, ni, adr) do { _Pragma("unroll") for (int i_ = 0; i_ < 4; ++i_) { \
    v2i t_ = __builtin_amdgcn_ds_read_tr8_b64_v2i32((LDSP(v2i))(uintptr_t)(((adr) ^ ((ni) * 16)) + i_ * 1024)); \
    wf[sl][2 * i_] = t_.x; wf[sl][2 * i_ + 1] = t_.y; } } while (0)
#define RX(mi, adr) do { v4i t0_ = *(LDSP(v4i))(uintptr_t)((adr) + (mi) * 2048); v4i t1_ = *(LDSP(v4i))(uintptr_t)(((adr) ^ 16) + (mi) * 2048); \
    xf[mi] = (v8i){t0_.x, t0_.y, t0_.z, t0_.w, t1_.x, t1_.y, t1_.z, t1_.w}; } while (0)
#define MM(mi, ni) acc[ni][mi] = __builtin_amdgcn_mfma_scale_f32_16x16x128_f8f6f4(xf[mi], wf[(ni) & 3], acc[ni][mi], 0, 1, 0, 0, 0, 0)
#define SB __builtin_amdgcn_sched_barrier(0)
  float sc = __builtin_bit_cast(float, __builtin_amdgcn_readfirstlane(__builtin_bit_cast(int, p.sx[0] * p.sw[0])));
  asm volatile("" : "+s"(sc));
  float* outp = uni(p.out + (size_t)tm * BM * N + tn * BN);
  asm volatile("" : "+s"(outp));
  __amdgpu_buffer_rsrc_t ors = __builtin_amdgcn_make_buffer_rsrc(outp, 0, BM * N * 4, 0x00020000);
  unsigned have, wl;
  do { unsigned long long f = ldflg(); have = __builtin_amdgcn_readfirstlane((unsigned)(f >> 32)); } while (have == 0);
  const unsigned char* Xb = uni((own ? (p.x + (size_t)(tm & 1) * BM * K) : (myws + WS_STG + ((have >> 9) & 1) * STG_BYTES + (size_t)tm * BM * K)) + (size_t)w * 32 * K);
  have &= 255u;
  unsigned aoff[2];
  #pragma unroll
  for (int i = 0; i < 2; ++i) { int c = (l & 7) ^ (((l >> 4) & 1) | (i * 6)); aoff[i] = (i * 8 + (l >> 3)) * K + c * 16; }
  unsigned nx = 0;
  auto ldx = [&]() {
    const unsigned char* xa = Xb + nx * BK;
    glds16x4(xa, xa, xa + 16 * K, xa + 16 * K, aoff[0], aoff[1], aoff[0], aoff[1], lds0 + (nx & 1) * X_BYTES + w * 4096);
    ++nx;
  };
  auto rdf = [&](unsigned long long f) { have = __builtin_amdgcn_readfirstlane((unsigned)(f >> 32)) & 255u; wl = __builtin_amdgcn_readfirstlane((unsigned)f); };
  auto wlanded = [&](unsigned la) { return (wl & 0xffffu) >= la && (wl >> 16) >= la; };
  auto needx = [&](unsigned s) {
    while (nx <= s) { if (nx < have) ldx(); else rdf(ldflg()); }
    if (nx == s + 1 && nx < have) ldx();
    if (nx == s + 1) asm volatile("s_waitcnt vmcnt(0)" ::: "memory"); else asm volatile("s_waitcnt vmcnt(4)" ::: "memory");
  };
  const unsigned one = 1u;
  needx(0);
  RX(0, xa0); RX(1, xa0);
  do rdf(ldflg()); while (!wlanded(1));
  asm volatile("" ::: "memory");
  RW(0, 0, wa0); RW(1, 1, wa0); RW(2, 2, wa0);
  int buf = 0;
  for (unsigned s = 0; s < NKS - 1; ++s) {
    const unsigned wa = wa0 + buf * W_BYTES;
    int nb = buf + 1; if (nb >= NSW) nb = 0;
    const unsigned wn = wa0 + nb * W_BYTES;
    SB; RW(3, 3, wa); SB;
    MM(0, 0); MM(1, 0); SB;
    if (nx <= s + 2 && nx < have) ldx();
    SB; RW(0, 4, wa); SB;
    MM(0, 1); MM(1, 1); SB;
    const unsigned long long f = ldflg();
    RW(1, 5, wa); SB;
    MM(0, 2); MM(1, 2); SB;
    RW(2, 6, wa); SB;
    MM(0, 3); MM(1, 3); SB;
    RW(3, 7, wa); SB;
    MM(0, 4); MM(1, 4); SB;
    rdf(f);
    if (nx <= s + 2 && nx < have) ldx();
    while (!wlanded(s + 2)) rdf(ldflg());
    asm volatile("" ::: "memory");
    SB; RW(0, 0, wn); SB;
    MM(0, 5); MM(1, 5); SB;
    RW(1, 1, wn); SB;
    MM(0, 6); MM(1, 6); SB;
    RW(2, 2, wn); SB;
    MM(0, 7); SB;
    asm volatile("s_mov_b64 exec, 1\n\tds_add_u32 %0, %1\n\ts_mov_b64 exec, -1" :: "v"(lds0 + LDS_CTRL + 32 + buf * 4), "v"(one) : "memory");
    SB; MM(1, 7); SB;
    needx(s + 1);
    const unsigned xa = xa0 + ((s + 1) & 1) * X_BYTES;
    RX(0, xa); RX(1, xa);
    buf = nb;
  }
  const unsigned ooff = (unsigned)((w * 32 + (l >> 5) * 8) * N + (l & 31)) * 4;
#define ST2(mi, nj) do { v4f a_ = acc[2 * (nj)][mi] * sc, b_ = acc[2 * (nj) + 1][mi] * sc; \
    _Pragma("unroll") for (int e_ = 0; e_ < 4; ++e_) { \
      auto r_ = __builtin_amdgcn_permlane16_swap(__builtin_bit_cast(unsigned, fmaxf(a_[e_], 0.f)), __builtin_bit_cast(unsigned, fmaxf(b_[e_], 0.f)), false, false); \
      __builtin_amdgcn_raw_buffer_store_b32(r_[0], ors, ooff + (nj) * 128, ((mi) * 16 + e_) * N * 4, OAUX); \
      __builtin_amdgcn_raw_buffer_store_b32(r_[1], ors, ooff + (nj) * 128, ((mi) * 16 + 4 + e_) * N * 4, OAUX); } } while (0)
#define WAITP() do { if ((WP & 1) && own || (WP & 2) && !own) while (__builtin_amdgcn_readfirstlane(__hip_atomic_load(ctrl + 2, __ATOMIC_RELAXED, __HIP_MEMORY_SCOPE_WORKGROUP)) < (own ? 2 : 1)) __builtin_amdgcn_s_sleep(2); } while (0)
  {
    const unsigned wa = wa0 + buf * W_BYTES;
    SB; RW(3, 3, wa); SB;
    MM(0, 0); MM(1, 0); SB;
    RW(0, 4, wa); SB;
    MM(0, 1); MM(1, 1); SB;
    RW(1, 5, wa); SB;
    MM(0, 2); MM(1, 2); SB;
    RW(2, 6, wa); SB;
    MM(0, 3); MM(1, 3); SB;
    RW(3, 7, wa); SB;
    WAITP();
    SB;
    MM(0, 4); MM(1, 4); ST2(0, 0); SB;
    MM(0, 5); MM(1, 5); ST2(1, 0); SB;
    MM(0, 6); MM(1, 6); ST2(0, 1); SB;
    MM(0, 7); MM(1, 7); ST2(1, 1); SB;
    ST2(0, 2); ST2(1, 2); ST2(0, 3); ST2(1, 3);
  }
  if (__builtin_amdgcn_readfirstlane(__hip_atomic_load(ctrl, __ATOMIC_RELAXED, __HIP_MEMORY_SCOPE_WORKGROUP))) {
    for (int i = 0; i < 64; ++i) __builtin_amdgcn_raw_buffer_store_b32(0x7fc00000u, ors, (unsigned)((w * 32 + (i >> 1)) * N + (i & 1) * 64 + l) * 4, 0, OAUX);
  }
}

extern "C" void kernel_launch(int rank, int nranks, ncclComm_t comm,
                              hipStream_t stream,
                              void* const* d_in, const int* in_sizes, int n_in,
                              void* d_out, int out_size,
                              const int* in_dtypes, int out_dtype,
                              void* const* d_out_all,
                              void* const* d_ws_all, size_t ws_bytes) {
  static bool init_a[NR];
  if (nranks != NR || ws_bytes < WS_NEED || (unsigned)rank >= (unsigned)NR) { fprintf(stderr, "bad config\n"); return; }
  if (!init_a[rank]) {
    HIP_CHECK(hipFuncSetAttribute((const void*)ag_gemm_kernel, hipFuncAttributeMaxDynamicSharedMemorySize, LDS_BYTES));
    init_a[rank] = true;
  }
  Params p;
  p.x = (const unsigned char*)d_in[0]; p.w = (const unsigned char*)d_in[1];
  p.sx = (const float*)d_in[2]; p.sw = (const float*)d_in[3]; p.out = (float*)d_out;
  for (int i = 0; i < NR; ++i) p.ws[i] = (unsigned char*)d_ws_all[i];
  p.rank = rank;
  ag_gemm_kernel<<<GEMM_WGS, NTHREADS, LDS_BYTES, stream>>>(p);
}
```

```cpp
#include <hip/hip_runtime.h>
#include <rccl/rccl.h>
#include <cstdio>
#include <cstdint>

#define HIP_CHECK(expr) do { hipError_t _e = (expr); if (_e != hipSuccess) { \
    fprintf(stderr, "HIP error %s at %s:%d\n", hipGetErrorString(_e), __FILE__, __LINE__); \
    return; } } while (0)

typedef int v2i __attribute__((ext_vector_type(2)));
typedef int v4i __attribute__((ext_vector_type(4)));
typedef unsigned v4u __attribute__((ext_vector_type(4)));
typedef int v8i __attribute__((ext_vector_type(8)));
typedef float v4f __attribute__((ext_vector_type(4)));
#define LDSP(T) __attribute__((address_space(3))) T*

constexpr int NR = 8;
constexpr int M = 4096, N = 2048, K = 4096;
constexpr int MS = M / NR;
constexpr int BM = 256, BN = 128, BK = 128;
constexpr int X_BYTES = BM * BK, W_BYTES = BK * BN;
#define NSWM 5
constexpr int NSX = 2, NSW = NSWM;
constexpr int NKS = K / BK;
constexpr int KC = 256;
constexpr int NCH = K / KC;
#define SPL 1
constexpr int NCT = NCH + SPL;
constexpr int PARTS = 64;
constexpr int ARRIVALS = 4;
constexpr int GEMM_WGS = (M / BM) * (N / BN);
constexpr int NTHREADS = 768;
#define PWN 3
#define PAUX 17
#define PLW 3
#define OAUX 17
#define TL 1
#define HL TL
#define HMAJ 0
#define PL 1
#define WP 0
#define NWV 8
#define STID ((NWV - 1) * 64)
#define LAY 0
#define SI 3
#define WPRE 2
constexpr unsigned NPUSH = 256 + 32;
constexpr int LDS_W = NSX * X_BYTES;
constexpr int LDS_CTRL = LDS_W + NSW * W_BYTES;
constexpr int LDS_DBG = LDS_CTRL + 128;
constexpr int LDS_BYTES = LDS_DBG + 768;
constexpr size_t WS_TMO = 0;
constexpr size_t WS_CNT = 4096;
constexpr size_t WS_PTICK = 69632;
constexpr size_t WS_PCALL = 524288;
constexpr int NITEM = NCH * PARTS;
constexpr size_t WS_STG = 1u << 20;
constexpr size_t STG_BYTES = (size_t)M * K;
constexpr size_t WS_NEED = WS_STG + 2 * STG_BYTES;
constexpr unsigned long long DEADLINE = 40000000ull;

struct Params {
  const unsigned char* x; const unsigned char* w; const float* sx; const float* sw; float* out;
  unsigned char* ws[NR];
  int rank;
};

__device__ __forceinline__ void glds16x4(const unsigned char* s0, const unsigned char* s1, const unsigned char* s2, const unsigned char* s3,
                                         unsigned v0, unsigned v1, unsigned v2, unsigned v3, unsigned lds_off) {
  asm volatile("s_mov_b32 m0, %8\n\ts_nop 0\n\tglobal_load_lds_dwordx4 %0, %4\n\tglobal_load_lds_dwordx4 %1, %5 offset:1024\n\t"
               "global_load_lds_dwordx4 %2, %6 offset:2048\n\tglobal_load_lds_dwordx4 %3, %7 offset:3072"
               :: "v"(v0), "v"(v1), "v"(v2), "v"(v3), "s"(s0), "s"(s1 - 1024), "s"(s2 - 2048), "s"(s3 - 3072), "s"(lds_off) : "memory", "m0");
}

template <typename T> __device__ __forceinline__ T* uni(T* p) {
  unsigned long long v = (unsigned long long)p;
  unsigned lo = __builtin_amdgcn_readfirstlane((unsigned)v), hi = __builtin_amdgcn_readfirstlane((unsigned)(v >> 32));
  return (T*)(((unsigned long long)hi << 32) | lo);
}

__global__ void __launch_bounds__(NTHREADS, 3) ag_gemm_kernel(Params p) {
  extern __shared__ __attribute__((aligned(128))) unsigned char smem[];
  const int tid = threadIdx.x, l = tid & 63, w = __builtin_amdgcn_readfirstlane(tid >> 6);
  unsigned char* myws = p.ws[p.rank];
  const unsigned lds0 = (unsigned)(uintptr_t)smem;
  LDSP(unsigned) ctrl = (LDSP(unsigned))(uintptr_t)(lds0 + LDS_CTRL);

  const int bid = blockIdx.x, xcd = bid & 7, idx = bid >> 3;
  const int tm = (xcd >> 1) * 4 + (idx >> 3), tn = (xcd & 1) * 8 + (idx & 7);
  const int src = tm >> 1;
  const bool own = (src == p.rank);

  if (w == 8 || (w == 11 && own)) {
    const unsigned v = (w == 8) ? (unsigned)bid : (unsigned)(GEMM_WGS + (tm & 1) * 16 + tn);
    const unsigned j = v % (NR - 1), nj = (NPUSH - j + NR - 2) / (NR - 1);
    const unsigned tfirst = v / (NR - 1);
    const unsigned loff = (unsigned)(l >> 4) * K + (l & 15) * 16;
    const unsigned loff2 = (unsigned)(l >> 3) * K + (l & 7) * 16;
    constexpr unsigned TB = NITEM - HL * PARTS;
    auto canon = [&](unsigned t) {
      if (t < TB) return t;
      const unsigned u = t - TB, b = u / (8 * TL), r = u % (8 * TL);
      return TB + (r >> 3) * 64 + b * 8 + (r & 7);
    };
    auto spl = [&](unsigned c) { return c >= (unsigned)(NITEM - PARTS); };
    auto item_off = [&](unsigned c) {
      return spl(c) ? (((c >> 1) & 1) * 256 + ((c >> 3) & 7) * 32 + (c & 1) * 16) * K + (NCH - 1) * KC + ((c >> 2) & 1) * 128 + loff2
                    : (((c >> 2) & 1) * 256 + ((c >> 3) & 7) * 32 + (c & 3) * 8) * K + (c / PARTS) * KC + loff; };
    auto item_st = [&](unsigned c) { return spl(c) ? 8u * K : 4u * K; };
#define PST(d, t, i) __builtin_amdgcn_raw_buffer_store_b128(d, rs, o0, (i) * item_st(t), PAUX)
    auto cidx = [&](unsigned c) {
      return spl(c) ? ((((c >> 1) & 1) * NCT + NCH - 1 + ((c >> 2) & 1)) * 32 + ((c >> 3) & 7)) | 0x20000u
                    : ((((c >> 2) & 1) * NCT + c / PARTS) * 32 + ((c >> 3) & 7)) | 0x10000u; };
    auto tk = [&](unsigned t) { t = __builtin_amdgcn_readfirstlane(t) + SI * nj; return t < (unsigned)NITEM ? canon(t) : t; };
    unsigned* pcall = (unsigned*)(myws + WS_PCALL + ((w == 11 ? 256 : 0) + blockIdx.x) * 128);
    unsigned t0 = tfirst, t1;
    unsigned o0 = item_off(t0), o1;
    v4u cur[2], nxt[2];
    unsigned n = __hip_atomic_load(pcall, __ATOMIC_RELAXED, __HIP_MEMORY_SCOPE_AGENT);
    #pragma unroll
    for (int i = 0; i < 2; ++i) cur[i] = *(const v4u*)(p.x + o0 + i * item_st(t0));
    o1 = item_off(t0 + nj);
    #pragma unroll
    for (int i = 0; i < 2; ++i) nxt[i] = *(const v4u*)(p.x + o1 + i * item_st(t0));
    __builtin_amdgcn_s_barrier();
    n = __builtin_amdgcn_readfirstlane(n);
    const unsigned parity = (n + 1) & 1;
    unsigned vz; asm volatile("v_mov_b32 %0, 0" : "=v"(vz));
    unsigned char* dws = p.ws[(unsigned)(p.rank + 1 + j) % NR];
    __amdgpu_buffer_rsrc_t rs = __builtin_amdgcn_make_buffer_rsrc(dws + WS_STG + parity * STG_BYTES + (size_t)p.rank * MS * K, 0, MS * K, 0x00020000);
    unsigned* cnt = (unsigned*)(dws + WS_CNT + (size_t)p.rank * 2 * NCT * 128) + vz;
    unsigned* tick = (unsigned*)(myws + WS_PTICK + parity * 1024 + j * 128) + vz;
    unsigned cprev;
#define PW(n) asm volatile("s_waitcnt vmcnt(%0)" :: "i"(n) : "memory")
#define PSIG() do { if (l == 0) __hip_atomic_fetch_add(cnt + (cprev & 0xffffu), cprev >> 16, __ATOMIC_RELAXED, __HIP_MEMORY_SCOPE_SYSTEM); } while (0)
#define DRAW(t) do { if (l == 0) t = __hip_atomic_fetch_add(tick, 1u, __ATOMIC_RELAXED, __HIP_MEMORY_SCOPE_AGENT); } while (0)
    unsigned t2 = 0, t3 = 0;
    asm volatile("" :: "v"(cur[0]), "v"(cur[1]));
#define PS(i)
    PS(0);
    asm volatile("" :: "v"(nxt[0]), "v"(nxt[1]));
    PST(cur[0], t0, 0);
    PST(cur[1], t0, 1);
    if (v == 0 && l < 8)
      __hip_atomic_store((unsigned*)(myws + WS_PTICK + (parity ^ 1) * 1024) + l * 32, 0u, __ATOMIC_RELAXED, __HIP_MEMORY_SCOPE_SYSTEM);
    cprev = cidx(t0);
    t2 = t3 = tfirst - nj;
    asm volatile("" : "+v"(t2), "+v"(t3) :: "memory");
    t0 += nj; o0 = o1;
    PS(1);
    unsigned s0 = item_st(t0), s1;
    cur[0] = nxt[0]; cur[1] = nxt[1];
    if (own) {
      t1 = tk(t3);
      asm volatile("" :: "v"(cur[0]), "v"(cur[1]));
      while (t0 < (unsigned)NITEM) {
        o1 = item_off(t1 < (unsigned)NITEM ? t1 : t0); s1 = item_st(t1 < (unsigned)NITEM ? t1 : t0);
        DRAW(t2);
        #pragma unroll
        for (int i = 0; i < 2; ++i) nxt[i] = *(const v4u*)(p.x + o1 + i * s1);
        PW(4);
        PST(cur[0], t0, 0);
        PW(4);
        PSIG();
        PST(cur[1], t0, 1);
        asm volatile("" ::: "memory");
        cprev = cidx(t0); t0 = t1; o0 = o1; s0 = s1;
        t1 = tk(t2);
        #pragma unroll
        for (int i = 0; i < 2; ++i) cur[i] = nxt[i];
        asm volatile("" :: "v"(cur[0]), "v"(cur[1]));
      }
    } else {
      while (t0 < (unsigned)NITEM) {
        PW(0);
        asm volatile("" : "+v"(t2), "+v"(cur[0]), "+v"(cur[1]));
        PST(cur[0], t0, 0);
        asm volatile("" ::: "memory");
        PSIG();
        t1 = tk(t2);
        o1 = item_off(t1 < (unsigned)NITEM ? t1 : t0); s1 = item_st(t1 < (unsigned)NITEM ? t1 : t0);
        #pragma unroll
        for (int i = 0; i < 2; ++i) nxt[i] = *(const v4u*)(p.x + o1 + i * s1);
        cprev = cidx(t0);
        PW(PWN);
        PST(cur[1], t0, 1);
        if (t1 < (unsigned)NITEM) DRAW(t2);
        asm volatile("" ::: "memory");
        t0 = t1; o0 = o1; s0 = s1;
        cur[0] = nxt[0]; cur[1] = nxt[1];
      }
    }
    PW(0);
    PSIG();
    if (l == 0) __hip_atomic_fetch_add(ctrl + 2, 1u, __ATOMIC_RELAXED, __HIP_MEMORY_SCOPE_WORKGROUP);
    if (l == 0) __hip_atomic_store(pcall, n + 1, __ATOMIC_RELAXED, __HIP_MEMORY_SCOPE_AGENT);
    return;
  }
  if (w == 11) {
    unsigned* pcall = (unsigned*)(myws + WS_PCALL + (256 + blockIdx.x) * 128);
    unsigned n = __hip_atomic_load(pcall, __ATOMIC_RELAXED, __HIP_MEMORY_SCOPE_AGENT);
    __builtin_amdgcn_s_barrier();
    __builtin_amdgcn_fence(__ATOMIC_ACQUIRE, "");
    asm volatile("s_waitcnt vmcnt(0)" ::: "memory");
    const unsigned call = __builtin_amdgcn_readfirstlane(n) + 1;
    const unsigned* cnt = (const unsigned*)(myws + WS_CNT + (size_t)tm * NCT * 128) + (l >> 3) * 32 + (l & 7);
    unsigned* tmo = (unsigned*)(myws + WS_TMO);
    const unsigned target = call * ARRIVALS;
    const unsigned tag = 0x100u | ((call & 1) << 9);
    LDSP(unsigned) pub = ctrl + 17 + 2 * (l & 7);
    if (l < 8) __hip_atomic_store(pub, tag, __ATOMIC_RELAXED, __HIP_MEMORY_SCOPE_WORKGROUP);
    unsigned base = 0, hv = 0;
    const unsigned long long t0 = __builtin_amdgcn_s_memrealtime();
    while (base < (unsigned)NCT) {
      bool ok = false;
      if (l < 8 * PLW && base + (l >> 3) < (unsigned)NCT) ok = __hip_atomic_load(cnt + base * 32, __ATOMIC_RELAXED, __HIP_MEMORY_SCOPE_SYSTEM) >= target;
      const unsigned long long m = __builtin_amdgcn_ballot_w64(ok);
      const unsigned now = base + ((unsigned)__builtin_ctzll(~(m >> (l & 7)) & 0x0101010101010101ull) >> 3);
      if (__builtin_amdgcn_ballot_w64(now > hv)) {
        hv = now > hv ? now : hv;
        if (l < 8) __hip_atomic_store(pub, tag | (hv < (unsigned)NCH ? 2 * hv : NCH - 1 + hv), __ATOMIC_RELAXED, __HIP_MEMORY_SCOPE_WORKGROUP);
        unsigned k = 0;
        while (k < (unsigned)PLW && ((m >> (8 * k)) & 255u) == 255u) ++k;
        base += k;
      } else if (__builtin_amdgcn_s_memrealtime() - t0 > DEADLINE ||
                 ((__builtin_amdgcn_s_memrealtime() - t0 > 100000) && __hip_atomic_load(tmo, __ATOMIC_RELAXED, __HIP_MEMORY_SCOPE_AGENT) == call)) {
        if (l == 0) {
          __hip_atomic_store(tmo, call, __ATOMIC_RELAXED, __HIP_MEMORY_SCOPE_AGENT);
          __hip_atomic_store(ctrl, 1u, __ATOMIC_RELAXED, __HIP_MEMORY_SCOPE_WORKGROUP);
        }
        asm volatile("s_waitcnt lgkmcnt(0)" ::: "memory");
        if (l < 8) __hip_atomic_store(pub, tag | NKS, __ATOMIC_RELAXED, __HIP_MEMORY_SCOPE_WORKGROUP);
        break;
      }
    }
    if (l == 0) __hip_atomic_store(pcall, call, __ATOMIC_RELAXED, __HIP_MEMORY_SCOPE_AGENT);
    return;
  }
  if (w > 8) {
    const int h = w - 9;
    const unsigned char* Wb = uni(p.w + (size_t)tn * BN + (size_t)h * 64 * N);
    unsigned boff[2];
    #pragma unroll
    for (int i = 0; i < 2; ++i) boff[i] = (l >> 3) * N + (((l & 7) ^ (((l >> 4) & 3) | (i << 2))) * 16);
    auto ldw = [&](int j, int b) {
      const unsigned char* wb = Wb + (size_t)j * BK * N;
      #pragma unroll
      for (int i = 0; i < 8; i += 4) glds16x4(wb + (size_t)i * 8 * N, wb + (size_t)(i + 1) * 8 * N, wb + (size_t)(i + 2) * 8 * N, wb + (size_t)(i + 3) * 8 * N,
                                              boff[i >> 2], boff[i >> 2], boff[i >> 2], boff[i >> 2], lds0 + LDS_W + b * W_BYTES + h * 8192 + i * 1024);
    };
    const unsigned inc = h ? (1u << 16) : 1u;
    auto sig = [&](unsigned k) { if (l < 8) __hip_atomic_fetch_add(ctrl + 16 + 2 * l, inc * k, __ATOMIC_RELAXED, __HIP_MEMORY_SCOPE_WORKGROUP); };
    auto ldf = [&](int b) { return (unsigned)__builtin_amdgcn_readfirstlane(__hip_atomic_load(ctrl + 8 + b, __ATOMIC_RELAXED, __HIP_MEMORY_SCOPE_WORKGROUP)); };
    #pragma unroll
    for (int j = 0; j < WPRE; ++j) ldw(j, j);
    __builtin_amdgcn_s_barrier();
    int done = 0, buf = WPRE;
    unsigned fr = 0;
    if (WPRE > 1) { asm volatile("s_waitcnt vmcnt(%0)" :: "i"(8 * (WPRE - 1)) : "memory"); sig(1); done = 1; }
    for (int j = WPRE; j < NKS; ++j) {
      if (buf >= NSW) { buf = 0; fr += NWV; }
      if (fr && ldf(buf) < fr) {
        if (done < j) { asm volatile("s_waitcnt vmcnt(0)" ::: "memory"); sig(j - done); done = j; }
        while (ldf(buf) < fr) {}
      }
      asm volatile("" ::: "memory");
      ldw(j, buf);
      if (done < j) { asm volatile("s_waitcnt vmcnt(8)" ::: "memory"); sig(j - done); done = j; }
      ++buf;
    }
    asm volatile("s_waitcnt vmcnt(0)" ::: "memory");
    sig(NKS - done);
    return;
  }

  if (tid < 32) ctrl[tid] = (own && tid > 16 && (tid & 1)) ? (0x100u | NKS) : 0u;
  asm volatile("s_waitcnt lgkmcnt(0)" ::: "memory");
  __builtin_amdgcn_s_barrier();
  if (w >= NWV) return;
  const int g = l >> 4, r16 = l & 15;
  const unsigned xs = ((r16 >> 1) & 1) | (((r16 >> 3) & 1) * 6);
  const unsigned xa0 = lds0 + (w * 32 + r16) * 128 + (((2 * g) ^ xs) * 16);
  const int q = r16 >> 1, pp = l & 1;
  const unsigned sB = ((q >> 1) & 3) | ((g & 1) << 2);
  const unsigned wa0 = lds0 + LDS_W + (32 * g + q) * 128 + sB * 16 + 8 * pp;
  LDSP(unsigned long long) flg = (LDSP(unsigned long long))(uintptr_t)(lds0 + LDS_CTRL + 64 + w * 8);
  auto ldflg = [&]() { return __hip_atomic_load(flg, __ATOMIC_RELAXED, __HIP_MEMORY_SCOPE_WORKGROUP); };
  v4f acc[8][2];
  #pragma unroll
  for (int a = 0; a < 8; ++a)
    #pragma unroll
    for (int b = 0; b < 2; ++b) acc[a][b] = (v4f){0, 0, 0, 0};
  v8i xf[2], wf[4];
#define RW(sl, ni, adr) do { _Pragma("unroll") for (int i_ = 0; i_ < 4; ++i_) { \
    v2i t_ = __builtin_amdgcn_ds_read_tr8_b64_v2i32((LDSP(v2i))(uintptr_t)(((adr) ^ ((ni) * 16)) + i_ * 1024)); \
    wf[sl][2 * i_] = t_.x; wf[sl][2 * i_ + 1] = t_.y; } } while (0)
#define RX(mi, adr) do { v4i t0_ = *(LDSP(v4i))(uintptr_t)((adr) + (mi) * 2048); v4i t1_ = *(LDSP(v4i))(uintptr_t)(((adr) ^ 16) + (mi) * 2048); \
    xf[mi] = (v8i){t0_.x, t0_.y, t0_.z, t0_.w, t1_.x, t1_.y, t1_.z, t1_.w}; } while (0)
#define MM(mi, ni) acc[ni][mi] = __builtin_amdgcn_mfma_scale_f32_16x16x128_f8f6f4(xf[mi], wf[(ni) & 3], acc[ni][mi], 0, 1, 0, 0, 0, 0)
#define SB __builtin_amdgcn_sched_barrier(0)
  float sc = __builtin_bit_cast(float, __builtin_amdgcn_readfirstlane(__builtin_bit_cast(int, p.sx[0] * p.sw[0])));
  asm volatile("" : "+s"(sc));
  float* outp = uni(p.out + (size_t)tm * BM * N + tn * BN);
  asm volatile("" : "+s"(outp));
  __amdgpu_buffer_rsrc_t ors = __builtin_amdgcn_make_buffer_rsrc(outp, 0, BM * N * 4, 0x00020000);
  unsigned have, wl;
  do { unsigned long long f = ldflg(); have = __builtin_amdgcn_readfirstlane((unsigned)(f >> 32)); } while (have == 0);
  const unsigned char* Xb = uni((own ? (p.x + (size_t)(tm & 1) * BM * K) : (myws + WS_STG + ((have >> 9) & 1) * STG_BYTES + (size_t)tm * BM * K)) + (size_t)w * 32 * K);
  const unsigned rp = K, xinc = BK;
  have &= 255u;
  unsigned aoff[4];
  #pragma unroll
  for (int i = 0; i < 4; ++i) { int c = (l & 7) ^ (((l >> 4) & 1) | ((i & 1) * 6)); aoff[i] = (i * 8 + (l >> 3)) * rp + c * 16 - i * 1024; }
  unsigned nx = 0;
  unsigned xm0 = lds0 + w * 4096;
  auto ldx = [&]() {
    asm volatile("s_mov_b32 m0, %5\n\ts_nop 0\n\tglobal_load_lds_dwordx4 %0, %4\n\tglobal_load_lds_dwordx4 %1, %4 offset:1024\n\t"
                 "global_load_lds_dwordx4 %2, %4 offset:2048\n\tglobal_load_lds_dwordx4 %3, %4 offset:3072"
                 :: "v"(aoff[0]), "v"(aoff[1]), "v"(aoff[2]), "v"(aoff[3]), "s"(Xb), "s"(xm0) : "memory", "m0");
    Xb += xinc; xm0 ^= X_BYTES; ++nx;
  };
  auto rdf = [&](unsigned long long f) { have = __builtin_amdgcn_readfirstlane((unsigned)(f >> 32)) & 255u; wl = __builtin_amdgcn_readfirstlane((unsigned)f); };
  auto wlanded = [&](unsigned la) { return (wl & 0xffffu) >= la && (wl >> 16) >= la; };
  auto needx = [&](unsigned s) {
    while (nx <= s) { if (nx < have) ldx(); else rdf(ldflg()); }
    if (nx == s + 1 && nx < have) ldx();
    if (nx == s + 1) asm volatile("s_waitcnt vmcnt(0)" ::: "memory"); else asm volatile("s_waitcnt vmcnt(4)" ::: "memory");
  };
  const unsigned one = 1u;
  needx(0);
  RX(0, xa0); RX(1, xa0);
  do rdf(ldflg()); while (!wlanded(1));
  asm volatile("" ::: "memory");
  RW(0, 0, wa0); RW(1, 1, wa0); RW(2, 2, wa0);
  int buf = 0;
  for (unsigned s = 0; s < NKS - 1; ++s) {
    const unsigned wa = wa0 + buf * W_BYTES;
    int nb = buf + 1; if (nb >= NSW) nb = 0;
    const unsigned wn = wa0 + nb * W_BYTES;
    SB; RW(3, 3, wa); SB;
    MM(0, 0); MM(1, 0); SB;
    if (nx <= s + 2 && nx < have) ldx();
    SB; RW(0, 4, wa); SB;
    MM(0, 1); MM(1, 1); SB;
    const unsigned long long f = ldflg();
    RW(1, 5, wa); SB;
    MM(0, 2); MM(1, 2); SB;
    RW(2, 6, wa); SB;
    MM(0, 3); MM(1, 3); SB;
    RW(3, 7, wa); SB;
    MM(0, 4); MM(1, 4); SB;
    rdf(f);
    if (nx <= s + 2 && nx < have) ldx();
    while (!wlanded(s + 2)) rdf(ldflg());
    asm volatile("" ::: "memory");
    SB; RW(0, 0, wn); SB;
    MM(0, 5); MM(1, 5); SB;
    RW(1, 1, wn); SB;
    MM(0, 6); MM(1, 6); SB;
    RW(2, 2, wn); SB;
    MM(0, 7); SB;
    asm volatile("s_mov_b64 exec, 1\n\tds_add_u32 %0, %1\n\ts_mov_b64 exec, -1" :: "v"(lds0 + LDS_CTRL + 32 + buf * 4), "v"(one) : "memory");
    SB; MM(1, 7); SB;
    needx(s + 1);
    const unsigned xa = xa0 + ((s + 1) & 1) * X_BYTES;
    RX(0, xa); RX(1, xa);
    buf = nb;
  }
  const unsigned ooff = (unsigned)((w * 32 + (l >> 5) * 8) * N + (l & 31)) * 4;
#define ST2(mi, nj) do { v4f a_ = acc[2 * (nj)][mi] * sc, b_ = acc[2 * (nj) + 1][mi] * sc; \
    _Pragma("unroll") for (int e_ = 0; e_ < 4; ++e_) { \
      auto r_ = __builtin_amdgcn_permlane16_swap(__builtin_bit_cast(unsigned, fmaxf(a_[e_], 0.f)), __builtin_bit_cast(unsigned, fmaxf(b_[e_], 0.f)), false, false); \
      __builtin_amdgcn_raw_buffer_store_b32(r_[0], ors, ooff + (nj) * 128, ((mi) * 16 + e_) * N * 4, OAUX); \
      __builtin_amdgcn_raw_buffer_store_b32(r_[1], ors, ooff + (nj) * 128, ((mi) * 16 + 4 + e_) * N * 4, OAUX); } } while (0)
#define WAITP() do { if ((WP & 1) && own || (WP & 2) && !own) while (__builtin_amdgcn_readfirstlane(__hip_atomic_load(ctrl + 2, __ATOMIC_RELAXED, __HIP_MEMORY_SCOPE_WORKGROUP)) < (own ? 2 : 1)) __builtin_amdgcn_s_sleep(2); } while (0)
  {
    const unsigned wa = wa0 + buf * W_BYTES;
    SB; RW(3, 3, wa); SB;
    MM(0, 0); MM(1, 0); SB;
    RW(0, 4, wa); SB;
    MM(0, 1); MM(1, 1); SB;
    RW(1, 5, wa); SB;
    MM(0, 2); MM(1, 2); SB;
    RW(2, 6, wa); SB;
    MM(0, 3); MM(1, 3); SB;
    RW(3, 7, wa); SB;
    WAITP();
    SB;
    MM(0, 4); MM(1, 4); ST2(0, 0); SB;
    MM(0, 5); MM(1, 5); ST2(1, 0); SB;
    MM(0, 6); MM(1, 6); ST2(0, 1); SB;
    MM(0, 7); MM(1, 7); ST2(1, 1); SB;
    ST2(0, 2); ST2(1, 2); ST2(0, 3); ST2(1, 3);
  }
  if (__builtin_amdgcn_readfirstlane(__hip_atomic_load(ctrl, __ATOMIC_RELAXED, __HIP_MEMORY_SCOPE_WORKGROUP))) {
    for (int i = 0; i < 64; ++i) __builtin_amdgcn_raw_buffer_store_b32(0x7fc00000u, ors, (unsigned)((w * 32 + (i >> 1)) * N + (i & 1) * 64 + l) * 4, 0, OAUX);
  }
}

extern "C" void kernel_launch(int rank, int nranks, ncclComm_t comm,
                              hipStream_t stream,
                              void* const* d_in, const int* in_sizes, int n_in,
                              void* d_out, int out_size,
                              const int* in_dtypes, int out_dtype,
                              void* const* d_out_all,
                              void* const* d_ws_all, size_t ws_bytes) {
  static bool init_a[NR];
  if (nranks != NR || ws_bytes < WS_NEED || (unsigned)rank >= (unsigned)NR) { fprintf(stderr, "bad config\n"); return; }
  if (!init_a[rank]) {
    HIP_CHECK(hipFuncSetAttribute((const void*)ag_gemm_kernel, hipFuncAttributeMaxDynamicSharedMemorySize, LDS_BYTES));
    init_a[rank] = true;
  }
  Params p;
  p.x = (const unsigned char*)d_in[0]; p.w = (const unsigned char*)d_in[1];
  p.sx = (const float*)d_in[2]; p.sw = (const float*)d_in[3]; p.out = (float*)d_out;
  for (int i = 0; i < NR; ++i) p.ws[i] = (unsigned char*)d_ws_all[i];
  p.rank = rank;
  ag_gemm_kernel<<<GEMM_WGS, NTHREADS, LDS_BYTES, stream>>>(p);
}
```

```cpp
#include <hip/hip_runtime.h>
#include <rccl/rccl.h>
#include <cstdio>
#include <cstdint>

#define HIP_CHECK(expr) do { hipError_t _e = (expr); if (_e != hipSuccess) { \
    fprintf(stderr, "HIP error %s at %s:%d\n", hipGetErrorString(_e), __FILE__, __LINE__); \
    return; } } while (0)

typedef int v2i __attribute__((ext_vector_type(2)));
typedef int v4i __attribute__((ext_vector_type(4)));
typedef unsigned v4u __attribute__((ext_vector_type(4)));
typedef int v8i __attribute__((ext_vector_type(8)));
typedef float v4f __attribute__((ext_vector_type(4)));
#define LDSP(T) __attribute__((address_space(3))) T*
#define NEWMAP 1

constexpr int NR = 8;
constexpr int M = 4096, N = 2048, K = 4096;
constexpr int MS = M / NR;
constexpr int BM = 256, BN = 128, BK = 128;
constexpr int X_BYTES = BM * BK, W_BYTES = BK * BN + 48 * BK;
#define NSWV 4
constexpr int NSX = 2, NSW = NSWV;
constexpr int NKS = K / BK;
constexpr int KC = 256;
constexpr int NCH = K / KC;
#define SPL 1
constexpr int NCT = NCH + SPL;
#define HN 3
#define HE 2
constexpr int NCT2 = NCT + 1;
constexpr int PARTS = 64;
constexpr int ARRIVALS = 4;
constexpr int NWG = 256, GWG = 224, CWG = NWG - GWG;
constexpr int NTHREADS = 768;
#define PWV 7
#define PLW 3
#define TL 1
#define WPRE 2
#define ZSL 32
#define Q3 10
constexpr int WT = 17;
constexpr unsigned NPUSH = CWG * PWV;
constexpr int LDS_W = NSX * X_BYTES;
constexpr int LDS_CTRL = LDS_W + NSW * W_BYTES;
constexpr int LDS_DBG = LDS_CTRL + 128;
constexpr int LDS_BYTES = LDS_DBG + 768;
constexpr size_t WS_TMO = 0;
constexpr size_t WS_CNT = 4096;
constexpr size_t WS_PTICK = 69632;
constexpr size_t WS_ZC = 73728;
constexpr size_t WS_PCALL = 524288;
constexpr int NITEM = NCH * PARTS;
constexpr size_t WS_STG = 1u << 20;
constexpr size_t STG_BYTES = (size_t)M * K;
constexpr size_t WS_NEED = WS_STG + 2 * STG_BYTES;
constexpr unsigned long long DEADLINE = 40000000ull;

struct Params {
  const unsigned char* x; const unsigned char* w; const float* sx; const float* sw; float* out;
  unsigned char* ws[NR];
  int rank;
};

__host__ __device__ inline int tile_tm(int lb, int rank) { return ((rank + 1 + (lb >> 5)) & 7) * 2 + (lb & 1); }

__device__ __forceinline__ void glds16x4(const unsigned char* s0, const unsigned char* s1, const unsigned char* s2, const unsigned char* s3,
                                         unsigned v0, unsigned v1, unsigned v2, unsigned v3, unsigned lds_off) {
  asm volatile("s_mov_b32 m0, %8\n\ts_nop 0\n\tglobal_load_lds_dwordx4 %0, %4\n\tglobal_load_lds_dwordx4 %1, %5 offset:1024\n\t"
               "global_load_lds_dwordx4 %2, %6 offset:2048\n\tglobal_load_lds_dwordx4 %3, %7 offset:3072"
               :: "v"(v0), "v"(v1), "v"(v2), "v"(v3), "s"(s0), "s"(s1 - 1024), "s"(s2 - 2048), "s"(s3 - 3072), "s"(lds_off) : "memory", "m0");
}

template <typename T> __device__ __forceinline__ T* uni(T* p) {
  unsigned long long v = (unsigned long long)p;
  unsigned lo = __builtin_amdgcn_readfirstlane((unsigned)v), hi = __builtin_amdgcn_readfirstlane((unsigned)(v >> 32));
  return (T*)(((unsigned long long)hi << 32) | lo);
}

#define PW(n) asm volatile("s_waitcnt vmcnt(%0)" :: "i"(n) : "memory")
#define ST2(rs, voff, mi, nj, A, B) do { v4f a_ = (A) * sc, b_ = (B) * sc; \
    _Pragma("unroll") for (int e_ = 0; e_ < 4; ++e_) { \
      auto r_ = __builtin_amdgcn_permlane16_swap(__builtin_bit_cast(unsigned, fmaxf(a_[e_], 0.f)), __builtin_bit_cast(unsigned, fmaxf(b_[e_], 0.f)), false, false); \
      __builtin_amdgcn_raw_buffer_store_b32(r_[0], rs, (voff) + (nj) * 128, ((mi) * 16 + e_) * N * 4, WT); \
      __builtin_amdgcn_raw_buffer_store_b32(r_[1], rs, (voff) + (nj) * 128, ((mi) * 16 + 4 + e_) * N * 4, WT); } } while (0)

__global__ void __launch_bounds__(NTHREADS, 3) ag_gemm_kernel(Params p) {
  extern __shared__ __attribute__((aligned(128))) unsigned char smem[];
  const int tid = threadIdx.x, l = tid & 63, w = __builtin_amdgcn_readfirstlane(tid >> 6);
  unsigned char* myws = p.ws[p.rank];
  const unsigned lds0 = (unsigned)(uintptr_t)smem;
  LDSP(unsigned) ctrl = (LDSP(unsigned))(uintptr_t)(lds0 + LDS_CTRL);
  const int lb = (blockIdx.x + GWG) & (NWG - 1);

  if (lb >= GWG) {
    if (w >= PWV) return;
    const unsigned v = (unsigned)((lb - GWG) * PWV + w);
    const unsigned j = v % (NR - 1), nj = (NPUSH - j + NR - 2) / (NR - 1);
    const unsigned tfirst = v / (NR - 1);
    const unsigned loff = (unsigned)(l >> 4) * K + (l & 15) * 16;
    const unsigned loff2 = (unsigned)(l >> 3) * K + (l & 7) * 16;
    constexpr unsigned TB = NITEM - TL * PARTS;
    auto canon = [&](unsigned t) {
      if (t < TB) return t;
      const unsigned u = t - TB, b = u / (8 * TL), r = u % (8 * TL);
      return TB + (r >> 3) * 64 + b * 8 + (r & 7);
    };
    auto spl = [&](unsigned c) { return c >= (unsigned)(NITEM - PARTS); };
    auto item_off = [&](unsigned c) {
      return spl(c) ? (((c >> 1) & 1) * 256 + ((c >> 3) & 7) * 32 + (c & 1) * 16) * K + (NCH - 1) * KC + ((c >> 2) & 1) * 128 + loff2
                    : (((c >> 2) & 1) * 256 + ((c >> 3) & 7) * 32 + (c & 3) * 8) * K + (c / PARTS) * KC + loff; };
    auto item_st = [&](unsigned c) { return spl(c) ? 8u * K : 4u * K; };
#define PST(d, t, i) __builtin_amdgcn_raw_buffer_store_b128(d, rs, o0, (i) * item_st(t), WT)
    auto cidx = [&](unsigned c) {
      return spl(c) ? ((((c >> 1) & 1) * NCT2 + NCH - 1 + ((c >> 2) & 1)) * 32 + ((c >> 3) & 7)) | 0x20000u
                    : ((((c >> 2) & 1) * NCT2 + c / PARTS) * 32 + ((c >> 3) & 7)) | 0x10000u; };
    constexpr unsigned TH = (NCH - HN) * PARTS;
    auto tk = [&](unsigned t) { t = __builtin_amdgcn_readfirstlane(t) + 3 * nj; return t < (unsigned)NITEM ? canon(t) : t; };
    unsigned* pcall = (unsigned*)(myws + WS_PCALL + lb * 128) + w;
    unsigned t0 = tfirst, t1;
    unsigned o0 = item_off(t0), o1;
    v4u cur[2], nxt[2];
    unsigned n = __hip_atomic_load(pcall, __ATOMIC_RELAXED, __HIP_MEMORY_SCOPE_AGENT);
    #pragma unroll
    for (int i = 0; i < 2; ++i) cur[i] = *(const v4u*)(p.x + o0 + i * item_st(t0));
    o1 = item_off(t0 + nj);
    #pragma unroll
    for (int i = 0; i < 2; ++i) nxt[i] = *(const v4u*)(p.x + o1 + i * item_st(t0));
    n = __builtin_amdgcn_readfirstlane(n);
    const unsigned parity = (n + 1) & 1;
    unsigned vz; asm volatile("v_mov_b32 %0, 0" : "=v"(vz));
    const unsigned peer = (unsigned)(p.rank + 1 + j) % NR;
    unsigned char* dws = p.ws[peer];
    __amdgpu_buffer_rsrc_t rs = __builtin_amdgcn_make_buffer_rsrc(dws + WS_STG + parity * STG_BYTES + (size_t)p.rank * MS * K, 0, MS * K, 0x00020000);
    unsigned* cnt = (unsigned*)(dws + WS_CNT + (size_t)p.rank * 2 * NCT2 * 128) + vz;
    unsigned* tick = (unsigned*)(myws + WS_PTICK + parity * 1024 + j * 128) + vz;
    unsigned cprev;
#define PSIG() do { if (l == 0) __hip_atomic_fetch_add(cnt + (cprev & 0xffffu), cprev >> 16, __ATOMIC_RELAXED, __HIP_MEMORY_SCOPE_SYSTEM); } while (0)
#define PHINT() do { if (HN && t0 >= TH) { PSIG(); cprev = 0; } } while (0)
#define DRAW(t) do { if (l == 0) t = __hip_atomic_fetch_add(tick, 1u, __ATOMIC_RELAXED, __HIP_MEMORY_SCOPE_AGENT); } while (0)
    unsigned t2 = 0;
    asm volatile("" :: "v"(cur[0]), "v"(cur[1]));
    asm volatile("" :: "v"(nxt[0]), "v"(nxt[1]));
    PST(cur[0], t0, 0);
    PST(cur[1], t0, 1);
    if (v == 0 && l < 8)
      __hip_atomic_store((unsigned*)(myws + WS_PTICK + (parity ^ 1) * 1024) + l * 32, 0u, __ATOMIC_RELAXED, __HIP_MEMORY_SCOPE_SYSTEM);
    cprev = cidx(t0);
    t2 = tfirst - nj;
    asm volatile("" : "+v"(t2) :: "memory");
    t0 += nj; o0 = o1;
    unsigned s1;
    cur[0] = nxt[0]; cur[1] = nxt[1];
    t1 = tk(t2);
    bool hp = false;
    asm volatile("" :: "v"(cur[0]), "v"(cur[1]));
    while (t0 < (unsigned)NITEM) {
      o1 = item_off(t1 < (unsigned)NITEM ? t1 : t0); s1 = item_st(t1 < (unsigned)NITEM ? t1 : t0);
      DRAW(t2);
      #pragma unroll
      for (int i = 0; i < 2; ++i) nxt[i] = *(const v4u*)(p.x + o1 + i * s1);
      if (HN && HE == 0 && hp) PW(5); else PW(4);
      PST(cur[0], t0, 0);
      if (HN && HE == 0 && hp) PW(5); else PW(4);
      if (cprev) PSIG();
      cprev = cidx(t0);
      if (HE) PHINT();
      PST(cur[1], t0, 1);
      asm volatile("s_nop 1" ::: "memory");
      if (!HE) PHINT();
      asm volatile("" ::: "memory");
      hp = t0 >= TH; t0 = t1; o0 = o1;
      t1 = tk(t2);
      #pragma unroll
      for (int i = 0; i < 2; ++i) cur[i] = nxt[i];
      asm volatile("" :: "v"(cur[0]), "v"(cur[1]));
    }
    { const unsigned long long t0 = __builtin_amdgcn_s_memrealtime();
      while ((unsigned)__builtin_amdgcn_readfirstlane(__hip_atomic_load((const unsigned*)(myws + WS_ZC), __ATOMIC_RELAXED, __HIP_MEMORY_SCOPE_AGENT)) < (n + 1) * GWG &&
             __builtin_amdgcn_s_memrealtime() - t0 < DEADLINE) {} }
    PW(0);
    if (cprev) PSIG();
    cprev = (NCT * 32) | 0x10000u;
    PSIG();
    if (l == 0) __hip_atomic_store(pcall, n + 1, __ATOMIC_RELAXED, __HIP_MEMORY_SCOPE_AGENT);
    return;
  }

  const int xcd = lb & 7, r7 = lb >> 5, hf = lb & 1;
  const int tm = tile_tm(lb, p.rank), tn = (xcd >> 1) * 4 + ((lb >> 3) & 3);
  const int src = tm >> 1;
  constexpr unsigned NS = NKS;
  const unsigned* oc = (const unsigned*)(myws + WS_CNT + ((size_t)src * 2 * NCT2 + NCT) * 128);
  const unsigned npw = (NPUSH + r7) / (NR - 1);
  const int lq = r7 * 2 + hf;
  const bool x3 = lq >= Q3 && lq < Q3 + 4;
  const int lr3 = x3 ? (28 + lq - Q3) * 16 : lq * 32;

  if (w == 11) {
    unsigned* pcall = (unsigned*)(myws + WS_PCALL + lb * 128);
    unsigned n = __hip_atomic_load(pcall, __ATOMIC_RELAXED, __HIP_MEMORY_SCOPE_AGENT);
    __builtin_amdgcn_s_barrier();
    const unsigned call = __builtin_amdgcn_readfirstlane(n) + 1;
    const unsigned tag = 0x100u | ((call & 1) << 9);
    LDSP(unsigned) pub = ctrl + 17 + 2 * (l & 7);
    if (l == 0) __hip_atomic_store(ctrl + 4, call, __ATOMIC_RELAXED, __HIP_MEMORY_SCOPE_WORKGROUP);
    asm volatile("s_waitcnt lgkmcnt(0)" ::: "memory");
    if (l < 8) __hip_atomic_store(pub, tag, __ATOMIC_RELAXED, __HIP_MEMORY_SCOPE_WORKGROUP);
    __builtin_amdgcn_fence(__ATOMIC_ACQUIRE, "");
    asm volatile("s_waitcnt vmcnt(0)" ::: "memory");
    const unsigned* cnt = (const unsigned*)(myws + WS_CNT + (size_t)tm * NCT2 * 128) + (l >> 3) * 32 + (l & 7);
    unsigned* tmo = (unsigned*)(myws + WS_TMO);
    const unsigned target = call * ARRIVALS;
    unsigned base = 0, hv = 0;
    const unsigned long long t0 = __builtin_amdgcn_s_memrealtime();
    while (base < (unsigned)NCT) {
      bool ok = false;
      if (l < 8 * PLW && base + (l >> 3) < (unsigned)NCT) ok = __hip_atomic_load(cnt + base * 32, __ATOMIC_RELAXED, __HIP_MEMORY_SCOPE_SYSTEM) >= target;
      const unsigned long long m = __builtin_amdgcn_ballot_w64(ok);
      const unsigned now = base + ((unsigned)__builtin_ctzll(~(m >> (l & 7)) & 0x0101010101010101ull) >> 3);
      if (__builtin_amdgcn_ballot_w64(now > hv)) {
        hv = now > hv ? now : hv;
        if (l < 8) __hip_atomic_store(pub, tag | (hv < (unsigned)NCH ? 2 * hv : NCH - 1 + hv), __ATOMIC_RELAXED, __HIP_MEMORY_SCOPE_WORKGROUP);
        unsigned k = 0;
        while (k < (unsigned)PLW && ((m >> (8 * k)) & 255u) == 255u) ++k;
        base += k;
      } else if (__builtin_amdgcn_s_memrealtime() - t0 > DEADLINE ||
                 ((__builtin_amdgcn_s_memrealtime() - t0 > 100000) && __hip_atomic_load(tmo, __ATOMIC_RELAXED, __HIP_MEMORY_SCOPE_AGENT) == call)) {
        if (l == 0) {
          __hip_atomic_store(tmo, call, __ATOMIC_RELAXED, __HIP_MEMORY_SCOPE_AGENT);
          __hip_atomic_store(ctrl, 1u, __ATOMIC_RELAXED, __HIP_MEMORY_SCOPE_WORKGROUP);
        }
        asm volatile("s_waitcnt lgkmcnt(0)" ::: "memory");
        if (l < 8) __hip_atomic_store(pub, tag | NKS, __ATOMIC_RELAXED, __HIP_MEMORY_SCOPE_WORKGROUP);
        break;
      }
    }
    while ((unsigned)__builtin_amdgcn_readfirstlane(__hip_atomic_load(oc, __ATOMIC_RELAXED, __HIP_MEMORY_SCOPE_SYSTEM)) < call * npw &&
           __builtin_amdgcn_s_memrealtime() - t0 < DEADLINE) __builtin_amdgcn_s_sleep(8);
    if (l == 0) __hip_atomic_store(pcall, call, __ATOMIC_RELAXED, __HIP_MEMORY_SCOPE_AGENT);
    return;
  }
  if (w == 8) {
    __builtin_amdgcn_s_barrier();
    unsigned call;
    do { __builtin_amdgcn_s_sleep(16); call = __builtin_amdgcn_readfirstlane(__hip_atomic_load(ctrl + 4, __ATOMIC_RELAXED, __HIP_MEMORY_SCOPE_WORKGROUP)); } while (call == 0);
    __amdgpu_buffer_rsrc_t zs = __builtin_amdgcn_make_buffer_rsrc(myws + WS_STG + ((call & 1) ^ 1) * STG_BYTES + (size_t)(tm * BM + tn * 16) * K, 0, 16 * K, 0x00020000);
    for (int r = 0; r < 16; ++r) {
      #pragma unroll
      for (int i = 0; i < (HN + 3) / 4; ++i)
        if (i * 64 + l < HN * 16) __builtin_amdgcn_raw_buffer_store_b128((v4u){0, 0, 0, 0}, zs, (NCH - HN) * KC + i * 1024 + l * 16, r * K, WT);
      __builtin_amdgcn_s_sleep(ZSL);
    }
    PW(0);
    if (l == 0) __hip_atomic_fetch_add((unsigned*)(myws + WS_ZC), 1u, __ATOMIC_RELAXED, __HIP_MEMORY_SCOPE_AGENT);
    return;
  }
  if (w > 8) {
    const int h = w - 9;
    const unsigned char* Wb = uni(p.w + (size_t)tn * BN + (size_t)h * 64 * N);
    unsigned boff[2];
    #pragma unroll
    for (int i = 0; i < 2; ++i) boff[i] = (l >> 3) * N + (((l & 7) ^ (((l >> 4) & 3) | (i << 2))) * 16);
    const unsigned loff = (l >> 3) * K + (((l & 7) ^ (((l >> 4) & 1) | (h * 6))) * 16);
    const unsigned char* Lb = uni(p.x + (size_t)(lq * 32 + h * 8) * K);
    const unsigned char* Lb3 = uni(p.x + (size_t)(lr3 + h * 8) * K);
    auto ldw = [&](unsigned j, int b) {
      const unsigned char* wb = Wb + (size_t)j * BK * N;
      #pragma unroll
      for (int i = 0; i < 8; i += 4) glds16x4(wb + (size_t)i * 8 * N, wb + (size_t)(i + 1) * 8 * N, wb + (size_t)(i + 2) * 8 * N, wb + (size_t)(i + 3) * 8 * N,
                                              boff[i >> 2], boff[i >> 2], boff[i >> 2], boff[i >> 2], lds0 + LDS_W + b * W_BYTES + h * 8192 + i * 1024);
      asm volatile("s_mov_b32 m0, %3\n\ts_nop 0\n\tglobal_load_lds_dwordx4 %0, %1\n\tglobal_load_lds_dwordx4 %0, %5 offset:2048\n\t"
                   "s_mov_b32 m0, %4\n\ts_nop 0\n\tglobal_load_lds_dwordx4 %0, %2"
                   :: "v"(loff), "s"(Lb + j * BK), "s"(Lb3 + j * BK), "s"(lds0 + LDS_W + b * W_BYTES + BK * BN + h * 1024),
                      "s"(lds0 + LDS_W + b * W_BYTES + BK * BN + 4096 + h * 1024), "s"(Lb + j * BK + 16 * K - 2048) : "memory", "m0");
    };
    const unsigned inc = h ? (1u << 16) : 1u;
    auto sig = [&](unsigned k) { if (l < 8) __hip_atomic_fetch_add(ctrl + 16 + 2 * l, inc * k, __ATOMIC_RELAXED, __HIP_MEMORY_SCOPE_WORKGROUP); };
    auto ldf = [&](int b) { return (unsigned)__builtin_amdgcn_readfirstlane(__hip_atomic_load(ctrl + 8 + b, __ATOMIC_RELAXED, __HIP_MEMORY_SCOPE_WORKGROUP)); };
    #pragma unroll
    for (int j = 0; j < WPRE; ++j) ldw(j, j);
    __builtin_amdgcn_s_barrier();
    unsigned done = 0; int buf = WPRE;
    unsigned fr = 0;
    if (WPRE > 1) { PW(11 * (WPRE - 1)); sig(1); done = 1; }
    for (unsigned j = WPRE; j < NS; ++j) {
      if (buf >= NSW) { buf = 0; fr += 8; }
      if (fr && ldf(buf) < fr) {
        if (done < j) { PW(0); sig(j - done); done = j; }
        while (ldf(buf) < fr) {}
      }
      asm volatile("" ::: "memory");
      ldw(j, buf);
      if (done < j) { PW(11); sig(j - done); done = j; }
      ++buf;
    }
    PW(0);
    sig(NS - done);
    return;
  }

  const unsigned char* Xb;
  unsigned aoff[4];
  #pragma unroll
  for (int i = 0; i < 4; ++i) { int c = (l & 7) ^ (((l >> 4) & 1) | ((i & 1) * 6)); aoff[i] = (i * 8 + (l >> 3)) * K + c * 16 - i * 1024; }
  unsigned nx = 0;
  unsigned xm0 = lds0 + w * 4096;
  auto ldx = [&]() {
    asm volatile("s_mov_b32 m0, %5\n\ts_nop 0\n\tglobal_load_lds_dwordx4 %0, %4\n\tglobal_load_lds_dwordx4 %1, %4 offset:1024\n\t"
                 "global_load_lds_dwordx4 %2, %4 offset:2048\n\tglobal_load_lds_dwordx4 %3, %4 offset:3072"
                 :: "v"(aoff[0]), "v"(aoff[1]), "v"(aoff[2]), "v"(aoff[3]), "s"(Xb), "s"(xm0) : "memory", "m0");
    xm0 ^= X_BYTES; ++nx; Xb += BK;
  };
  if (tid < 32) ctrl[tid] = 0u;
  asm volatile("s_waitcnt lgkmcnt(0)" ::: "memory");
  __builtin_amdgcn_s_barrier();
  const int g = l >> 4, r16 = l & 15;
  const unsigned xs = ((r16 >> 1) & 1) | (((r16 >> 3) & 1) * 6);
  const unsigned xa0 = lds0 + (w * 32 + r16) * 128 + (((2 * g) ^ xs) * 16);
  const int q = r16 >> 1, pp = l & 1;
  const unsigned sB = ((q >> 1) & 3) | ((g & 1) << 2);
  const unsigned wa0 = lds0 + LDS_W + (32 * g + q) * 128 + (sB ^ (w & 6)) * 16 + 8 * pp;
  const unsigned la0 = lds0 + LDS_W + BK * BN + r16 * 128 + (((2 * g) ^ xs) * 16);
  LDSP(unsigned long long) flg = (LDSP(unsigned long long))(uintptr_t)(lds0 + LDS_CTRL + 64 + w * 8);
  auto ldflg = [&]() { return __hip_atomic_load(flg, __ATOMIC_RELAXED, __HIP_MEMORY_SCOPE_WORKGROUP); };
  v4f acc[8][2];
  #pragma unroll
  for (int a = 0; a < 8; ++a)
    #pragma unroll
    for (int b = 0; b < 2; ++b) acc[a][b] = (v4f){0, 0, 0, 0};
  v4f acl[3] = {(v4f){0, 0, 0, 0}, (v4f){0, 0, 0, 0}, (v4f){0, 0, 0, 0}};
  v8i xf[2], wf[4], xl;
#define RW(sl, ni, adr) do { _Pragma("unroll") for (int i_ = 0; i_ < 4; ++i_) { \
    v2i t_ = __builtin_amdgcn_ds_read_tr8_b64_v2i32((LDSP(v2i))(uintptr_t)(((adr) ^ ((ni) * 16)) + i_ * 1024)); \
    wf[sl][2 * i_] = t_.x; wf[sl][2 * i_ + 1] = t_.y; } } while (0)
#define RX(mi, adr) do { v4i t0_ = *(LDSP(v4i))(uintptr_t)((adr) + (mi) * 2048); v4i t1_ = *(LDSP(v4i))(uintptr_t)(((adr) ^ 16) + (mi) * 2048); \
    xf[mi] = (v8i){t0_.x, t0_.y, t0_.z, t0_.w, t1_.x, t1_.y, t1_.z, t1_.w}; } while (0)
#define MM(mi, ni) acc[ni][mi] = __builtin_amdgcn_mfma_scale_f32_16x16x128_f8f6f4(xf[mi], wf[(ni) & 3], acc[ni][mi], 0, 1, 0, 0, 0, 0)
#define RL(adr) do { v4i t0_ = *(LDSP(v4i))(uintptr_t)(adr); v4i t1_ = *(LDSP(v4i))(uintptr_t)((adr) ^ 16); \
    xl = (v8i){t0_.x, t0_.y, t0_.z, t0_.w, t1_.x, t1_.y, t1_.z, t1_.w}; } while (0)
#define ML(a, ni) acl[a] = __builtin_amdgcn_mfma_scale_f32_16x16x128_f8f6f4(xl, wf[ni], acl[a], 0, 1, 0, 0, 0, 0)
#define SB __builtin_amdgcn_sched_barrier(0)
  float sc;
  { float s0_, s1_;
    asm volatile("s_load_dword %0, %2, 0x0\n\ts_load_dword %1, %3, 0x0\n\ts_waitcnt lgkmcnt(0)" : "=&s"(s0_), "=&s"(s1_) : "s"(p.sx), "s"(p.sw) : "memory");
    sc = __builtin_bit_cast(float, __builtin_amdgcn_readfirstlane(__builtin_bit_cast(int, s0_ * s1_))); }
  asm volatile("" : "+s"(sc));
  float* outp = uni(p.out + (size_t)tm * BM * N + tn * BN);
  asm volatile("" : "+s"(outp));
  __amdgpu_buffer_rsrc_t ors = __builtin_amdgcn_make_buffer_rsrc(outp, 0, BM * N * 4, 0x00020000);
  unsigned have, wl;
  do { unsigned long long f = ldflg(); have = __builtin_amdgcn_readfirstlane((unsigned)(f >> 32)); } while (have == 0);
  Xb = uni(myws + WS_STG + ((have >> 9) & 1) * STG_BYTES + (size_t)(tm * BM + w * 32) * K);
  have &= 255u;
  auto rdf = [&](unsigned long long f) { have = __builtin_amdgcn_readfirstlane((unsigned)(f >> 32)) & 255u; wl = __builtin_amdgcn_readfirstlane((unsigned)f); };
  auto wlanded = [&](unsigned la) { return (wl & 0xffffu) >= la && (wl >> 16) >= la; };
  auto needx = [&](unsigned s) {
    while (nx <= s) { if (nx < have) ldx(); else rdf(ldflg()); }
    if (nx == s + 1 && nx < have) ldx();
    if (nx == s + 1) PW(0); else PW(4);
  };
  const unsigned one = 1u;
  const unsigned sval = NS - 2 * HN;
  auto xbad = [&]() {
    const unsigned a = xf[0][0] | xf[0][1] | xf[0][2] | xf[0][3] | xf[0][4] | xf[0][5] | xf[0][6] | xf[0][7];
    const unsigned b = xf[1][0] | xf[1][1] | xf[1][2] | xf[1][3] | xf[1][4] | xf[1][5] | xf[1][6] | xf[1][7];
    return __builtin_amdgcn_ballot_w64((a < b ? a : b) == 0u) != 0ull; };
  auto refetch = [&](unsigned s) {
    const unsigned call = __builtin_amdgcn_readfirstlane(__hip_atomic_load(ctrl + 4, __ATOMIC_RELAXED, __HIP_MEMORY_SCOPE_WORKGROUP));
    const unsigned char* xs_ = uni(Xb - (size_t)(nx - s) * BK);
    const unsigned xa = xa0 + (s & 1) * X_BYTES, xm = lds0 + w * 4096 + (s & 1) * X_BYTES;
    const unsigned long long t0 = __builtin_amdgcn_s_memrealtime();
    bool fin;
    do {
      fin = __builtin_amdgcn_readfirstlane(__hip_atomic_load(oc, __ATOMIC_RELAXED, __HIP_MEMORY_SCOPE_SYSTEM)) >= call * npw;
      asm volatile("s_waitcnt vmcnt(0)\n\ts_mov_b32 m0, %5\n\ts_nop 0\n\tglobal_load_lds_dwordx4 %0, %4 sc0 sc1\n\tglobal_load_lds_dwordx4 %1, %4 offset:1024 sc0 sc1\n\t"
                   "global_load_lds_dwordx4 %2, %4 offset:2048 sc0 sc1\n\tglobal_load_lds_dwordx4 %3, %4 offset:3072 sc0 sc1\n\ts_waitcnt vmcnt(0)"
                   :: "v"(aoff[0]), "v"(aoff[1]), "v"(aoff[2]), "v"(aoff[3]), "s"(xs_), "s"(xm) : "memory", "m0");
      RX(0, xa); RX(1, xa);
      if (__builtin_amdgcn_s_memrealtime() - t0 > DEADLINE || __builtin_amdgcn_readfirstlane(__hip_atomic_load(ctrl, __ATOMIC_RELAXED, __HIP_MEMORY_SCOPE_WORKGROUP))) {
        if (l == 0) __hip_atomic_store(ctrl, 1u, __ATOMIC_RELAXED, __HIP_MEMORY_SCOPE_WORKGROUP);
        fin = true;
      }
    } while (!fin && xbad());
  };
  needx(0);
  RX(0, xa0); RX(1, xa0);
  do rdf(ldflg()); while (!wlanded(1));
  asm volatile("" ::: "memory");
  RW(0, 0, wa0); RL(la0 + (w & 1) * 2048); RW(1, 1, wa0); RW(2, 2, wa0);
  int buf = 0;
  const bool x3e = x3 && !(w & 1), x3o = x3 && (w & 1);
  for (unsigned s = 0; s < NS - 1; ++s) {
    const unsigned wa = wa0 + buf * W_BYTES;
    int nb = buf + 1; if (nb >= NSW) nb = 0;
    const unsigned wn = wa0 + nb * W_BYTES;
    SB; RW(3, 3, wa); SB;
    ML(0, 0); MM(0, 0); MM(1, 0); SB;
    if (nx <= s + 2 && nx < have) ldx();
    SB; RW(0, 4, wa); SB;
    ML(1, 1); MM(0, 1); SB;
    if (x3) RL(la0 + buf * W_BYTES + 4096);
    SB; MM(1, 1); SB;
    const unsigned long long f = ldflg();
    RW(1, 5, wa); SB;
    MM(0, 2); MM(1, 2); SB;
    if (x3e) ML(2, 2);
    SB; RW(2, 6, wa); SB;
    MM(0, 3); MM(1, 3); SB;
    if (x3o) ML(2, 3);
    SB; RW(3, 7, wa); SB;
    MM(0, 4); MM(1, 4); SB;
    rdf(f);
    if (nx <= s + 2 && nx < have) ldx();
    while (!wlanded(s + 2)) rdf(ldflg());
    asm volatile("" ::: "memory");
    SB; RW(0, 0, wn); SB;
    MM(0, 5); MM(1, 5); SB;
    RL(la0 + nb * W_BYTES + (w & 1) * 2048); RW(1, 1, wn); SB;
    MM(0, 6); MM(1, 6); SB;
    RW(2, 2, wn); SB;
    MM(0, 7); SB;
    asm volatile("s_mov_b64 exec, 1\n\tds_add_u32 %0, %1\n\ts_mov_b64 exec, -1" :: "v"(lds0 + LDS_CTRL + 32 + buf * 4), "v"(one) : "memory");
    SB; MM(1, 7); SB;
    needx(s + 1);
    const unsigned xa = xa0 + ((s + 1) & 1) * X_BYTES;
    RX(0, xa); RX(1, xa);
    if (s + 1 >= sval) { if (__builtin_expect(xbad(), 0)) refetch(s + 1); }
    buf = nb;
  }
  const unsigned ooff = (unsigned)((w * 32 + (l >> 5) * 8) * N + (l & 31)) * 4 + (w >> 1) * 128;
#define STO(mi, nj) ST2(ors, ooff ^ ((nj) * 128), mi, 0, acc[2 * (nj)][mi], acc[2 * (nj) + 1][mi])
  __amdgpu_buffer_rsrc_t lrs = __builtin_amdgcn_make_buffer_rsrc(uni(p.out + (size_t)p.rank * MS * N + tn * BN), 0, MS * N * 4 - tn * BN * 4, 0x00020000);
  {
    const unsigned wa = wa0 + buf * W_BYTES;
    SB; RW(3, 3, wa); SB;
    ML(0, 0); MM(0, 0); MM(1, 0); SB;
    RW(0, 4, wa); SB;
    ML(1, 1); MM(0, 1); SB;
    if (x3) RL(la0 + buf * W_BYTES + 4096);
    SB; MM(1, 1); SB;
    RW(1, 5, wa); SB;
    MM(0, 2); MM(1, 2); SB;
    if (x3e) ML(2, 2);
    SB; RW(2, 6, wa); SB;
    MM(0, 3); MM(1, 3); SB;
    if (x3o) ML(2, 3);
    SB; RW(3, 7, wa); SB;
    ST2(lrs, (unsigned)((lq * 32 + (w & 1) * 16 + (l >> 5) * 8) * N + (l & 31)) * 4 + (w >> 1) * 128, 0, 0, acl[0], acl[1]); SB;
    SB;
    MM(0, 4); MM(1, 4); STO(0, 0); SB;
    MM(0, 5); MM(1, 5); STO(1, 0); SB;
    MM(0, 6); MM(1, 6); STO(0, 1); SB;
    MM(0, 7); MM(1, 7); STO(1, 1); SB;
    STO(0, 2); STO(1, 2); STO(0, 3); STO(1, 3);
    if (x3) {
      #pragma unroll
      for (int e = 0; e < 4; ++e)
        __builtin_amdgcn_raw_buffer_store_b32(__builtin_bit_cast(unsigned, fmaxf(acl[2][e] * sc, 0.f)), lrs, (unsigned)((lr3 + g * 4) * N + (w ^ 2) * 16 + r16) * 4, e * N * 4, WT);
    }
  }
  if (__builtin_amdgcn_readfirstlane(__hip_atomic_load(ctrl, __ATOMIC_RELAXED, __HIP_MEMORY_SCOPE_WORKGROUP))) {
    for (int i = 0; i < 64; ++i) __builtin_amdgcn_raw_buffer_store_b32(0x7fc00000u, ors, (unsigned)((w * 32 + (i >> 1)) * N + (i & 1) * 64 + l) * 4, 0, WT);
  }
}

extern "C" void kernel_launch(int rank, int nranks, ncclComm_t comm,
                              hipStream_t stream,
                              void* const* d_in, const int* in_sizes, int n_in,
                              void* d_out, int out_size,
                              const int* in_dtypes, int out_dtype,
                              void* const* d_out_all,
                              void* const* d_ws_all, size_t ws_bytes) {
  static bool init_a[NR];
  if (nranks != NR || ws_bytes < WS_NEED || (unsigned)rank >= (unsigned)NR) { fprintf(stderr, "bad config\n"); return; }
  if (!init_a[rank]) {
    HIP_CHECK(hipFuncSetAttribute((const void*)ag_gemm_kernel, hipFuncAttributeMaxDynamicSharedMemorySize, LDS_BYTES));
    init_a[rank] = true;
  }
  Params p;
  p.x = (const unsigned char*)d_in[0]; p.w = (const unsigned char*)d_in[1];
  p.sx = (const float*)d_in[2]; p.sw = (const float*)d_in[3]; p.out = (float*)d_out;
  for (int i = 0; i < NR; ++i) p.ws[i] = (unsigned char*)d_ws_all[i];
  p.rank = rank;
  ag_gemm_kernel<<<NWG, NTHREADS, LDS_BYTES, stream>>>(p);
}
```

```cpp
#include <hip/hip_runtime.h>
#include <rccl/rccl.h>
#include <cstdio>
#include <cstdint>
#include <type_traits>

#define HIP_CHECK(expr) do { hipError_t _e = (expr); if (_e != hipSuccess) { \
    fprintf(stderr, "HIP error %s at %s:%d\n", hipGetErrorString(_e), __FILE__, __LINE__); \
    return; } } while (0)

typedef int v2i __attribute__((ext_vector_type(2)));
typedef int v4i __attribute__((ext_vector_type(4)));
typedef unsigned v4u __attribute__((ext_vector_type(4)));
typedef int v8i __attribute__((ext_vector_type(8)));
typedef float v4f __attribute__((ext_vector_type(4)));
#define LDSP(T) __attribute__((address_space(3))) T*
#define NEWMAP 1

constexpr int NR = 8;
constexpr int M = 4096, N = 2048, K = 4096;
constexpr int MS = M / NR;
constexpr int BM = 256, BN = 128, BK = 128;
constexpr int X_BYTES = BM * BK, W_BYTES = BK * BN + 48 * BK;
#define NSWV 4
constexpr int NSX = 2, NSW = NSWV;
constexpr int NKS = K / BK;
constexpr int KC = 256;
constexpr int NCH = K / KC;
#define SPL 1
constexpr int NCT = NCH + SPL;
#define HN 3
#define HE 2
constexpr int NCT2 = NCT + 1;
constexpr int PARTS = 64;
constexpr int ARRIVALS = 4;
constexpr int NWG = 256, GWG = 224, CWG = NWG - GWG;
constexpr int NTHREADS = 768;
#define PWV 7
#define PLW 3
#define TL 1
#define WPRE 2
#define ZSL 32
#define P2 9
#define P3 P2
#define WARM 1
#define ZB 10
#define JROT 1
#define JSH 0
#define PPRIO 0
#define EFREE 1
#define Q3 10
constexpr int WT = 17;
constexpr unsigned NPUSH = CWG * PWV;
constexpr int LDS_W = NSX * X_BYTES;
constexpr int LDS_CTRL = LDS_W + NSW * W_BYTES;
constexpr int LDS_DBG = LDS_CTRL + 128;
constexpr int LDS_BYTES = LDS_DBG + 768;
constexpr size_t WS_TMO = 0;
constexpr size_t WS_CNT = 4096;
constexpr size_t WS_PTICK = 69632;
constexpr size_t WS_ZC = 73728;
constexpr size_t WS_PCALL = 524288;
constexpr int NITEM = NCH * PARTS;
constexpr size_t WS_STG = 1u << 20;
constexpr size_t STG_BYTES = (size_t)M * K;
constexpr size_t WS_NEED = WS_STG + 2 * STG_BYTES;
constexpr unsigned long long DEADLINE = 40000000ull;

struct Params {
  const unsigned char* x; const unsigned char* w; const float* sx; const float* sw; float* out;
  unsigned char* ws[NR];
  int rank;
};

__host__ __device__ inline int tile_tm(int lb, int rank) { return ((rank + 1 + (lb >> 5)) & 7) * 2 + (lb & 1); }

__device__ __forceinline__ void glds16x4(const unsigned char* s0, const unsigned char* s1, const unsigned char* s2, const unsigned char* s3,
                                         unsigned v0, unsigned v1, unsigned v2, unsigned v3, unsigned lds_off) {
  asm volatile("s_mov_b32 m0, %8\n\ts_nop 0\n\tglobal_load_lds_dwordx4 %0, %4\n\tglobal_load_lds_dwordx4 %1, %5 offset:1024\n\t"
               "global_load_lds_dwordx4 %2, %6 offset:2048\n\tglobal_load_lds_dwordx4 %3, %7 offset:3072"
               :: "v"(v0), "v"(v1), "v"(v2), "v"(v3), "s"(s0), "s"(s1 - 1024), "s"(s2 - 2048), "s"(s3 - 3072), "s"(lds_off) : "memory", "m0");
}

template <typename T> __device__ __forceinline__ T* uni(T* p) {
  unsigned long long v = (unsigned long long)p;
  unsigned lo = __builtin_amdgcn_readfirstlane((unsigned)v), hi = __builtin_amdgcn_readfirstlane((unsigned)(v >> 32));
  return (T*)(((unsigned long long)hi << 32) | lo);
}

#define PW(n) asm volatile("s_waitcnt vmcnt(%0)" :: "i"(n) : "memory")
#define ST2(rs, voff, mi, nj, A, B) do { v4f a_ = (A) * sc, b_ = (B) * sc; \
    _Pragma("unroll") for (int e_ = 0; e_ < 4; ++e_) { \
      auto r_ = __builtin_amdgcn_permlane16_swap(__builtin_bit_cast(unsigned, fmaxf(a_[e_], 0.f)), __builtin_bit_cast(unsigned, fmaxf(b_[e_], 0.f)), false, false); \
      __builtin_amdgcn_raw_buffer_store_b32(r_[0], rs, (voff) + (nj) * 128, ((mi) * 16 + e_) * N * 4, WT); \
      __builtin_amdgcn_raw_buffer_store_b32(r_[1], rs, (voff) + (nj) * 128, ((mi) * 16 + 4 + e_) * N * 4, WT); } } while (0)

__global__ void __launch_bounds__(NTHREADS, 3) ag_gemm_kernel(Params p) {
  extern __shared__ __attribute__((aligned(128))) unsigned char smem[];
  const int tid = threadIdx.x, l = tid & 63, w = __builtin_amdgcn_readfirstlane(tid >> 6);
  unsigned char* myws = p.ws[p.rank];
  const unsigned lds0 = (unsigned)(uintptr_t)smem;
  LDSP(unsigned) ctrl = (LDSP(unsigned))(uintptr_t)(lds0 + LDS_CTRL);
  const int lb = (blockIdx.x + GWG) & (NWG - 1);

  if (lb >= GWG) {
    if (w >= PWV) return;
    const unsigned v = (unsigned)((lb - GWG) * PWV + w);
    static_assert(PWV == NR - 1, "one wave per destination");
    const unsigned j = (unsigned)(w + JROT * ((lb - GWG) >> JSH)) % (NR - 1), nj = CWG;
    const unsigned tfirst = v / (NR - 1);
    const unsigned loff = (unsigned)(l >> 4) * K + (l & 15) * 16;
    const unsigned loff2 = (unsigned)(l >> 3) * K + (l & 7) * 16;
    constexpr unsigned TB = NITEM - TL * PARTS;
    auto canon = [&](unsigned t) {
      if (t < TB) return t;
      const unsigned u = t - TB, b = u / (8 * TL), r = u % (8 * TL);
      return TB + (r >> 3) * 64 + b * 8 + (r & 7);
    };
    auto spl = [&](unsigned c) { return c >= (unsigned)(NITEM - PARTS); };
    auto item_off = [&](unsigned c) {
      return spl(c) ? (((c >> 1) & 1) * 256 + ((c >> 3) & 7) * 32 + (c & 1) * 16) * K + (NCH - 1) * KC + ((c >> 2) & 1) * 128 + loff2
                    : (((c >> 2) & 1) * 256 + ((c >> 3) & 7) * 32 + (c & 3) * 8) * K + (c / PARTS) * KC + loff; };
    auto item_st = [&](unsigned c) { return spl(c) ? 8u * K : 4u * K; };
#define PST(d, t, i) __builtin_amdgcn_raw_buffer_store_b128(d, rs, o0, (i) * item_st(t), WT)
    auto cidx = [&](unsigned c) {
      return spl(c) ? ((((c >> 1) & 1) * NCT2 + NCH - 1 + ((c >> 2) & 1)) * 32 + ((c >> 3) & 7)) | 0x20000u
                    : ((((c >> 2) & 1) * NCT2 + c / PARTS) * 32 + ((c >> 3) & 7)) | 0x10000u; };
    constexpr unsigned TH = (NCH - HN) * PARTS;
    auto tk = [&](unsigned t) { t = __builtin_amdgcn_readfirstlane(t) + 3 * nj; return t < (unsigned)NITEM ? canon(t) : t; };
    unsigned* pcall = (unsigned*)(myws + WS_PCALL + lb * 128) + w;
    unsigned t0 = tfirst, t1;
    unsigned o0 = item_off(t0), o1;
    v4u cur[2], nxt[2];
    unsigned n = __hip_atomic_load(pcall, __ATOMIC_RELAXED, __HIP_MEMORY_SCOPE_AGENT);
    #pragma unroll
    for (int i = 0; i < 2; ++i) cur[i] = *(const v4u*)(p.x + o0 + i * item_st(t0));
    o1 = item_off(t0 + nj);
    #pragma unroll
    for (int i = 0; i < 2; ++i) nxt[i] = *(const v4u*)(p.x + o1 + i * item_st(t0));
    n = __builtin_amdgcn_readfirstlane(n);
    const unsigned parity = (n + 1) & 1;
    unsigned vz; asm volatile("v_mov_b32 %0, 0" : "=v"(vz));
    const unsigned peer = (unsigned)(p.rank + 1 + j) % NR;
    unsigned char* dws = p.ws[peer];
    __amdgpu_buffer_rsrc_t rs = __builtin_amdgcn_make_buffer_rsrc(dws + WS_STG + parity * STG_BYTES + (size_t)p.rank * MS * K, 0, MS * K, 0x00020000);
    unsigned* cnt = (unsigned*)(dws + WS_CNT + (size_t)p.rank * 2 * NCT2 * 128) + vz;
    unsigned* tick = (unsigned*)(myws + WS_PTICK + parity * 1024 + j * 128) + vz;
    unsigned cprev;
#define PSIG() do { if (l == 0) __hip_atomic_fetch_add(cnt + (cprev & 0xffffu), cprev >> 16, __ATOMIC_RELAXED, __HIP_MEMORY_SCOPE_SYSTEM); } while (0)
#define PHINT() do { if (HN && t0 >= TH) { PSIG(); cprev = 0; } } while (0)
#define DRAW(t) do { if (l == 0) t = __hip_atomic_fetch_add(tick, 1u, __ATOMIC_RELAXED, __HIP_MEMORY_SCOPE_AGENT); } while (0)
    unsigned t2 = 0;
    asm volatile("" :: "v"(cur[0]), "v"(cur[1]));
    asm volatile("" :: "v"(nxt[0]), "v"(nxt[1]));
    PST(cur[0], t0, 0);
    PST(cur[1], t0, 1);
    if (v == 0 && l < 8)
      __hip_atomic_store((unsigned*)(myws + WS_PTICK + (parity ^ 1) * 1024) + l * 32, 0u, __ATOMIC_RELAXED, __HIP_MEMORY_SCOPE_SYSTEM);
    cprev = cidx(t0);
    t2 = tfirst - nj;
    asm volatile("" : "+v"(t2) :: "memory");
    t0 += nj; o0 = o1;
    unsigned s1;
    cur[0] = nxt[0]; cur[1] = nxt[1];
    t1 = tk(t2);
    bool hp = false;
    asm volatile("" :: "v"(cur[0]), "v"(cur[1]));
    while (t0 < (unsigned)NITEM) {
      o1 = item_off(t1 < (unsigned)NITEM ? t1 : t0); s1 = item_st(t1 < (unsigned)NITEM ? t1 : t0);
      DRAW(t2);
      #pragma unroll
      for (int i = 0; i < 2; ++i) nxt[i] = *(const v4u*)(p.x + o1 + i * s1);
      if (HN && HE == 0 && hp) PW(5); else PW(4);
      PST(cur[0], t0, 0);
      if (HN && HE == 0 && hp) PW(5); else PW(4);
      if (cprev) PSIG();
      cprev = cidx(t0);
      if (HE) PHINT();
      PST(cur[1], t0, 1);
      asm volatile("s_nop 1" ::: "memory");
      if (!HE) PHINT();
      asm volatile("" ::: "memory");
      hp = t0 >= TH; t0 = t1; o0 = o1;
      t1 = tk(t2);
      #pragma unroll
      for (int i = 0; i < 2; ++i) cur[i] = nxt[i];
      asm volatile("" :: "v"(cur[0]), "v"(cur[1]));
    }
    { const unsigned long long t0 = __builtin_amdgcn_s_memrealtime();
      while ((unsigned)__builtin_amdgcn_readfirstlane(__hip_atomic_load((const unsigned*)(myws + WS_ZC), __ATOMIC_RELAXED, __HIP_MEMORY_SCOPE_AGENT)) < (n + 1) * GWG &&
             __builtin_amdgcn_s_memrealtime() - t0 < DEADLINE) {} }
    PW(0);
    if (cprev) PSIG();
    cprev = (NCT * 32) | 0x10000u;
    PSIG();
    if (l == 0) __hip_atomic_store(pcall, n + 1, __ATOMIC_RELAXED, __HIP_MEMORY_SCOPE_AGENT);
    return;
  }

  const int xcd = lb & 7, r7 = lb >> 5, hf = lb & 1;
  const int tm = tile_tm(lb, p.rank), tn = (xcd >> 1) * 4 + ((lb >> 3) & 3);
  const int src = tm >> 1;
  constexpr unsigned NS = NKS;
  const unsigned* oc = (const unsigned*)(myws + WS_CNT + ((size_t)src * 2 * NCT2 + NCT) * 128);
  const unsigned npw = (NPUSH + r7) / (NR - 1);
  const int lq = r7 * 2 + hf;
  const bool x3 = lq >= Q3 && lq < Q3 + 4;
  const int lr3 = x3 ? (28 + lq - Q3) * 16 : lq * 32;
  const unsigned P = x3 ? P3 : P2, H = NKS - P;

  if (w == 11) {
    unsigned* pcall = (unsigned*)(myws + WS_PCALL + lb * 128);
    unsigned n = __hip_atomic_load(pcall, __ATOMIC_RELAXED, __HIP_MEMORY_SCOPE_AGENT);
    __builtin_amdgcn_s_barrier();
    const unsigned call = __builtin_amdgcn_readfirstlane(n) + 1;
    const unsigned tag = 0x100u | ((call & 1) << 9);
    LDSP(unsigned) pub = ctrl + 17 + 2 * (l & 7);
    if (l == 0) __hip_atomic_store(ctrl + 4, call, __ATOMIC_RELAXED, __HIP_MEMORY_SCOPE_WORKGROUP);
    asm volatile("s_waitcnt lgkmcnt(0)" ::: "memory");
    if (l < 8) __hip_atomic_store(pub, tag, __ATOMIC_RELAXED, __HIP_MEMORY_SCOPE_WORKGROUP);
    __builtin_amdgcn_fence(__ATOMIC_ACQUIRE, "");
    asm volatile("s_waitcnt vmcnt(0)" ::: "memory");
    const unsigned* cnt = (const unsigned*)(myws + WS_CNT + (size_t)tm * NCT2 * 128) + (l >> 3) * 32 + (l & 7);
    unsigned* tmo = (unsigned*)(myws + WS_TMO);
    const unsigned target = call * ARRIVALS;
    unsigned base = 0, hv = 0;
    const unsigned long long t0 = __builtin_amdgcn_s_memrealtime();
    while (base < (unsigned)NCT) {
      bool ok = false;
      if (l < 8 * PLW && base + (l >> 3) < (unsigned)NCT) ok = __hip_atomic_load(cnt + base * 32, __ATOMIC_RELAXED, __HIP_MEMORY_SCOPE_SYSTEM) >= target;
      const unsigned long long m = __builtin_amdgcn_ballot_w64(ok);
      const unsigned now = base + ((unsigned)__builtin_ctzll(~(m >> (l & 7)) & 0x0101010101010101ull) >> 3);
      if (__builtin_amdgcn_ballot_w64(now > hv)) {
        hv = now > hv ? now : hv;
        if (l < 8) __hip_atomic_store(pub, tag | (hv < (unsigned)NCH ? 2 * hv : NCH - 1 + hv), __ATOMIC_RELAXED, __HIP_MEMORY_SCOPE_WORKGROUP);
        unsigned k = 0;
        while (k < (unsigned)PLW && ((m >> (8 * k)) & 255u) == 255u) ++k;
        base += k;
      } else if (__builtin_amdgcn_s_memrealtime() - t0 > DEADLINE ||
                 ((__builtin_amdgcn_s_memrealtime() - t0 > 100000) && __hip_atomic_load(tmo, __ATOMIC_RELAXED, __HIP_MEMORY_SCOPE_AGENT) == call)) {
        if (l == 0) {
          __hip_atomic_store(tmo, call, __ATOMIC_RELAXED, __HIP_MEMORY_SCOPE_AGENT);
          __hip_atomic_store(ctrl, 1u, __ATOMIC_RELAXED, __HIP_MEMORY_SCOPE_WORKGROUP);
        }
        asm volatile("s_waitcnt lgkmcnt(0)" ::: "memory");
        if (l < 8) __hip_atomic_store(pub, tag | NKS, __ATOMIC_RELAXED, __HIP_MEMORY_SCOPE_WORKGROUP);
        break;
      }
    }
    while ((unsigned)__builtin_amdgcn_readfirstlane(__hip_atomic_load(oc, __ATOMIC_RELAXED, __HIP_MEMORY_SCOPE_SYSTEM)) < call * npw &&
           __builtin_amdgcn_s_memrealtime() - t0 < DEADLINE) __builtin_amdgcn_s_sleep(8);
    if (l == 0) __hip_atomic_store(pcall, call, __ATOMIC_RELAXED, __HIP_MEMORY_SCOPE_AGENT);
    return;
  }
  if (w >= 8) {
    const int h = w == 8 ? 2 : w - 9;
    const unsigned char* Wb = uni(h < 2 ? p.w + (size_t)tn * BN + (size_t)h * 64 * N : p.x + (size_t)lq * 32 * K);
    const unsigned char* Lb3 = uni(p.x + (size_t)lr3 * K);
    unsigned boff[2];
    #pragma unroll
    for (int i = 0; i < 2; ++i) boff[i] = h < 2 ? (l >> 3) * N + (((l & 7) ^ (((l >> 4) & 3) | (i << 2))) * 16)
                                                : (l >> 3) * K + (((l & 7) ^ (((l >> 4) & 1) | (i * 6))) * 16);
    auto ldw = [&](unsigned jj, int b) {
      const unsigned j = (jj + H) & (NKS - 1);
      const unsigned ld = lds0 + LDS_W + b * W_BYTES + h * 8192;
      if (h < 2) {
        const unsigned char* wb = Wb + (size_t)j * BK * N;
        #pragma unroll
        for (int i = 0; i < 8; i += 4) glds16x4(wb + (size_t)i * 8 * N, wb + (size_t)(i + 1) * 8 * N, wb + (size_t)(i + 2) * 8 * N, wb + (size_t)(i + 3) * 8 * N,
                                                boff[i >> 2], boff[i >> 2], boff[i >> 2], boff[i >> 2], ld + i * 1024);
      } else {
        const unsigned char* wb = Wb + j * BK;
        glds16x4(wb, wb + 8 * K, wb + 16 * K, wb + 24 * K, boff[0], boff[1], boff[0], boff[1], ld);
        if (x3) asm volatile("s_mov_b32 m0, %4\n\ts_nop 0\n\tglobal_load_lds_dwordx4 %0, %2\n\tglobal_load_lds_dwordx4 %1, %3 offset:1024"
                             :: "v"(boff[0]), "v"(boff[1]), "s"(Lb3 + j * BK), "s"(Lb3 + j * BK + 8 * K - 1024), "s"(ld + 4096) : "memory", "m0");
      }
    };
    bool z = false;
    auto prev = [&]() { if (h < 2) PW(8); else if (x3) { if (z) PW(7); else PW(6); } else { if (z) PW(5); else PW(4); } };
    const unsigned inc = 1u << (10 * h), nst = h < 2 ? NS + P : NS;
    auto sig = [&](unsigned k) { if (l < 8) __hip_atomic_fetch_add(ctrl + 16 + 2 * l, inc * k, __ATOMIC_RELAXED, __HIP_MEMORY_SCOPE_WORKGROUP); };
    auto ldf = [&](int b) { return (unsigned)__builtin_amdgcn_readfirstlane(__hip_atomic_load(ctrl + 8 + b, __ATOMIC_RELAXED, __HIP_MEMORY_SCOPE_WORKGROUP)); };
    ldw(0, 0); ldw(1, 1);
    __builtin_amdgcn_s_barrier();
    unsigned done = 1; int buf = 2;
    unsigned fr = 0;
    prev(); sig(1);
    for (unsigned j = 2; j < nst; ++j) {
      if (buf >= NSW) { buf = 0; fr += 8; }
      if (fr && ldf(buf) < fr) {
        if (done < j) { if (z) PW(1); else PW(0); sig(j - done); done = j; }
        while (ldf(buf) < fr) {}
      }
      asm volatile("" ::: "memory");
      ldw(j, buf);
      z = h == 2 && j - ZB < 16u;
      if (z) {
        static_assert(HN <= 4 && ZB >= NSW, "one store per row, after the call number is known");
        const unsigned call = __builtin_amdgcn_readfirstlane(__hip_atomic_load(ctrl + 4, __ATOMIC_RELAXED, __HIP_MEMORY_SCOPE_WORKGROUP));
        __amdgpu_buffer_rsrc_t zs = __builtin_amdgcn_make_buffer_rsrc(myws + WS_STG + ((call & 1) ^ 1) * STG_BYTES + (size_t)(tm * BM + tn * 16) * K, 0, 16 * K, 0x00020000);
        __builtin_amdgcn_raw_buffer_store_b128((v4u){0, 0, 0, 0}, zs, l < HN * 16 ? (NCH - HN) * KC + l * 16 + (j - ZB) * K : 16 * K, 0, WT);
      }
      if (done < j) { prev(); sig(j - done); done = j; }
      ++buf;
    }
    PW(0);
    sig(NS + P - done);
    if (h == 2 && l == 0) __hip_atomic_fetch_add((unsigned*)(myws + WS_ZC), 1u, __ATOMIC_RELAXED, __HIP_MEMORY_SCOPE_AGENT);
    return;
  }

  const unsigned char* Xb;
  unsigned aoff[4];
  #pragma unroll
  for (int i = 0; i < 4; ++i) { int c = (l & 7) ^ (((l >> 4) & 1) | ((i & 1) * 6)); aoff[i] = (i * 8 + (l >> 3)) * K + c * 16 - i * 1024; }
  unsigned nx = 0;
  unsigned xm0 = lds0 + w * 4096;
  auto ldx = [&]() {
    asm volatile("s_mov_b32 m0, %5\n\ts_nop 0\n\tglobal_load_lds_dwordx4 %0, %4\n\tglobal_load_lds_dwordx4 %1, %4 offset:1024\n\t"
                 "global_load_lds_dwordx4 %2, %4 offset:2048\n\tglobal_load_lds_dwordx4 %3, %4 offset:3072"
                 :: "v"(aoff[0]), "v"(aoff[1]), "v"(aoff[2]), "v"(aoff[3]), "s"(Xb), "s"(xm0) : "memory", "m0");
    xm0 ^= X_BYTES; ++nx; Xb += BK;
  };
  if (tid < 32) ctrl[tid] = 0u;
  asm volatile("s_waitcnt lgkmcnt(0)" ::: "memory");
  __builtin_amdgcn_s_barrier();
  const int g = l >> 4, r16 = l & 15;
  const unsigned xs = ((r16 >> 1) & 1) | (((r16 >> 3) & 1) * 6);
  const unsigned xa0 = lds0 + (w * 32 + r16) * 128 + (((2 * g) ^ xs) * 16);
  const int q = r16 >> 1, pp = l & 1;
  const unsigned sB = ((q >> 1) & 3) | ((g & 1) << 2);
  const unsigned wa0 = lds0 + LDS_W + (32 * g + q) * 128 + (sB ^ (w & 6)) * 16 + 8 * pp;
  const unsigned la0 = lds0 + LDS_W + BK * BN + r16 * 128 + (((2 * g) ^ xs) * 16);
  LDSP(unsigned long long) flg = (LDSP(unsigned long long))(uintptr_t)(lds0 + LDS_CTRL + 64 + w * 8);
  auto ldflg = [&]() { return __hip_atomic_load(flg, __ATOMIC_RELAXED, __HIP_MEMORY_SCOPE_WORKGROUP); };
  v4f acc[8][2];
  #pragma unroll
  for (int a = 0; a < 8; ++a)
    #pragma unroll
    for (int b = 0; b < 2; ++b) acc[a][b] = (v4f){0, 0, 0, 0};
  v4f acl[3] = {(v4f){0, 0, 0, 0}, (v4f){0, 0, 0, 0}, (v4f){0, 0, 0, 0}};
  v8i xf[2], wf[4], xl;
#define RW(sl, ni, adr) do { _Pragma("unroll") for (int i_ = 0; i_ < 4; ++i_) { \
    v2i t_ = __builtin_amdgcn_ds_read_tr8_b64_v2i32((LDSP(v2i))(uintptr_t)(((adr) ^ ((ni) * 16)) + i_ * 1024)); \
    wf[sl][2 * i_] = t_.x; wf[sl][2 * i_ + 1] = t_.y; } } while (0)
#define RX(mi, adr) do { v4i t0_ = *(LDSP(v4i))(uintptr_t)((adr) + (mi) * 2048); v4i t1_ = *(LDSP(v4i))(uintptr_t)(((adr) ^ 16) + (mi) * 2048); \
    xf[mi] = (v8i){t0_.x, t0_.y, t0_.z, t0_.w, t1_.x, t1_.y, t1_.z, t1_.w}; } while (0)
#define MM(mi, ni) acc[ni][mi] = __builtin_amdgcn_mfma_scale_f32_16x16x128_f8f6f4(xf[mi], wf[(ni) & 3], acc[ni][mi], 0, 1, 0, 0, 0, 0)
#define RL(adr) do { v4i t0_ = *(LDSP(v4i))(uintptr_t)(adr); v4i t1_ = *(LDSP(v4i))(uintptr_t)((adr) ^ 16); \
    xl = (v8i){t0_.x, t0_.y, t0_.z, t0_.w, t1_.x, t1_.y, t1_.z, t1_.w}; } while (0)
#define ML(a, ni) acl[a] = __builtin_amdgcn_mfma_scale_f32_16x16x128_f8f6f4(xl, wf[ni], acl[a], 0, 1, 0, 0, 0, 0)
#define SB __builtin_amdgcn_sched_barrier(0)
  float sc;
  { float s0_, s1_;
    asm volatile("s_load_dword %0, %2, 0x0\n\ts_load_dword %1, %3, 0x0\n\ts_waitcnt lgkmcnt(0)" : "=&s"(s0_), "=&s"(s1_) : "s"(p.sx), "s"(p.sw) : "memory");
    sc = __builtin_bit_cast(float, __builtin_amdgcn_readfirstlane(__builtin_bit_cast(int, s0_ * s1_))); }
  asm volatile("" : "+s"(sc));
  float* outp = uni(p.out + (size_t)tm * BM * N + tn * BN);
  asm volatile("" : "+s"(outp));
  __amdgpu_buffer_rsrc_t ors = __builtin_amdgcn_make_buffer_rsrc(outp, 0, BM * N * 4, 0x00020000);
  unsigned have, wl;
  do { unsigned long long f = ldflg(); have = __builtin_amdgcn_readfirstlane((unsigned)(f >> 32)); } while (have == 0);
  Xb = uni(myws + WS_STG + ((have >> 9) & 1) * STG_BYTES + (size_t)(tm * BM + w * 32) * K);
  have &= 255u;
  auto rdf = [&](unsigned long long f) { have = __builtin_amdgcn_readfirstlane((unsigned)(f >> 32)) & 255u; wl = __builtin_amdgcn_readfirstlane((unsigned)f); };
  auto wlanded = [&](unsigned la) { return (((wl | 0x20080200u) - la * 0x100401u) & 0x20080200u) == 0x20080200u; };
  auto needx = [&](unsigned s) {
    while (nx <= s) { if (nx < have) ldx(); else rdf(ldflg()); }
    if (nx == s + 1 && nx < have) ldx();
    if (nx == s + 1) PW(0); else PW(4);
  };
  const unsigned one = 1u;
  const unsigned sval = NS - 2 * HN;
  auto xbad = [&]() {
    const unsigned a = xf[0][0] | xf[0][1] | xf[0][2] | xf[0][3] | xf[0][4] | xf[0][5] | xf[0][6] | xf[0][7];
    const unsigned b = xf[1][0] | xf[1][1] | xf[1][2] | xf[1][3] | xf[1][4] | xf[1][5] | xf[1][6] | xf[1][7];
    return __builtin_amdgcn_ballot_w64((a < b ? a : b) == 0u) != 0ull; };
  auto refetch = [&](unsigned s) {
    const unsigned call = __builtin_amdgcn_readfirstlane(__hip_atomic_load(ctrl + 4, __ATOMIC_RELAXED, __HIP_MEMORY_SCOPE_WORKGROUP));
    const unsigned char* xs_ = uni(Xb - (size_t)(nx - s) * BK);
    const unsigned xa = xa0 + (s & 1) * X_BYTES, xm = lds0 + w * 4096 + (s & 1) * X_BYTES;
    const unsigned long long t0 = __builtin_amdgcn_s_memrealtime();
    bool fin;
    do {
      fin = __builtin_amdgcn_readfirstlane(__hip_atomic_load(oc, __ATOMIC_RELAXED, __HIP_MEMORY_SCOPE_SYSTEM)) >= call * npw;
      asm volatile("s_waitcnt vmcnt(0)\n\ts_mov_b32 m0, %5\n\ts_nop 0\n\tglobal_load_lds_dwordx4 %0, %4 sc0 sc1\n\tglobal_load_lds_dwordx4 %1, %4 offset:1024 sc0 sc1\n\t"
                   "global_load_lds_dwordx4 %2, %4 offset:2048 sc0 sc1\n\tglobal_load_lds_dwordx4 %3, %4 offset:3072 sc0 sc1\n\ts_waitcnt vmcnt(0)"
                   :: "v"(aoff[0]), "v"(aoff[1]), "v"(aoff[2]), "v"(aoff[3]), "s"(xs_), "s"(xm) : "memory", "m0");
      RX(0, xa); RX(1, xa);
      if (__builtin_amdgcn_s_memrealtime() - t0 > DEADLINE || __builtin_amdgcn_readfirstlane(__hip_atomic_load(ctrl, __ATOMIC_RELAXED, __HIP_MEMORY_SCOPE_WORKGROUP))) {
        if (l == 0) __hip_atomic_store(ctrl, 1u, __ATOMIC_RELAXED, __HIP_MEMORY_SCOPE_WORKGROUP);
        fin = true;
      }
    } while (!fin && xbad());
  };
#define FREE(b) asm volatile("s_mov_b64 exec, 1\n\tds_add_u32 %0, %1\n\ts_mov_b64 exec, -1" :: "v"(lds0 + LDS_CTRL + 32 + (b) * 4), "v"(one) : "memory")
  int buf = 0;
  for (unsigned j = 0; j < P; ++j) {
    do rdf(ldflg()); while (!wlanded(j + 1));
    asm volatile("" ::: "memory");
    const unsigned wa = wa0 + buf * W_BYTES, la = la0 + buf * W_BYTES;
    RW(0, 0, wa); RL(la + (w & 1) * 2048); RW(1, 1, wa);
    if (x3) { RW(2, 2, wa ^ ((w & 1) * 16)); RX(0, la + 4096); }
    FREE(buf);
    ML(0, 0); ML(1, 1);
    if (x3) acl[2] = __builtin_amdgcn_mfma_scale_f32_16x16x128_f8f6f4(xf[0], wf[2], acl[2], 0, 1, 0, 0, 0, 0);
    if (nx < 2 && nx < have) ldx();
    if (++buf >= NSW) buf = 0;
  }
  needx(0);
  RX(0, xa0); RX(1, xa0);
  do rdf(ldflg()); while (!wlanded(P + 1));
  asm volatile("" ::: "memory");
  { const unsigned wa = wa0 + buf * W_BYTES;
    RW(0, 0, wa); RL(la0 + buf * W_BYTES + (w & 1) * 2048); RW(1, 1, wa); RW(2, 2, wa); }
  const bool x3e = x3 && !(w & 1), x3o = x3 && (w & 1);
  auto step = [&](auto loc, unsigned s) {
    constexpr bool LOC = decltype(loc)::value;
    const unsigned wa = wa0 + buf * W_BYTES;
    int nb = buf + 1; if (nb >= NSW) nb = 0;
    const unsigned wn = wa0 + nb * W_BYTES;
    SB; RW(3, 3, wa); SB;
    if (LOC) ML(0, 0);
    MM(0, 0); MM(1, 0); SB;
    if (nx <= s + 2 && nx < have) ldx();
    SB; RW(0, 4, wa); SB;
    if (LOC) ML(1, 1);
    MM(0, 1); SB;
    if (LOC && x3) RL(la0 + buf * W_BYTES + 4096);
    SB; MM(1, 1); SB;
    const unsigned long long f = ldflg();
    RW(1, 5, wa); SB;
    MM(0, 2); MM(1, 2); SB;
    if (LOC && x3e) ML(2, 2);
    SB; RW(2, 6, wa); SB;
    MM(0, 3); MM(1, 3); SB;
    if (LOC && x3o) ML(2, 3);
    SB; RW(3, 7, wa); SB;
    FREE(buf); SB;
    MM(0, 4); MM(1, 4); SB;
    rdf(f);
    if (nx <= s + 2 && nx < have) ldx();
    while (!wlanded(P + s + 2)) rdf(ldflg());
    asm volatile("" ::: "memory");
    SB; RW(0, 0, wn); SB;
    MM(0, 5); MM(1, 5); SB;
    if (LOC) RL(la0 + nb * W_BYTES + (w & 1) * 2048);
    RW(1, 1, wn); SB;
    MM(0, 6); MM(1, 6); SB;
    RW(2, 2, wn); SB;
    MM(0, 7); SB;
    SB; MM(1, 7); SB;
    needx(s + 1);
    const unsigned xa = xa0 + ((s + 1) & 1) * X_BYTES;
    RX(0, xa); RX(1, xa);
    if (s + 1 >= sval) { if (__builtin_expect(xbad(), 0)) refetch(s + 1); }
    buf = nb;
  };
  for (unsigned s = 0; s < H; ++s) step(std::true_type(), s);
  {
    __amdgpu_buffer_rsrc_t lrs = __builtin_amdgcn_make_buffer_rsrc(uni(p.out + (size_t)p.rank * MS * N + tn * BN), 0, MS * N * 4 - tn * BN * 4, 0x00020000);
    ST2(lrs, (unsigned)((lq * 32 + (w & 1) * 16 + (l >> 5) * 8) * N + (l & 31)) * 4 + (w >> 1) * 128, 0, 0, acl[0], acl[1]);
    if (x3) {
      #pragma unroll
      for (int e = 0; e < 4; ++e)
        __builtin_amdgcn_raw_buffer_store_b32(__builtin_bit_cast(unsigned, fmaxf(acl[2][e] * sc, 0.f)), lrs, (unsigned)((lr3 + g * 4) * N + (w ^ 2) * 16 + r16) * 4, e * N * 4, WT);
    }
  }
  for (unsigned s = H; s < NS - 1; ++s) step(std::false_type(), s);
  const unsigned ooff = (unsigned)((w * 32 + (l >> 5) * 8) * N + (l & 31)) * 4 + (w >> 1) * 128;
#define STO(mi, nj) ST2(ors, ooff ^ ((nj) * 128), mi, 0, acc[2 * (nj)][mi], acc[2 * (nj) + 1][mi])
  {
    const unsigned wa = wa0 + buf * W_BYTES;
    SB; RW(3, 3, wa); SB;
    MM(0, 0); MM(1, 0); SB;
    RW(0, 4, wa); SB;
    MM(0, 1); MM(1, 1); SB;
    RW(1, 5, wa); SB;
    MM(0, 2); MM(1, 2); SB;
    RW(2, 6, wa); SB;
    MM(0, 3); MM(1, 3); SB;
    RW(3, 7, wa); SB;
    SB;
    MM(0, 4); MM(1, 4); STO(0, 0); SB;
    MM(0, 5); MM(1, 5); STO(1, 0); SB;
    MM(0, 6); MM(1, 6); STO(0, 1); SB;
    MM(0, 7); MM(1, 7); STO(1, 1); SB;
    STO(0, 2); STO(1, 2); STO(0, 3); STO(1, 3);
  }
  if (__builtin_amdgcn_readfirstlane(__hip_atomic_load(ctrl, __ATOMIC_RELAXED, __HIP_MEMORY_SCOPE_WORKGROUP))) {
    for (int i = 0; i < 64; ++i) __builtin_amdgcn_raw_buffer_store_b32(0x7fc00000u, ors, (unsigned)((w * 32 + (i >> 1)) * N + (i & 1) * 64 + l) * 4, 0, WT);
  }
}

extern "C" void kernel_launch(int rank, int nranks, ncclComm_t comm,
                              hipStream_t stream,
                              void* const* d_in, const int* in_sizes, int n_in,
                              void* d_out, int out_size,
                              const int* in_dtypes, int out_dtype,
                              void* const* d_out_all,
                              void* const* d_ws_all, size_t ws_bytes) {
  static bool init_a[NR];
  if (nranks != NR || ws_bytes < WS_NEED || (unsigned)rank >= (unsigned)NR) { fprintf(stderr, "bad config\n"); return; }
  if (!init_a[rank]) {
    HIP_CHECK(hipFuncSetAttribute((const void*)ag_gemm_kernel, hipFuncAttributeMaxDynamicSharedMemorySize, LDS_BYTES));
    init_a[rank] = true;
  }
  Params p;
  p.x = (const unsigned char*)d_in[0]; p.w = (const unsigned char*)d_in[1];
  p.sx = (const float*)d_in[2]; p.sw = (const float*)d_in[3]; p.out = (float*)d_out;
  for (int i = 0; i < NR; ++i) p.ws[i] = (unsigned char*)d_ws_all[i];
  p.rank = rank;
  ag_gemm_kernel<<<NWG, NTHREADS, LDS_BYTES, stream>>>(p);
}
```

```cpp
#include <hip/hip_runtime.h>
#include <rccl/rccl.h>
#include <cstdio>
#include <cstdint>
#include <type_traits>

#define HIP_CHECK(expr) do { hipError_t _e = (expr); if (_e != hipSuccess) { \
    fprintf(stderr, "HIP error %s at %s:%d\n", hipGetErrorString(_e), __FILE__, __LINE__); \
    return; } } while (0)

typedef int v2i __attribute__((ext_vector_type(2)));
typedef int v4i __attribute__((ext_vector_type(4)));
typedef unsigned v4u __attribute__((ext_vector_type(4)));
typedef int v8i __attribute__((ext_vector_type(8)));
typedef float v4f __attribute__((ext_vector_type(4)));
#define LDSP(T) __attribute__((address_space(3))) T*
#define NEWMAP 1

constexpr int NR = 8;
constexpr int M = 4096, N = 2048, K = 4096;
constexpr int MS = M / NR;
constexpr int BM = 256, BN = 128, BK = 128;
constexpr int X_BYTES = BM * BK, W_BYTES = BK * BN + 48 * BK;
#define NSWV 4
constexpr int NSX = 2, NSW = NSWV;
constexpr int NKS = K / BK;
constexpr int KC = 256;
constexpr int NCH = K / KC;
#define SPL 1
constexpr int NCT = NCH + SPL;
#define HN 16
#define HE 2
constexpr int NCT2 = NCT + 1;
constexpr int PARTS = 64;
constexpr int ARRIVALS = 4;
constexpr int NWG = 256, GWG = 224, CWG = NWG - GWG;
constexpr int NTHREADS = 768;
#define PWV 7
#define PLW 3
#define TL 1
#define P2 7
#define P3 6
#define ZB 8
#define JROT 1
#define JSH 0
#define PPRIO 0
#define EFREE 1
#define XORD 1
#define Q3 10
constexpr int WT = 17;
constexpr unsigned NPUSH = CWG * PWV;
constexpr int LDS_W = NSX * X_BYTES;
constexpr int LDS_CTRL = LDS_W + NSW * W_BYTES;
constexpr int LDS_DBG = LDS_CTRL + 128;
constexpr int LDS_BYTES = LDS_DBG + 768;
constexpr size_t WS_TMO = 0;
constexpr size_t WS_CNT = 4096;
constexpr size_t WS_PTICK = 69632;
constexpr size_t WS_ZC = 73728;
constexpr size_t WS_PCALL = 524288;
constexpr int NITEM = NCH * PARTS;
constexpr size_t WS_STG = 1u << 20;
constexpr size_t STG_BYTES = (size_t)M * K;
constexpr size_t WS_NEED = WS_STG + 2 * STG_BYTES;
constexpr unsigned long long DEADLINE = 40000000ull;

struct Params {
  const unsigned char* x; const unsigned char* w; const float* sx; const float* sw; float* out;
  unsigned char* ws[NR];
  int rank;
};

__host__ __device__ inline int tile_tm(int lb, int rank) { return ((rank + 1 + (lb >> 5)) & 7) * 2 + (lb & 1); }

__device__ __forceinline__ void glds16x4(const unsigned char* s0, const unsigned char* s1, const unsigned char* s2, const unsigned char* s3,
                                         unsigned v0, unsigned v1, unsigned v2, unsigned v3, unsigned lds_off) {
  asm volatile("s_mov_b32 m0, %8\n\ts_nop 0\n\tglobal_load_lds_dwordx4 %0, %4\n\tglobal_load_lds_dwordx4 %1, %5 offset:1024\n\t"
               "global_load_lds_dwordx4 %2, %6 offset:2048\n\tglobal_load_lds_dwordx4 %3, %7 offset:3072"
               :: "v"(v0), "v"(v1), "v"(v2), "v"(v3), "s"(s0), "s"(s1 - 1024), "s"(s2 - 2048), "s"(s3 - 3072), "s"(lds_off) : "memory", "m0");
}

template <typename T> __device__ __forceinline__ T* uni(T* p) {
  unsigned long long v = (unsigned long long)p;
  unsigned lo = __builtin_amdgcn_readfirstlane((unsigned)v), hi = __builtin_amdgcn_readfirstlane((unsigned)(v >> 32));
  return (T*)(((unsigned long long)hi << 32) | lo);
}

#define PW(n) asm volatile("s_waitcnt vmcnt(%0)" :: "i"(n) : "memory")
#define ST2(rs, voff, mi, nj, A, B) do { v4f a_ = (A) * sc, b_ = (B) * sc; \
    _Pragma("unroll") for (int e_ = 0; e_ < 4; ++e_) { \
      auto r_ = __builtin_amdgcn_permlane16_swap(__builtin_bit_cast(unsigned, fmaxf(a_[e_], 0.f)), __builtin_bit_cast(unsigned, fmaxf(b_[e_], 0.f)), false, false); \
      __builtin_amdgcn_raw_buffer_store_b32(r_[0], rs, (voff) + (nj) * 128, ((mi) * 16 + e_) * N * 4, WT); \
      __builtin_amdgcn_raw_buffer_store_b32(r_[1], rs, (voff) + (nj) * 128, ((mi) * 16 + 4 + e_) * N * 4, WT); } } while (0)

__global__ void __launch_bounds__(NTHREADS, 3) ag_gemm_kernel(Params p) {
  extern __shared__ __attribute__((aligned(128))) unsigned char smem[];
  const int tid = threadIdx.x, l = tid & 63, w = __builtin_amdgcn_readfirstlane(tid >> 6);
  unsigned char* myws = p.ws[p.rank];
  const unsigned lds0 = (unsigned)(uintptr_t)smem;
  LDSP(unsigned) ctrl = (LDSP(unsigned))(uintptr_t)(lds0 + LDS_CTRL);
  const int lb = (blockIdx.x + GWG) & (NWG - 1);

  if (lb >= GWG) {
    if (w >= PWV) return;
    const unsigned v = (unsigned)((lb - GWG) * PWV + w);
    static_assert(PWV == NR - 1, "one wave per destination");
    const unsigned j = (unsigned)(w + JROT * ((lb - GWG) >> JSH)) % (NR - 1), nj = CWG;
    const unsigned tfirst = v / (NR - 1);
    const unsigned loff = (unsigned)(l >> 4) * K + (l & 15) * 16;
    const unsigned loff2 = (unsigned)(l >> 3) * K + (l & 7) * 16;
    constexpr unsigned TB = NITEM - TL * PARTS;
    auto canon = [&](unsigned t) {
      if (t < TB) return t;
      const unsigned u = t - TB, b = u / (8 * TL), r = u % (8 * TL);
      return TB + (r >> 3) * 64 + b * 8 + (r & 7);
    };
    auto spl = [&](unsigned c) { return c >= (unsigned)(NITEM - PARTS); };
    auto item_off = [&](unsigned c) {
      return spl(c) ? (((c >> 1) & 1) * 256 + ((c >> 3) & 7) * 32 + (c & 1) * 16) * K + (NCH - 1) * KC + ((c >> 2) & 1) * 128 + loff2
                    : (((c >> 2) & 1) * 256 + ((c >> 3) & 7) * 32 + (c & 3) * 8) * K + (c / PARTS) * KC + loff; };
    auto item_st = [&](unsigned c) { return spl(c) ? 8u * K : 4u * K; };
#define PST(d, t, i) __builtin_amdgcn_raw_buffer_store_b128(d, rs, o0, (i) * item_st(t), WT)
    auto cidx = [&](unsigned c) {
      return spl(c) ? ((((c >> 1) & 1) * NCT2 + NCH - 1 + ((c >> 2) & 1)) * 32 + ((c >> 3) & 7)) | 0x20000u
                    : ((((c >> 2) & 1) * NCT2 + c / PARTS) * 32 + ((c >> 3) & 7)) | 0x10000u; };
    constexpr unsigned TH = (NCH - HN) * PARTS;
    auto tk = [&](unsigned t) { t = __builtin_amdgcn_readfirstlane(t) + 3 * nj; return t < (unsigned)NITEM ? canon(t) : t; };
    unsigned* pcall = (unsigned*)(myws + WS_PCALL + lb * 128) + w;
    unsigned t0 = tfirst, t1;
    unsigned o0 = item_off(t0), o1;
    v4u cur[2], nxt[2];
    unsigned n = __hip_atomic_load(pcall, __ATOMIC_RELAXED, __HIP_MEMORY_SCOPE_AGENT);
    #pragma unroll
    for (int i = 0; i < 2; ++i) cur[i] = *(const v4u*)(p.x + o0 + i * item_st(t0));
    o1 = item_off(t0 + nj);
    #pragma unroll
    for (int i = 0; i < 2; ++i) nxt[i] = *(const v4u*)(p.x + o1 + i * item_st(t0));
    n = __builtin_amdgcn_readfirstlane(n);
    const unsigned parity = (n + 1) & 1;
    unsigned vz; asm volatile("v_mov_b32 %0, 0" : "=v"(vz));
    const unsigned peer = (unsigned)(p.rank + 1 + j) % NR;
    unsigned char* dws = p.ws[peer];
    __amdgpu_buffer_rsrc_t rs = __builtin_amdgcn_make_buffer_rsrc(dws + WS_STG + parity * STG_BYTES + (size_t)p.rank * MS * K, 0, MS * K, 0x00020000);
    unsigned* cnt = (unsigned*)(dws + WS_CNT + (size_t)p.rank * 2 * NCT2 * 128) + vz;
    unsigned* tick = (unsigned*)(myws + WS_PTICK + parity * 1024 + j * 128) + vz;
    unsigned cprev;
#define PSIG() do { if (l == 0) __hip_atomic_fetch_add(cnt + (cprev & 0xffffu), cprev >> 16, __ATOMIC_RELAXED, __HIP_MEMORY_SCOPE_SYSTEM); } while (0)
#define PHINT() do { if (HN && t0 >= TH) { PSIG(); cprev = 0; } } while (0)
#define DRAW(t) do { if (l == 0) t = __hip_atomic_fetch_add(tick, 1u, __ATOMIC_RELAXED, __HIP_MEMORY_SCOPE_AGENT); } while (0)
    unsigned t2 = 0;
    asm volatile("" :: "v"(cur[0]), "v"(cur[1]));
    asm volatile("" :: "v"(nxt[0]), "v"(nxt[1]));
    PST(cur[0], t0, 0);
    PST(cur[1], t0, 1);
    if (v == 0 && l < 8)
      __hip_atomic_store((unsigned*)(myws + WS_PTICK + (parity ^ 1) * 1024) + l * 32, 0u, __ATOMIC_RELAXED, __HIP_MEMORY_SCOPE_SYSTEM);
    cprev = cidx(t0);
    PHINT();
    t2 = tfirst - nj;
    asm volatile("" : "+v"(t2) :: "memory");
    t0 += nj; o0 = o1;
    unsigned s1;
    cur[0] = nxt[0]; cur[1] = nxt[1];
    t1 = tk(t2);
    bool hp = false;
    asm volatile("" :: "v"(cur[0]), "v"(cur[1]));
    while (t0 < (unsigned)NITEM) {
      o1 = item_off(t1 < (unsigned)NITEM ? t1 : t0); s1 = item_st(t1 < (unsigned)NITEM ? t1 : t0);
      DRAW(t2);
      #pragma unroll
      for (int i = 0; i < 2; ++i) nxt[i] = *(const v4u*)(p.x + o1 + i * s1);
      if (HN && HE == 0 && hp) PW(5); else PW(4);
      PST(cur[0], t0, 0);
      if (HN && HE == 0 && hp) PW(5); else PW(4);
      if (cprev) PSIG();
      cprev = cidx(t0);
      if (HE) PHINT();
      PST(cur[1], t0, 1);
      asm volatile("s_nop 1" ::: "memory");
      if (!HE) PHINT();
      asm volatile("" ::: "memory");
      hp = t0 >= TH; t0 = t1; o0 = o1;
      t1 = tk(t2);
      #pragma unroll
      for (int i = 0; i < 2; ++i) cur[i] = nxt[i];
      asm volatile("" :: "v"(cur[0]), "v"(cur[1]));
    }
    { const unsigned long long t0 = __builtin_amdgcn_s_memrealtime();
      while ((unsigned)__builtin_amdgcn_readfirstlane(__hip_atomic_load((const unsigned*)(myws + WS_ZC), __ATOMIC_RELAXED, __HIP_MEMORY_SCOPE_AGENT)) < (n + 1) * GWG &&
             __builtin_amdgcn_s_memrealtime() - t0 < DEADLINE) {} }
    PW(0);
    if (cprev) PSIG();
    cprev = (NCT * 32) | 0x10000u;
    PSIG();
    if (l == 0) __hip_atomic_store(pcall, n + 1, __ATOMIC_RELAXED, __HIP_MEMORY_SCOPE_AGENT);
    return;
  }

  const int xcd = lb & 7, r7 = lb >> 5, hf = lb & 1;
  const int tm = tile_tm(lb, p.rank), tn = (xcd >> 1) * 4 + ((lb >> 3) & 3);
  const int src = tm >> 1;
  constexpr unsigned NS = NKS;
  const unsigned* oc = (const unsigned*)(myws + WS_CNT + ((size_t)src * 2 * NCT2 + NCT) * 128);
  const unsigned npw = (NPUSH + r7) / (NR - 1);
  const int lq = r7 * 2 + hf;
  const bool x3 = lq >= Q3 && lq < Q3 + 4;
  const int lr3 = x3 ? (28 + lq - Q3) * 16 : lq * 32;
  const unsigned P = x3 ? P3 : P2, H = NKS - P;

  if (w == 11) {
    unsigned* pcall = (unsigned*)(myws + WS_PCALL + lb * 128);
    unsigned n = __hip_atomic_load(pcall, __ATOMIC_RELAXED, __HIP_MEMORY_SCOPE_AGENT);
    __builtin_amdgcn_s_barrier();
    const unsigned call = __builtin_amdgcn_readfirstlane(n) + 1;
    const unsigned tag = 0x100u | ((call & 1) << 9);
    LDSP(unsigned) pub = ctrl + 17 + 2 * (l & 7);
    if (l == 0) __hip_atomic_store(ctrl + 4, call, __ATOMIC_RELAXED, __HIP_MEMORY_SCOPE_WORKGROUP);
    asm volatile("s_waitcnt lgkmcnt(0)" ::: "memory");
    if (l < 8) __hip_atomic_store(pub, tag, __ATOMIC_RELAXED, __HIP_MEMORY_SCOPE_WORKGROUP);
    __builtin_amdgcn_fence(__ATOMIC_ACQUIRE, "");
    asm volatile("s_waitcnt vmcnt(0)" ::: "memory");
    const unsigned* cnt = (const unsigned*)(myws + WS_CNT + (size_t)tm * NCT2 * 128) + (l >> 3) * 32 + (l & 7);
    unsigned* tmo = (unsigned*)(myws + WS_TMO);
    const unsigned target = call * ARRIVALS;
    unsigned base = 0, hv = 0;
    const unsigned long long t0 = __builtin_amdgcn_s_memrealtime();
    while (base < (unsigned)NCT) {
      bool ok = false;
      if (l < 8 * PLW && base + (l >> 3) < (unsigned)NCT) ok = __hip_atomic_load(cnt + base * 32, __ATOMIC_RELAXED, __HIP_MEMORY_SCOPE_SYSTEM) >= target;
      const unsigned long long m = __builtin_amdgcn_ballot_w64(ok);
      const unsigned now = base + ((unsigned)__builtin_ctzll(~(m >> (l & 7)) & 0x0101010101010101ull) >> 3);
      if (__builtin_amdgcn_ballot_w64(now > hv)) {
        hv = now > hv ? now : hv;
        if (l < 8) __hip_atomic_store(pub, tag | (hv < (unsigned)NCH ? 2 * hv : NCH - 1 + hv), __ATOMIC_RELAXED, __HIP_MEMORY_SCOPE_WORKGROUP);
        unsigned k = 0;
        while (k < (unsigned)PLW && ((m >> (8 * k)) & 255u) == 255u) ++k;
        base += k;
      } else if (__builtin_amdgcn_s_memrealtime() - t0 > DEADLINE ||
                 ((__builtin_amdgcn_s_memrealtime() - t0 > 100000) && __hip_atomic_load(tmo, __ATOMIC_RELAXED, __HIP_MEMORY_SCOPE_AGENT) == call)) {
        if (l == 0) {
          __hip_atomic_store(tmo, call, __ATOMIC_RELAXED, __HIP_MEMORY_SCOPE_AGENT);
          __hip_atomic_store(ctrl, 1u, __ATOMIC_RELAXED, __HIP_MEMORY_SCOPE_WORKGROUP);
        }
        asm volatile("s_waitcnt lgkmcnt(0)" ::: "memory");
        if (l < 8) __hip_atomic_store(pub, tag | NKS, __ATOMIC_RELAXED, __HIP_MEMORY_SCOPE_WORKGROUP);
        break;
      }
    }
    while ((unsigned)__builtin_amdgcn_readfirstlane(__hip_atomic_load(oc, __ATOMIC_RELAXED, __HIP_MEMORY_SCOPE_SYSTEM)) < call * npw &&
           __builtin_amdgcn_s_memrealtime() - t0 < DEADLINE) __builtin_amdgcn_s_sleep(8);
    if (l == 0) __hip_atomic_store(pcall, call, __ATOMIC_RELAXED, __HIP_MEMORY_SCOPE_AGENT);
    return;
  }
  if (w >= 8) {
    const int h = w == 8 ? 2 : w - 9;
    const unsigned char* Wb = uni(h < 2 ? p.w + (size_t)tn * BN + (size_t)h * 64 * N : p.x + (size_t)lq * 32 * K);
    const unsigned char* Lb3 = uni(p.x + (size_t)lr3 * K);
    unsigned boff[2];
    #pragma unroll
    for (int i = 0; i < 2; ++i) boff[i] = h < 2 ? (l >> 3) * N + (((l & 7) ^ (((l >> 4) & 3) | (i << 2))) * 16)
                                                : (l >> 3) * K + (((l & 7) ^ (((l >> 4) & 1) | (i * 6))) * 16);
    auto ldw = [&](unsigned jj, int b) {
      const unsigned j = (jj + H) & (NKS - 1);
      const unsigned ld = lds0 + LDS_W + b * W_BYTES + h * 8192;
      if (h < 2) {
        const unsigned char* wb = Wb + (size_t)j * BK * N;
        #pragma unroll
        for (int i = 0; i < 8; i += 4) glds16x4(wb + (size_t)i * 8 * N, wb + (size_t)(i + 1) * 8 * N, wb + (size_t)(i + 2) * 8 * N, wb + (size_t)(i + 3) * 8 * N,
                                                boff[i >> 2], boff[i >> 2], boff[i >> 2], boff[i >> 2], ld + i * 1024);
      } else {
        const unsigned char* wb = Wb + j * BK;
        glds16x4(wb, wb + 8 * K, wb + 16 * K, wb + 24 * K, boff[0], boff[1], boff[0], boff[1], ld);
        if (x3) asm volatile("s_mov_b32 m0, %4\n\ts_nop 0\n\tglobal_load_lds_dwordx4 %0, %2\n\tglobal_load_lds_dwordx4 %1, %3 offset:1024"
                             :: "v"(boff[0]), "v"(boff[1]), "s"(Lb3 + j * BK), "s"(Lb3 + j * BK + 8 * K - 1024), "s"(ld + 4096) : "memory", "m0");
      }
    };
    constexpr int NZ = (HN + 3) / 4;
    bool z = false;
    auto prev = [&]() { if (h < 2) PW(8); else if (x3) { if (z) PW(6 + NZ); else PW(6); } else { if (z) PW(4 + NZ); else PW(4); } };
    const unsigned inc = 1u << (10 * h), nst = h < 2 ? NS + P : NS;
    auto sig = [&](unsigned k) { if (l < 8) __hip_atomic_fetch_add(ctrl + 16 + 2 * l, inc * k, __ATOMIC_RELAXED, __HIP_MEMORY_SCOPE_WORKGROUP); };
    auto ldf = [&](int b) { return (unsigned)__builtin_amdgcn_readfirstlane(__hip_atomic_load(ctrl + 8 + b, __ATOMIC_RELAXED, __HIP_MEMORY_SCOPE_WORKGROUP)); };
    ldw(0, 0); ldw(1, 1);
    __builtin_amdgcn_s_barrier();
    unsigned done = 1; int buf = 2;
    unsigned fr = 0;
    prev(); sig(1);
    for (unsigned j = 2; j < nst; ++j) {
      if (buf >= NSW) { buf = 0; fr += 8; }
      if (fr && ldf(buf) < fr) {
        if (done < j) { if (z) PW(NZ); else PW(0); sig(j - done); done = j; }
        while (ldf(buf) < fr) {}
      }
      asm volatile("" ::: "memory");
      ldw(j, buf);
      z = h == 2 && j - ZB < 16u;
      if (z) {
        static_assert(ZB >= NSW, "after the call number is known");
        const unsigned call = __builtin_amdgcn_readfirstlane(__hip_atomic_load(ctrl + 4, __ATOMIC_RELAXED, __HIP_MEMORY_SCOPE_WORKGROUP));
        __amdgpu_buffer_rsrc_t zs = __builtin_amdgcn_make_buffer_rsrc(myws + WS_STG + ((call & 1) ^ 1) * STG_BYTES + (size_t)(tm * BM + tn * 16) * K, 0, 16 * K, 0x00020000);
        #pragma unroll
        for (int i = 0; i < NZ; ++i)
          __builtin_amdgcn_raw_buffer_store_b128((v4u){0, 0, 0, 0}, zs, i * 64 + l < HN * 16 ? (NCH - HN) * KC + i * 1024 + l * 16 + (j - ZB) * K : 16 * K, 0, WT);
      }
      if (done < j) { prev(); sig(j - done); done = j; }
      ++buf;
    }
    PW(0);
    sig(NS + P - done);
    if (h == 2 && l == 0) __hip_atomic_fetch_add((unsigned*)(myws + WS_ZC), 1u, __ATOMIC_RELAXED, __HIP_MEMORY_SCOPE_AGENT);
    return;
  }

  const unsigned char* Xb;
  unsigned aoff[4];
  #pragma unroll
  for (int i = 0; i < 4; ++i) { int c = (l & 7) ^ (((l >> 4) & 1) | ((i & 1) * 6)); aoff[i] = (i * 8 + (l >> 3)) * K + c * 16 - i * 1024; }
  unsigned nx = 0;
  unsigned xm0 = lds0 + w * 4096;
  auto ldx = [&]() {
    asm volatile("s_mov_b32 m0, %5\n\ts_nop 0\n\tglobal_load_lds_dwordx4 %0, %4\n\tglobal_load_lds_dwordx4 %1, %4 offset:1024\n\t"
                 "global_load_lds_dwordx4 %2, %4 offset:2048\n\tglobal_load_lds_dwordx4 %3, %4 offset:3072"
                 :: "v"(aoff[0]), "v"(aoff[1]), "v"(aoff[2]), "v"(aoff[3]), "s"(Xb), "s"(xm0) : "memory", "m0");
    xm0 ^= X_BYTES; ++nx; Xb += BK;
  };
  if (tid < 32) ctrl[tid] = 0u;
  asm volatile("s_waitcnt lgkmcnt(0)" ::: "memory");
  __builtin_amdgcn_s_barrier();
  const int g = l >> 4, r16 = l & 15;
  const unsigned xs = ((r16 >> 1) & 1) | (((r16 >> 3) & 1) * 6);
  const unsigned xa0 = lds0 + (w * 32 + r16) * 128 + (((2 * g) ^ xs) * 16);
  const int q = r16 >> 1, pp = l & 1;
  const unsigned sB = ((q >> 1) & 3) | ((g & 1) << 2);
  const unsigned wa0 = lds0 + LDS_W + (32 * g + q) * 128 + (sB ^ (w & 6)) * 16 + 8 * pp;
  const unsigned la0 = lds0 + LDS_W + BK * BN + r16 * 128 + (((2 * g) ^ xs) * 16);
  LDSP(unsigned long long) flg = (LDSP(unsigned long long))(uintptr_t)(lds0 + LDS_CTRL + 64 + w * 8);
  auto ldflg = [&]() { return __hip_atomic_load(flg, __ATOMIC_RELAXED, __HIP_MEMORY_SCOPE_WORKGROUP); };
  v4f acc[8][2];
  #pragma unroll
  for (int a = 0; a < 8; ++a)
    #pragma unroll
    for (int b = 0; b < 2; ++b) acc[a][b] = (v4f){0, 0, 0, 0};
  v4f acl[3] = {(v4f){0, 0, 0, 0}, (v4f){0, 0, 0, 0}, (v4f){0, 0, 0, 0}};
  v8i xf[2], wf[4], xl;
#define RW(sl, ni, adr) do { _Pragma("unroll") for (int i_ = 0; i_ < 4; ++i_) { \
    v2i t_ = __builtin_amdgcn_ds_read_tr8_b64_v2i32((LDSP(v2i))(uintptr_t)(((adr) ^ ((ni) * 16)) + i_ * 1024)); \
    wf[sl][2 * i_] = t_.x; wf[sl][2 * i_ + 1] = t_.y; } } while (0)
#define RX(mi, adr) do { v4i t0_ = *(LDSP(v4i))(uintptr_t)((adr) + (mi) * 2048); v4i t1_ = *(LDSP(v4i))(uintptr_t)(((adr) ^ 16) + (mi) * 2048); \
    xf[mi] = (v8i){t0_.x, t0_.y, t0_.z, t0_.w, t1_.x, t1_.y, t1_.z, t1_.w}; } while (0)
#define MM(mi, ni) acc[ni][mi] = __builtin_amdgcn_mfma_scale_f32_16x16x128_f8f6f4(xf[mi], wf[(ni) & 3], acc[ni][mi], 0, 1, 0, 0, 0, 0)
#define RL(adr) do { v4i t0_ = *(LDSP(v4i))(uintptr_t)(adr); v4i t1_ = *(LDSP(v4i))(uintptr_t)((adr) ^ 16); \
    xl = (v8i){t0_.x, t0_.y, t0_.z, t0_.w, t1_.x, t1_.y, t1_.z, t1_.w}; } while (0)
#define ML(a, ni) acl[a] = __builtin_amdgcn_mfma_scale_f32_16x16x128_f8f6f4(xl, wf[ni], acl[a], 0, 1, 0, 0, 0, 0)
#define SB __builtin_amdgcn_sched_barrier(0)
  float sc;
  { float s0_, s1_;
    asm volatile("s_load_dword %0, %2, 0x0\n\ts_load_dword %1, %3, 0x0\n\ts_waitcnt lgkmcnt(0)" : "=&s"(s0_), "=&s"(s1_) : "s"(p.sx), "s"(p.sw) : "memory");
    sc = __builtin_bit_cast(float, __builtin_amdgcn_readfirstlane(__builtin_bit_cast(int, s0_ * s1_))); }
  asm volatile("" : "+s"(sc));
  float* outp = uni(p.out + (size_t)tm * BM * N + tn * BN);
  asm volatile("" : "+s"(outp));
  __amdgpu_buffer_rsrc_t ors = __builtin_amdgcn_make_buffer_rsrc(outp, 0, BM * N * 4, 0x00020000);
  unsigned have, wl;
  do { unsigned long long f = ldflg(); have = __builtin_amdgcn_readfirstlane((unsigned)(f >> 32)); } while (have == 0);
  Xb = uni(myws + WS_STG + ((have >> 9) & 1) * STG_BYTES + (size_t)(tm * BM + w * 32) * K);
  have &= 255u;
  auto rdf = [&](unsigned long long f) { have = __builtin_amdgcn_readfirstlane((unsigned)(f >> 32)) & 255u; wl = __builtin_amdgcn_readfirstlane((unsigned)f); };
  auto wlanded = [&](unsigned la) { return (((wl | 0x20080200u) - la * 0x100401u) & 0x20080200u) == 0x20080200u; };
  auto needx = [&](unsigned s) {
    while (nx <= s) { if (nx < have) ldx(); else rdf(ldflg()); }
    if (nx == s + 1 && nx < have) ldx();
    if (nx == s + 1) PW(0); else PW(4);
  };
  const unsigned one = 1u;
  const unsigned sval = NS - 2 * HN;
  auto xbad = [&]() {
    const unsigned a = xf[0][0] | xf[0][1] | xf[0][2] | xf[0][3] | xf[0][4] | xf[0][5] | xf[0][6] | xf[0][7];
    const unsigned b = xf[1][0] | xf[1][1] | xf[1][2] | xf[1][3] | xf[1][4] | xf[1][5] | xf[1][6] | xf[1][7];
    return __builtin_amdgcn_ballot_w64((a < b ? a : b) == 0u) != 0ull; };
  auto refetch = [&](unsigned s) {
    const unsigned call = __builtin_amdgcn_readfirstlane(__hip_atomic_load(ctrl + 4, __ATOMIC_RELAXED, __HIP_MEMORY_SCOPE_WORKGROUP));
    const unsigned char* xs_ = uni(Xb - (size_t)(nx - s) * BK);
    const unsigned xa = xa0 + (s & 1) * X_BYTES, xm = lds0 + w * 4096 + (s & 1) * X_BYTES;
    const unsigned long long t0 = __builtin_amdgcn_s_memrealtime();
    bool fin;
    do {
      fin = __builtin_amdgcn_readfirstlane(__hip_atomic_load(oc, __ATOMIC_RELAXED, __HIP_MEMORY_SCOPE_SYSTEM)) >= call * npw;
      asm volatile("s_waitcnt vmcnt(0)\n\ts_mov_b32 m0, %5\n\ts_nop 0\n\tglobal_load_lds_dwordx4 %0, %4 sc0 sc1\n\tglobal_load_lds_dwordx4 %1, %4 offset:1024 sc0 sc1\n\t"
                   "global_load_lds_dwordx4 %2, %4 offset:2048 sc0 sc1\n\tglobal_load_lds_dwordx4 %3, %4 offset:3072 sc0 sc1\n\ts_waitcnt vmcnt(0)"
                   :: "v"(aoff[0]), "v"(aoff[1]), "v"(aoff[2]), "v"(aoff[3]), "s"(xs_), "s"(xm) : "memory", "m0");
      RX(0, xa); RX(1, xa);
      if (__builtin_amdgcn_s_memrealtime() - t0 > DEADLINE || __builtin_amdgcn_readfirstlane(__hip_atomic_load(ctrl, __ATOMIC_RELAXED, __HIP_MEMORY_SCOPE_WORKGROUP))) {
        if (l == 0) __hip_atomic_store(ctrl, 1u, __ATOMIC_RELAXED, __HIP_MEMORY_SCOPE_WORKGROUP);
        fin = true;
      }
    } while (!fin && xbad());
  };
#define XCHK(s) do { if ((s) >= sval) { SB; const unsigned a_ = xf[0][7], b_ = xf[1][7]; \
    if (__builtin_expect(__builtin_amdgcn_ballot_w64((a_ < b_ ? a_ : b_) == 0u) != 0ull, 0)) { if (xbad()) refetch(s); } SB; } } while (0)
#define FREE(b) asm volatile("s_mov_b64 exec, 1\n\tds_add_u32 %0, %1\n\ts_mov_b64 exec, -1" :: "v"(lds0 + LDS_CTRL + 32 + (b) * 4), "v"(one) : "memory")
  int buf = 0;
  for (unsigned j = 0; j < P; ++j) {
    do rdf(ldflg()); while (!wlanded(j + 1));
    asm volatile("" ::: "memory");
    const unsigned wa = wa0 + buf * W_BYTES, la = la0 + buf * W_BYTES;
    RW(0, 0, wa); RL(la + (w & 1) * 2048); RW(1, 1, wa);
    if (x3) { RW(2, 2, wa ^ ((w & 1) * 16)); RX(0, la + 4096); }
    FREE(buf);
    ML(0, 0); ML(1, 1);
    if (x3) acl[2] = __builtin_amdgcn_mfma_scale_f32_16x16x128_f8f6f4(xf[0], wf[2], acl[2], 0, 1, 0, 0, 0, 0);
    if (nx < 2 && nx < have) ldx();
    if (++buf >= NSW) buf = 0;
  }
  needx(0);
  RX(0, xa0); RX(1, xa0);
  do rdf(ldflg()); while (!wlanded(P + 1));
  asm volatile("" ::: "memory");
  { const unsigned wa = wa0 + buf * W_BYTES;
    RW(0, 0, wa); RL(la0 + buf * W_BYTES + (w & 1) * 2048); RW(1, 1, wa); RW(2, 2, wa); }
  const bool x3e = x3 && !(w & 1), x3o = x3 && (w & 1);
  auto step = [&](auto loc, unsigned s) {
    constexpr bool LOC = decltype(loc)::value;
    const unsigned wa = wa0 + buf * W_BYTES;
    int nb = buf + 1; if (nb >= NSW) nb = 0;
    const unsigned wn = wa0 + nb * W_BYTES;
    SB; RW(3, 3, wa); SB;
    if (LOC) ML(0, 0);
    XCHK(s);
    MM(0, 0); MM(1, 0); SB;
    if (nx <= s + 2 && nx < have) ldx();
    SB; RW(0, 4, wa); SB;
    if (LOC) ML(1, 1);
    MM(0, 1); SB;
    if (LOC && x3) RL(la0 + buf * W_BYTES + 4096);
    SB; MM(1, 1); SB;
    const unsigned long long f = ldflg();
    RW(1, 5, wa); SB;
    MM(0, 2); MM(1, 2); SB;
    if (LOC && x3e) ML(2, 2);
    SB; RW(2, 6, wa); SB;
    MM(0, 3); MM(1, 3); SB;
    if (LOC && x3o) ML(2, 3);
    SB; RW(3, 7, wa); SB;
    FREE(buf); SB;
    MM(0, 4); MM(1, 4); SB;
    rdf(f);
    if (nx <= s + 2 && nx < have) ldx();
    while (!wlanded(P + s + 2)) rdf(ldflg());
    asm volatile("" ::: "memory");
    const unsigned xa = xa0 + ((s + 1) & 1) * X_BYTES;
    SB; RW(0, 0, wn);
    if (LOC) RL(la0 + nb * W_BYTES + (w & 1) * 2048);
    SB; MM(0, 5); MM(0, 6); MM(0, 7); SB;
    needx(s + 1);
    RX(0, xa); SB;
    MM(1, 5); SB;
    RW(1, 1, wn); SB;
    MM(1, 6); SB;
    RW(2, 2, wn); SB;
    MM(1, 7); SB;
    RX(1, xa);
    buf = nb;
  };
  for (unsigned s = 0; s < H; ++s) step(std::true_type(), s);
  {
    __amdgpu_buffer_rsrc_t lrs = __builtin_amdgcn_make_buffer_rsrc(uni(p.out + (size_t)p.rank * MS * N + tn * BN), 0, MS * N * 4 - tn * BN * 4, 0x00020000);
    ST2(lrs, (unsigned)((lq * 32 + (w & 1) * 16 + (l >> 5) * 8) * N + (l & 31)) * 4 + (w >> 1) * 128, 0, 0, acl[0], acl[1]);
    if (x3) {
      #pragma unroll
      for (int e = 0; e < 4; ++e)
        __builtin_amdgcn_raw_buffer_store_b32(__builtin_bit_cast(unsigned, fmaxf(acl[2][e] * sc, 0.f)), lrs, (unsigned)((lr3 + g * 4) * N + (w ^ 2) * 16 + r16) * 4, e * N * 4, WT);
    }
  }
  for (unsigned s = H; s < NS - 1; ++s) step(std::false_type(), s);
  const unsigned ooff = (unsigned)((w * 32 + (l >> 5) * 8) * N + (l & 31)) * 4 + (w >> 1) * 128;
#define STO(mi, nj) ST2(ors, ooff ^ ((nj) * 128), mi, 0, acc[2 * (nj)][mi], acc[2 * (nj) + 1][mi])
  {
    const unsigned wa = wa0 + buf * W_BYTES;
    SB; RW(3, 3, wa); SB;
    XCHK(NS - 1);
    MM(0, 0); MM(1, 0); SB;
    RW(0, 4, wa); SB;
    MM(0, 1); MM(1, 1); SB;
    RW(1, 5, wa); SB;
    MM(0, 2); MM(1, 2); SB;
    RW(2, 6, wa); SB;
    MM(0, 3); MM(1, 3); SB;
    RW(3, 7, wa); SB;
    SB;
    MM(0, 4); MM(1, 4); STO(0, 0); SB;
    MM(0, 5); MM(1, 5); STO(1, 0); SB;
    MM(0, 6); MM(1, 6); STO(0, 1); SB;
    MM(0, 7); MM(1, 7); STO(1, 1); SB;
    STO(0, 2); STO(1, 2); STO(0, 3); STO(1, 3);
  }
  if (__builtin_amdgcn_readfirstlane(__hip_atomic_load(ctrl, __ATOMIC_RELAXED, __HIP_MEMORY_SCOPE_WORKGROUP))) {
    for (int i = 0; i < 64; ++i) __builtin_amdgcn_raw_buffer_store_b32(0x7fc00000u, ors, (unsigned)((w * 32 + (i >> 1)) * N + (i & 1) * 64 + l) * 4, 0, WT);
  }
}

extern "C" void kernel_launch(int rank, int nranks, ncclComm_t comm,
                              hipStream_t stream,
                              void* const* d_in, const int* in_sizes, int n_in,
                              void* d_out, int out_size,
                              const int* in_dtypes, int out_dtype,
                              void* const* d_out_all,
                              void* const* d_ws_all, size_t ws_bytes) {
  static bool init_a[NR];
  if (nranks != NR || ws_bytes < WS_NEED || (unsigned)rank >= (unsigned)NR) { fprintf(stderr, "bad config\n"); return; }
  if (!init_a[rank]) {
    HIP_CHECK(hipFuncSetAttribute((const void*)ag_gemm_kernel, hipFuncAttributeMaxDynamicSharedMemorySize, LDS_BYTES));
    init_a[rank] = true;
  }
  Params p;
  p.x = (const unsigned char*)d_in[0]; p.w = (const unsigned char*)d_in[1];
  p.sx = (const float*)d_in[2]; p.sw = (const float*)d_in[3]; p.out = (float*)d_out;
  for (int i = 0; i < NR; ++i) p.ws[i] = (unsigned char*)d_ws_all[i];
  p.rank = rank;
  ag_gemm_kernel<<<NWG, NTHREADS, LDS_BYTES, stream>>>(p);
}
```

```cpp
#include <hip/hip_runtime.h>
#include <rccl/rccl.h>
#include <cstdio>
#include <cstdint>
#include <type_traits>

#define HIP_CHECK(expr) do { hipError_t _e = (expr); if (_e != hipSuccess) { \
    fprintf(stderr, "HIP error %s at %s:%d\n", hipGetErrorString(_e), __FILE__, __LINE__); \
    return; } } while (0)

typedef int v2i __attribute__((ext_vector_type(2)));
typedef int v4i __attribute__((ext_vector_type(4)));
typedef unsigned v4u __attribute__((ext_vector_type(4)));
typedef int v8i __attribute__((ext_vector_type(8)));
typedef float v4f __attribute__((ext_vector_type(4)));
#define LDSP(T) __attribute__((address_space(3))) T*
#define NEWMAP 1

constexpr int NR = 8;
constexpr int M = 4096, N = 2048, K = 4096;
constexpr int MS = M / NR;
constexpr int BM = 256, BN = 128, BK = 128;
constexpr int X_BYTES = BM * BK, W_BYTES = BK * BN + 48 * BK;
#define NSWV 4
constexpr int NSX = 2, NSW = NSWV;
constexpr int NKS = K / BK;
constexpr int KC = 256;
constexpr int NCH = K / KC;
#define SPL 1
constexpr int NCT = NCH + SPL;
#define HN 16
#define HE 2
constexpr int NCT2 = NCT + 1;
constexpr int PARTS = 64;
constexpr int ARRIVALS = 4;
constexpr int NWG = 256, GWG = 224, CWG = NWG - GWG;
constexpr int NTHREADS = 768;
#define PWV 7
#define PLW 3
#define TL 1
#define P2 7
#define P3 5
#define ZB 8
#define JROT 1
#define JSH 0
#define PPRIO 0
#define EFREE 1
#define XORD 1
#define Q3 10
constexpr int WT = 17;
constexpr unsigned NPUSH = CWG * PWV;
constexpr int LDS_W = NSX * X_BYTES;
constexpr int LDS_CTRL = LDS_W + NSW * W_BYTES;
constexpr int LDS_DBG = LDS_CTRL + 128;
constexpr int LDS_BYTES = LDS_DBG + 768;
constexpr size_t WS_TMO = 0;
constexpr size_t WS_CNT = 4096;
constexpr size_t WS_PTICK = 69632;
constexpr size_t WS_ZC = 73728;
constexpr size_t WS_PCALL = 524288;
constexpr int NITEM = NCH * PARTS;
constexpr size_t WS_STG = 1u << 20;
constexpr size_t STG_BYTES = (size_t)M * K;
constexpr size_t WS_NEED = WS_STG + 2 * STG_BYTES;
constexpr unsigned long long DEADLINE = 40000000ull;

struct Params {
  const unsigned char* x; const unsigned char* w; const float* sx; const float* sw; float* out;
  unsigned char* ws[NR];
  int rank;
};

__host__ __device__ inline int tile_tm(int lb, int rank) { return ((rank + 1 + (lb >> 5)) & 7) * 2 + (lb & 1); }

__device__ __forceinline__ void glds16x4(const unsigned char* s0, const unsigned char* s1, const unsigned char* s2, const unsigned char* s3,
                                         unsigned v0, unsigned v1, unsigned v2, unsigned v3, unsigned lds_off) {
  asm volatile("s_mov_b32 m0, %8\n\ts_nop 0\n\tglobal_load_lds_dwordx4 %0, %4\n\tglobal_load_lds_dwordx4 %1, %5 offset:1024\n\t"
               "global_load_lds_dwordx4 %2, %6 offset:2048\n\tglobal_load_lds_dwordx4 %3, %7 offset:3072"
               :: "v"(v0), "v"(v1), "v"(v2), "v"(v3), "s"(s0), "s"(s1 - 1024), "s"(s2 - 2048), "s"(s3 - 3072), "s"(lds_off) : "memory", "m0");
}

template <typename T> __device__ __forceinline__ T* uni(T* p) {
  unsigned long long v = (unsigned long long)p;
  unsigned lo = __builtin_amdgcn_readfirstlane((unsigned)v), hi = __builtin_amdgcn_readfirstlane((unsigned)(v >> 32));
  return (T*)(((unsigned long long)hi << 32) | lo);
}

#define PW(n) asm volatile("s_waitcnt vmcnt(%0)" :: "i"(n) : "memory")
#define ST2(rs, voff, mi, nj, A, B) do { v4f a_ = (A) * sc, b_ = (B) * sc; \
    _Pragma("unroll") for (int e_ = 0; e_ < 4; ++e_) { \
      auto r_ = __builtin_amdgcn_permlane16_swap(__builtin_bit_cast(unsigned, fmaxf(a_[e_], 0.f)), __builtin_bit_cast(unsigned, fmaxf(b_[e_], 0.f)), false, false); \
      __builtin_amdgcn_raw_buffer_store_b32(r_[0], rs, (voff) + (nj) * 128, ((mi) * 16 + e_) * N * 4, WT); \
      __builtin_amdgcn_raw_buffer_store_b32(r_[1], rs, (voff) + (nj) * 128, ((mi) * 16 + 4 + e_) * N * 4, WT); } } while (0)

__global__ void __launch_bounds__(NTHREADS, 3) ag_gemm_kernel(Params p) {
  extern __shared__ __attribute__((aligned(128))) unsigned char smem[];
  const int tid = threadIdx.x, l = tid & 63, w = __builtin_amdgcn_readfirstlane(tid >> 6);
  unsigned char* myws = p.ws[p.rank];
  const unsigned lds0 = (unsigned)(uintptr_t)smem;
  LDSP(unsigned) ctrl = (LDSP(unsigned))(uintptr_t)(lds0 + LDS_CTRL);
  const int lb = (blockIdx.x + GWG) & (NWG - 1);

  if (lb >= GWG) {
    if (w >= PWV) return;
    const unsigned v = (unsigned)((lb - GWG) * PWV + w);
    static_assert(PWV == NR - 1, "one wave per destination");
    const unsigned j = (unsigned)(w + JROT * ((lb - GWG) >> JSH)) % (NR - 1), nj = CWG;
    const unsigned tfirst = v / (NR - 1);
    const unsigned loff = (unsigned)(l >> 4) * K + (l & 15) * 16;
    const unsigned loff2 = (unsigned)(l >> 3) * K + (l & 7) * 16;
    constexpr unsigned TB = NITEM - TL * PARTS;
    auto canon = [&](unsigned t) {
      if (t < TB) return t;
      const unsigned u = t - TB, b = u / (8 * TL), r = u % (8 * TL);
      return TB + (r >> 3) * 64 + b * 8 + (r & 7);
    };
    auto spl = [&](unsigned c) { return c >= (unsigned)(NITEM - PARTS); };
    auto item_off = [&](unsigned c) {
      return spl(c) ? (((c >> 1) & 1) * 256 + ((c >> 3) & 7) * 32 + (c & 1) * 16) * K + (NCH - 1) * KC + ((c >> 2) & 1) * 128 + loff2
                    : (((c >> 2) & 1) * 256 + ((c >> 3) & 7) * 32 + (c & 3) * 8) * K + (c / PARTS) * KC + loff; };
    auto item_st = [&](unsigned c) { return spl(c) ? 8u * K : 4u * K; };
#define PST(d, t, i) __builtin_amdgcn_raw_buffer_store_b128(d, rs, o0, (i) * item_st(t), WT)
    auto cidx = [&](unsigned c) {
      return spl(c) ? ((((c >> 1) & 1) * NCT2 + NCH - 1 + ((c >> 2) & 1)) * 32 + ((c >> 3) & 7)) | 0x20000u
                    : ((((c >> 2) & 1) * NCT2 + c / PARTS) * 32 + ((c >> 3) & 7)) | 0x10000u; };
    constexpr unsigned TH = (NCH - HN) * PARTS;
    auto tk = [&](unsigned t) { t = __builtin_amdgcn_readfirstlane(t) + 3 * nj; return t < (unsigned)NITEM ? canon(t) : t; };
    unsigned* pcall = (unsigned*)(myws + WS_PCALL + lb * 128) + w;
    unsigned t0 = tfirst, t1;
    unsigned o0 = item_off(t0), o1;
    v4u cur[2], nxt[2];
    unsigned n = __hip_atomic_load(pcall, __ATOMIC_RELAXED, __HIP_MEMORY_SCOPE_AGENT);
    #pragma unroll
    for (int i = 0; i < 2; ++i) cur[i] = *(const v4u*)(p.x + o0 + i * item_st(t0));
    o1 = item_off(t0 + nj);
    #pragma unroll
    for (int i = 0; i < 2; ++i) nxt[i] = *(const v4u*)(p.x + o1 + i * item_st(t0));
    n = __builtin_amdgcn_readfirstlane(n);
    const unsigned parity = (n + 1) & 1;
    unsigned vz; asm volatile("v_mov_b32 %0, 0" : "=v"(vz));
    const unsigned peer = (unsigned)(p.rank + 1 + j) % NR;
    unsigned char* dws = p.ws[peer];
    __amdgpu_buffer_rsrc_t rs = __builtin_amdgcn_make_buffer_rsrc(dws + WS_STG + parity * STG_BYTES + (size_t)p.rank * MS * K, 0, MS * K, 0x00020000);
    unsigned* cnt = (unsigned*)(dws + WS_CNT + (size_t)p.rank * 2 * NCT2 * 128) + vz;
    unsigned* tick = (unsigned*)(myws + WS_PTICK + parity * 1024 + j * 128) + vz;
    unsigned cprev;
#define PSIG() do { if (l == 0) __hip_atomic_fetch_add(cnt + (cprev & 0xffffu), cprev >> 16, __ATOMIC_RELAXED, __HIP_MEMORY_SCOPE_SYSTEM); } while (0)
#define PHINT() do { if (HN && t0 >= TH) { PSIG(); cprev = 0; } } while (0)
#define DRAW(t) do { if (l == 0) t = __hip_atomic_fetch_add(tick, 1u, __ATOMIC_RELAXED, __HIP_MEMORY_SCOPE_AGENT); } while (0)
    unsigned t2 = 0;
    asm volatile("" :: "v"(cur[0]), "v"(cur[1]));
    asm volatile("" :: "v"(nxt[0]), "v"(nxt[1]));
    PST(cur[0], t0, 0);
    PST(cur[1], t0, 1);
    if (v == 0 && l < 8)
      __hip_atomic_store((unsigned*)(myws + WS_PTICK + (parity ^ 1) * 1024) + l * 32, 0u, __ATOMIC_RELAXED, __HIP_MEMORY_SCOPE_SYSTEM);
    cprev = cidx(t0);
    PHINT();
    t2 = tfirst - nj;
    asm volatile("" : "+v"(t2) :: "memory");
    t0 += nj; o0 = o1;
    unsigned s1;
    cur[0] = nxt[0]; cur[1] = nxt[1];
    t1 = tk(t2);
    bool hp = false;
    asm volatile("" :: "v"(cur[0]), "v"(cur[1]));
    while (t0 < (unsigned)NITEM) {
      o1 = item_off(t1 < (unsigned)NITEM ? t1 : t0); s1 = item_st(t1 < (unsigned)NITEM ? t1 : t0);
      DRAW(t2);
      #pragma unroll
      for (int i = 0; i < 2; ++i) nxt[i] = *(const v4u*)(p.x + o1 + i * s1);
      if (HN && HE == 0 && hp) PW(5); else PW(4);
      PST(cur[0], t0, 0);
      if (HN && HE == 0 && hp) PW(5); else PW(4);
      if (cprev) PSIG();
      cprev = cidx(t0);
      if (HE) PHINT();
      PST(cur[1], t0, 1);
      asm volatile("s_nop 1" ::: "memory");
      if (!HE) PHINT();
      asm volatile("" ::: "memory");
      hp = t0 >= TH; t0 = t1; o0 = o1;
      t1 = tk(t2);
      #pragma unroll
      for (int i = 0; i < 2; ++i) cur[i] = nxt[i];
      asm volatile("" :: "v"(cur[0]), "v"(cur[1]));
    }
    PW(0);
    if (cprev) PSIG();
    cprev = (NCT * 32) | 0x10000u;
    PSIG();
    if (tfirst == 0) {
      const unsigned long long t0 = __builtin_amdgcn_s_memrealtime();
      while ((unsigned)__builtin_amdgcn_readfirstlane(__hip_atomic_load((const unsigned*)(myws + WS_ZC), __ATOMIC_RELAXED, __HIP_MEMORY_SCOPE_AGENT)) < (n + 1) * GWG &&
             __builtin_amdgcn_s_memrealtime() - t0 < DEADLINE) {}
      cprev = (NCT * 32 + 1) | 0x10000u;
      PSIG();
    }
    if (l == 0) __hip_atomic_store(pcall, n + 1, __ATOMIC_RELAXED, __HIP_MEMORY_SCOPE_AGENT);
    return;
  }

  const int xcd = lb & 7, r7 = lb >> 5, hf = lb & 1;
  const int tm = tile_tm(lb, p.rank), tn = (xcd >> 1) * 4 + ((lb >> 3) & 3);
  const int src = tm >> 1;
  constexpr unsigned NS = NKS;
  const unsigned* oc = (const unsigned*)(myws + WS_CNT + ((size_t)src * 2 * NCT2 + NCT) * 128);
  const unsigned npw = (NPUSH + r7) / (NR - 1);
  const int lq = r7 * 2 + hf;
  const bool x3 = lq >= Q3 && lq < Q3 + 4;
  const int lr3 = x3 ? (28 + lq - Q3) * 16 : lq * 32;
  const unsigned P = x3 ? P3 : P2, H = NKS - P;

  if (w == 11) {
    unsigned* pcall = (unsigned*)(myws + WS_PCALL + lb * 128);
    unsigned n = __hip_atomic_load(pcall, __ATOMIC_RELAXED, __HIP_MEMORY_SCOPE_AGENT);
    __builtin_amdgcn_s_barrier();
    const unsigned call = __builtin_amdgcn_readfirstlane(n) + 1;
    const unsigned tag = 0x100u | ((call & 1) << 9);
    LDSP(unsigned) pub = ctrl + 17 + 2 * (l & 7);
    if (l == 0) __hip_atomic_store(ctrl + 4, call, __ATOMIC_RELAXED, __HIP_MEMORY_SCOPE_WORKGROUP);
    asm volatile("s_waitcnt lgkmcnt(0)" ::: "memory");
    if (l < 8) __hip_atomic_store(pub, tag, __ATOMIC_RELAXED, __HIP_MEMORY_SCOPE_WORKGROUP);
    __builtin_amdgcn_fence(__ATOMIC_ACQUIRE, "");
    asm volatile("s_waitcnt vmcnt(0)" ::: "memory");
    const unsigned* cnt = (const unsigned*)(myws + WS_CNT + (size_t)tm * NCT2 * 128) + (l >> 3) * 32 + (l & 7);
    unsigned* tmo = (unsigned*)(myws + WS_TMO);
    const unsigned target = call * ARRIVALS;
    unsigned base = 0, hv = 0;
    const unsigned long long t0 = __builtin_amdgcn_s_memrealtime();
    while (base < (unsigned)NCT) {
      bool ok = false;
      if (l < 8 * PLW && base + (l >> 3) < (unsigned)NCT) ok = __hip_atomic_load(cnt + base * 32, __ATOMIC_RELAXED, __HIP_MEMORY_SCOPE_SYSTEM) >= target;
      const unsigned long long m = __builtin_amdgcn_ballot_w64(ok);
      const unsigned now = base + ((unsigned)__builtin_ctzll(~(m >> (l & 7)) & 0x0101010101010101ull) >> 3);
      if (__builtin_amdgcn_ballot_w64(now > hv)) {
        hv = now > hv ? now : hv;
        if (l < 8) __hip_atomic_store(pub, tag | (hv < (unsigned)NCH ? 2 * hv : NCH - 1 + hv), __ATOMIC_RELAXED, __HIP_MEMORY_SCOPE_WORKGROUP);
        unsigned k = 0;
        while (k < (unsigned)PLW && ((m >> (8 * k)) & 255u) == 255u) ++k;
        base += k;
      } else if (__builtin_amdgcn_s_memrealtime() - t0 > DEADLINE ||
                 ((__builtin_amdgcn_s_memrealtime() - t0 > 100000) && __hip_atomic_load(tmo, __ATOMIC_RELAXED, __HIP_MEMORY_SCOPE_AGENT) == call)) {
        if (l == 0) {
          __hip_atomic_store(tmo, call, __ATOMIC_RELAXED, __HIP_MEMORY_SCOPE_AGENT);
          __hip_atomic_store(ctrl, 1u, __ATOMIC_RELAXED, __HIP_MEMORY_SCOPE_WORKGROUP);
        }
        asm volatile("s_waitcnt lgkmcnt(0)" ::: "memory");
        if (l < 8) __hip_atomic_store(pub, tag | NKS, __ATOMIC_RELAXED, __HIP_MEMORY_SCOPE_WORKGROUP);
        break;
      }
    }
    while ((unsigned)__builtin_amdgcn_readfirstlane(__hip_atomic_load(oc + 1, __ATOMIC_RELAXED, __HIP_MEMORY_SCOPE_SYSTEM)) < call &&
           __builtin_amdgcn_s_memrealtime() - t0 < DEADLINE) __builtin_amdgcn_s_sleep(8);
    if (l == 0) __hip_atomic_store(pcall, call, __ATOMIC_RELAXED, __HIP_MEMORY_SCOPE_AGENT);
    return;
  }
  if (w >= 8) {
    const int h = w == 8 ? 2 : w - 9;
    const unsigned char* Wb = uni(h < 2 ? p.w + (size_t)tn * BN + (size_t)h * 64 * N : p.x + (size_t)lq * 32 * K);
    const unsigned char* Lb3 = uni(p.x + (size_t)lr3 * K);
    unsigned boff[2];
    #pragma unroll
    for (int i = 0; i < 2; ++i) boff[i] = h < 2 ? (l >> 3) * N + (((l & 7) ^ (((l >> 4) & 3) | (i << 2))) * 16)
                                                : (l >> 3) * K + (((l & 7) ^ (((l >> 4) & 1) | (i * 6))) * 16);
    auto ldw = [&](unsigned jj, int b) {
      const unsigned j = (jj + H) & (NKS - 1);
      const unsigned ld = lds0 + LDS_W + b * W_BYTES + h * 8192;
      if (h < 2) {
        const unsigned char* wb = Wb + (size_t)j * BK * N;
        #pragma unroll
        for (int i = 0; i < 8; i += 4) glds16x4(wb + (size_t)i * 8 * N, wb + (size_t)(i + 1) * 8 * N, wb + (size_t)(i + 2) * 8 * N, wb + (size_t)(i + 3) * 8 * N,
                                                boff[i >> 2], boff[i >> 2], boff[i >> 2], boff[i >> 2], ld + i * 1024);
      } else {
        const unsigned char* wb = Wb + j * BK;
        glds16x4(wb, wb + 8 * K, wb + 16 * K, wb + 24 * K, boff[0], boff[1], boff[0], boff[1], ld);
        if (x3) asm volatile("s_mov_b32 m0, %4\n\ts_nop 0\n\tglobal_load_lds_dwordx4 %0, %2\n\tglobal_load_lds_dwordx4 %1, %3 offset:1024"
                             :: "v"(boff[0]), "v"(boff[1]), "s"(Lb3 + j * BK), "s"(Lb3 + j * BK + 8 * K - 1024), "s"(ld + 4096) : "memory", "m0");
      }
    };
    constexpr int NZ = (HN + 3) / 4;
    bool z = false;
    auto prev = [&]() { if (h < 2) PW(8); else if (x3) { if (z) PW(6 + NZ); else PW(6); } else { if (z) PW(4 + NZ); else PW(4); } };
    const unsigned inc = 1u << (10 * h), nst = h < 2 ? NS + P : NS;
    auto sig = [&](unsigned k) { if (l < 8) __hip_atomic_fetch_add(ctrl + 16 + 2 * l, inc * k, __ATOMIC_RELAXED, __HIP_MEMORY_SCOPE_WORKGROUP); };
    auto ldf = [&](int b) { return (unsigned)__builtin_amdgcn_readfirstlane(__hip_atomic_load(ctrl + 8 + b, __ATOMIC_RELAXED, __HIP_MEMORY_SCOPE_WORKGROUP)); };
    ldw(0, 0); ldw(1, 1);
    __builtin_amdgcn_s_barrier();
    unsigned done = 1; int buf = 2;
    unsigned fr = 0;
    prev(); sig(1);
    for (unsigned j = 2; j < nst; ++j) {
      if (buf >= NSW) { buf = 0; fr += 8; }
      if (fr && ldf(buf) < fr) {
        if (done < j) { if (z) PW(NZ); else PW(0); sig(j - done); done = j; }
        while (ldf(buf) < fr) {}
      }
      asm volatile("" ::: "memory");
      ldw(j, buf);
      z = h == 2 && j - ZB < 16u;
      if (z) {
        static_assert(ZB >= NSW, "after the call number is known");
        const unsigned call = __builtin_amdgcn_readfirstlane(__hip_atomic_load(ctrl + 4, __ATOMIC_RELAXED, __HIP_MEMORY_SCOPE_WORKGROUP));
        __amdgpu_buffer_rsrc_t zs = __builtin_amdgcn_make_buffer_rsrc(myws + WS_STG + ((call & 1) ^ 1) * STG_BYTES + (size_t)(tm * BM + tn * 16) * K, 0, 16 * K, 0x00020000);
        #pragma unroll
        for (int i = 0; i < NZ; ++i)
          __builtin_amdgcn_raw_buffer_store_b128((v4u){0, 0, 0, 0}, zs, i * 64 + l < HN * 16 ? (NCH - HN) * KC + i * 1024 + l * 16 + (j - ZB) * K : 16 * K, 0, WT);
      }
      if (done < j) { prev(); sig(j - done); done = j; }
      ++buf;
    }
    PW(0);
    sig(NS + P - done);
    if (h == 2 && l == 0) __hip_atomic_fetch_add((unsigned*)(myws + WS_ZC), 1u, __ATOMIC_RELAXED, __HIP_MEMORY_SCOPE_AGENT);
    return;
  }

  const unsigned char* Xb;
  unsigned aoff[4];
  #pragma unroll
  for (int i = 0; i < 4; ++i) { int c = (l & 7) ^ (((l >> 4) & 1) | ((i & 1) * 6)); aoff[i] = (i * 8 + (l >> 3)) * K + c * 16 - i * 1024; }
  unsigned nx = 0;
  unsigned xm0 = lds0 + w * 4096;
  auto ldx = [&]() __attribute__((always_inline)) {
    asm volatile("s_mov_b32 m0, %5\n\ts_nop 0\n\tglobal_load_lds_dwordx4 %0, %4\n\tglobal_load_lds_dwordx4 %1, %4 offset:1024\n\t"
                 "global_load_lds_dwordx4 %2, %4 offset:2048\n\tglobal_load_lds_dwordx4 %3, %4 offset:3072"
                 :: "v"(aoff[0]), "v"(aoff[1]), "v"(aoff[2]), "v"(aoff[3]), "s"(Xb), "s"(xm0) : "memory", "m0");
    xm0 ^= X_BYTES; ++nx; Xb += BK;
  };
  if (tid < 32) ctrl[tid] = 0u;
  asm volatile("s_waitcnt lgkmcnt(0)" ::: "memory");
  __builtin_amdgcn_s_barrier();
  const int g = l >> 4, r16 = l & 15;
  const unsigned xs = ((r16 >> 1) & 1) | (((r16 >> 3) & 1) * 6);
  const unsigned xa0 = lds0 + (w * 32 + r16) * 128 + (((2 * g) ^ xs) * 16);
  const int q = r16 >> 1, pp = l & 1;
  const unsigned sB = ((q >> 1) & 3) | ((g & 1) << 2);
  const unsigned wa0 = lds0 + LDS_W + (32 * g + q) * 128 + (sB ^ (w & 6)) * 16 + 8 * pp;
  const unsigned la0 = lds0 + LDS_W + BK * BN + r16 * 128 + (((2 * g) ^ xs) * 16);
  LDSP(unsigned long long) flg = (LDSP(unsigned long long))(uintptr_t)(lds0 + LDS_CTRL + 64 + w * 8);
  auto ldflg = [&]() { return __hip_atomic_load(flg, __ATOMIC_RELAXED, __HIP_MEMORY_SCOPE_WORKGROUP); };
  v4f acc[8][2];
  #pragma unroll
  for (int a = 0; a < 8; ++a)
    #pragma unroll
    for (int b = 0; b < 2; ++b) acc[a][b] = (v4f){0, 0, 0, 0};
  v4f acl[3] = {(v4f){0, 0, 0, 0}, (v4f){0, 0, 0, 0}, (v4f){0, 0, 0, 0}};
  v8i xf[2], wf[4], xl;
#define RW(sl, ni, adr) do { _Pragma("unroll") for (int i_ = 0; i_ < 4; ++i_) { \
    v2i t_ = __builtin_amdgcn_ds_read_tr8_b64_v2i32((LDSP(v2i))(uintptr_t)(((adr) ^ ((ni) * 16)) + i_ * 1024)); \
    wf[sl][2 * i_] = t_.x; wf[sl][2 * i_ + 1] = t_.y; } } while (0)
#define RX(mi, adr) do { v4i t0_ = *(LDSP(v4i))(uintptr_t)((adr) + (mi) * 2048); v4i t1_ = *(LDSP(v4i))(uintptr_t)(((adr) ^ 16) + (mi) * 2048); \
    xf[mi] = (v8i){t0_.x, t0_.y, t0_.z, t0_.w, t1_.x, t1_.y, t1_.z, t1_.w}; } while (0)
#define MM(mi, ni) acc[ni][mi] = __builtin_amdgcn_mfma_scale_f32_16x16x128_f8f6f4(xf[mi], wf[(ni) & 3], acc[ni][mi], 0, 1, 0, 0, 0, 0)
#define RL(adr) do { v4i t0_ = *(LDSP(v4i))(uintptr_t)(adr); v4i t1_ = *(LDSP(v4i))(uintptr_t)((adr) ^ 16); \
    xl = (v8i){t0_.x, t0_.y, t0_.z, t0_.w, t1_.x, t1_.y, t1_.z, t1_.w}; } while (0)
#define ML(a, ni) acl[a] = __builtin_amdgcn_mfma_scale_f32_16x16x128_f8f6f4(xl, wf[ni], acl[a], 0, 1, 0, 0, 0, 0)
#define SB __builtin_amdgcn_sched_barrier(0)
  float sc;
  { float s0_, s1_;
    asm volatile("s_load_dword %0, %2, 0x0\n\ts_load_dword %1, %3, 0x0\n\ts_waitcnt lgkmcnt(0)" : "=&s"(s0_), "=&s"(s1_) : "s"(p.sx), "s"(p.sw) : "memory");
    sc = __builtin_bit_cast(float, __builtin_amdgcn_readfirstlane(__builtin_bit_cast(int, s0_ * s1_))); }
  asm volatile("" : "+s"(sc));
  float* outp = uni(p.out + (size_t)tm * BM * N + tn * BN);
  asm volatile("" : "+s"(outp));
  __amdgpu_buffer_rsrc_t ors = __builtin_amdgcn_make_buffer_rsrc(outp, 0, BM * N * 4, 0x00020000);
  unsigned have, wl;
  do { unsigned long long f = ldflg(); have = __builtin_amdgcn_readfirstlane((unsigned)(f >> 32)); } while (have == 0);
  Xb = uni(myws + WS_STG + ((have >> 9) & 1) * STG_BYTES + (size_t)(tm * BM + w * 32) * K);
  have &= 255u;
  auto rdf = [&](unsigned long long f) { have = __builtin_amdgcn_readfirstlane((unsigned)(f >> 32)) & 255u; wl = __builtin_amdgcn_readfirstlane((unsigned)f); };
  auto wlanded = [&](unsigned la) { return (((wl | 0x20080200u) - la * 0x100401u) & 0x20080200u) == 0x20080200u; };
  auto needx = [&](unsigned s) __attribute__((always_inline)) {
    while (nx <= s) { if (nx < have) ldx(); else rdf(ldflg()); }
    if (nx == s + 1 && nx < have) ldx();
    if (nx == s + 1) PW(0); else PW(4);
  };
  const unsigned one = 1u;
  const unsigned sval = NS - 2 * HN;
  auto xbad = [&]() {
    const unsigned a = xf[0][0] | xf[0][1] | xf[0][2] | xf[0][3] | xf[0][4] | xf[0][5] | xf[0][6] | xf[0][7];
    const unsigned b = xf[1][0] | xf[1][1] | xf[1][2] | xf[1][3] | xf[1][4] | xf[1][5] | xf[1][6] | xf[1][7];
    return __builtin_amdgcn_ballot_w64((a < b ? a : b) == 0u) != 0ull; };
  auto refetch = [&](unsigned s) __attribute__((always_inline)) {
    const unsigned call = __builtin_amdgcn_readfirstlane(__hip_atomic_load(ctrl + 4, __ATOMIC_RELAXED, __HIP_MEMORY_SCOPE_WORKGROUP));
    const unsigned char* xs_ = uni(Xb - (size_t)(nx - s) * BK);
    const unsigned xa = xa0 + (s & 1) * X_BYTES, xm = lds0 + w * 4096 + (s & 1) * X_BYTES;
    const unsigned long long t0 = __builtin_amdgcn_s_memrealtime();
    bool fin;
    unsigned it = 0;
    do {
      asm volatile("s_cmp_lt_u32 %0, 4\n\ts_cbranch_scc1 1f\n\ts_sleep 100\n1:" :: "s"(it) : "scc");
      ++it;
      fin = __builtin_amdgcn_readfirstlane(__hip_atomic_load(oc, __ATOMIC_RELAXED, __HIP_MEMORY_SCOPE_SYSTEM)) >= call * npw;
      asm volatile("s_waitcnt vmcnt(0)\n\ts_mov_b32 m0, %5\n\ts_nop 0\n\tglobal_load_lds_dwordx4 %0, %4 sc0 sc1\n\tglobal_load_lds_dwordx4 %1, %4 offset:1024 sc0 sc1\n\t"
                   "global_load_lds_dwordx4 %2, %4 offset:2048 sc0 sc1\n\tglobal_load_lds_dwordx4 %3, %4 offset:3072 sc0 sc1\n\ts_waitcnt vmcnt(0)"
                   :: "v"(aoff[0]), "v"(aoff[1]), "v"(aoff[2]), "v"(aoff[3]), "s"(xs_), "s"(xm) : "memory", "m0");
      RX(0, xa); RX(1, xa);
      if (__builtin_amdgcn_s_memrealtime() - t0 > DEADLINE || __builtin_amdgcn_readfirstlane(__hip_atomic_load(ctrl, __ATOMIC_RELAXED, __HIP_MEMORY_SCOPE_WORKGROUP))) {
        if (l == 0) __hip_atomic_store(ctrl, 1u, __ATOMIC_RELAXED, __HIP_MEMORY_SCOPE_WORKGROUP);
        fin = true;
      }
    } while (!fin && xbad());
  };
#define XCHK(s) do { if ((s) >= sval) { SB; const unsigned a_ = xf[0][7], b_ = xf[1][7]; \
    if (__builtin_expect(__builtin_amdgcn_ballot_w64((a_ < b_ ? a_ : b_) == 0u) != 0ull, 0)) { if (xbad()) refetch(s); } SB; } } while (0)
#define FREE(b) asm volatile("s_mov_b64 exec, 1\n\tds_add_u32 %0, %1\n\ts_mov_b64 exec, -1" :: "v"(lds0 + LDS_CTRL + 32 + (b) * 4), "v"(one) : "memory")
  int buf = 0;
  for (unsigned j = 0; j < P; ++j) {
    do rdf(ldflg()); while (!wlanded(j + 1));
    asm volatile("" ::: "memory");
    const unsigned wa = wa0 + buf * W_BYTES, la = la0 + buf * W_BYTES;
    RW(0, 0, wa); RL(la + (w & 1) * 2048); RW(1, 1, wa);
    if (x3) { RW(2, 2, wa ^ ((w & 1) * 16)); RX(0, la + 4096); }
    FREE(buf);
    ML(0, 0); ML(1, 1);
    if (x3) acl[2] = __builtin_amdgcn_mfma_scale_f32_16x16x128_f8f6f4(xf[0], wf[2], acl[2], 0, 1, 0, 0, 0, 0);
    if (nx < 2 && nx < have) ldx();
    if (++buf >= NSW) buf = 0;
  }
  needx(0);
  RX(0, xa0); RX(1, xa0);
  do rdf(ldflg()); while (!wlanded(P + 1));
  asm volatile("" ::: "memory");
  { const unsigned wa = wa0 + buf * W_BYTES;
    RW(0, 0, wa); RL(la0 + buf * W_BYTES + (w & 1) * 2048); RW(1, 1, wa); RW(2, 2, wa); }
  const bool x3e = x3 && !(w & 1), x3o = x3 && (w & 1);
  auto step = [&](auto loc, unsigned s) __attribute__((always_inline)) {
    constexpr int MD = decltype(loc)::value;
    constexpr bool LOC = MD > 0;
    const unsigned wa = wa0 + buf * W_BYTES;
    int nb = buf + 1; if (nb >= NSW) nb = 0;
    const unsigned wn = wa0 + nb * W_BYTES;
    SB; RW(3, 3, wa); SB;
    if (LOC) ML(0, 0);
    XCHK(s);
    MM(0, 0); MM(1, 0); SB;
    if (nx <= s + 2 && nx < have) ldx();
    SB; RW(0, 4, wa); SB;
    if (LOC) ML(1, 1);
    MM(0, 1); SB;
    if (MD > 1) RL(la0 + buf * W_BYTES + 4096);
    SB; MM(1, 1); SB;
    const unsigned long long f = ldflg();
    RW(1, 5, wa); SB;
    MM(0, 2); MM(1, 2); SB;
    if (MD == 2 && !(w & 1)) ML(2, 2);
    SB; RW(2, 6, wa); SB;
    MM(0, 3); MM(1, 3); SB;
    if (MD == 2 && (w & 1)) ML(2, 3);
    SB; RW(3, 7, wa); SB;
    FREE(buf); SB;
    MM(0, 4); MM(1, 4); SB;
    rdf(f);
    if (nx <= s + 2 && nx < have) ldx();
    while (!wlanded(P + s + 2)) rdf(ldflg());
    asm volatile("" ::: "memory");
    const unsigned xa = xa0 + ((s + 1) & 1) * X_BYTES;
    SB; RW(0, 0, wn);
    if (LOC) RL(la0 + nb * W_BYTES + (w & 1) * 2048);
    SB; MM(0, 5); MM(0, 6); MM(0, 7); SB;
    needx(s + 1);
    RX(0, xa); SB;
    MM(1, 5); SB;
    RW(1, 1, wn); SB;
    MM(1, 6); SB;
    RW(2, 2, wn); SB;
    MM(1, 7); SB;
    RX(1, xa);
    buf = nb;
  };
  { unsigned s = 0;
    for (const unsigned e = x3 ? H : 0; s < e; ++s) step(std::integral_constant<int, 2>(), s);
    for (; s < H; ++s) step(std::integral_constant<int, 1>(), s); }
  {
    __amdgpu_buffer_rsrc_t lrs = __builtin_amdgcn_make_buffer_rsrc(uni(p.out + (size_t)p.rank * MS * N + tn * BN), 0, MS * N * 4 - tn * BN * 4, 0x00020000);
    ST2(lrs, (unsigned)((lq * 32 + (w & 1) * 16 + (l >> 5) * 8) * N + (l & 31)) * 4 + (w >> 1) * 128, 0, 0, acl[0], acl[1]);
    if (x3) {
      #pragma unroll
      for (int e = 0; e < 4; ++e)
        __builtin_amdgcn_raw_buffer_store_b32(__builtin_bit_cast(unsigned, fmaxf(acl[2][e] * sc, 0.f)), lrs, (unsigned)((lr3 + g * 4) * N + (w ^ 2) * 16 + r16) * 4, e * N * 4, WT);
    }
  }
  for (unsigned s = H; s < NS - 1; ++s) step(std::integral_constant<int, 0>(), s);
  const unsigned ooff = (unsigned)((w * 32 + (l >> 5) * 8) * N + (l & 31)) * 4 + (w >> 1) * 128;
#define STO(mi, nj) ST2(ors, ooff ^ ((nj) * 128), mi, 0, acc[2 * (nj)][mi], acc[2 * (nj) + 1][mi])
  {
    const unsigned wa = wa0 + buf * W_BYTES;
    SB; RW(3, 3, wa); SB;
    XCHK(NS - 1);
    MM(0, 0); MM(1, 0); SB;
    RW(0, 4, wa); SB;
    MM(0, 1); MM(1, 1); SB;
    RW(1, 5, wa); SB;
    MM(0, 2); MM(1, 2); SB;
    RW(2, 6, wa); SB;
    MM(0, 3); MM(1, 3); SB;
    RW(3, 7, wa); SB;
    SB;
    MM(0, 4); MM(1, 4); STO(0, 0); SB;
    MM(0, 5); MM(1, 5); STO(1, 0); SB;
    MM(0, 6); MM(1, 6); STO(0, 1); SB;
    MM(0, 7); MM(1, 7); STO(1, 1); SB;
    STO(0, 2); STO(1, 2); STO(0, 3); STO(1, 3);
  }
  if (__builtin_amdgcn_readfirstlane(__hip_atomic_load(ctrl, __ATOMIC_RELAXED, __HIP_MEMORY_SCOPE_WORKGROUP))) {
    for (int i = 0; i < 64; ++i) __builtin_amdgcn_raw_buffer_store_b32(0x7fc00000u, ors, (unsigned)((w * 32 + (i >> 1)) * N + (i & 1) * 64 + l) * 4, 0, WT);
  }
}

extern "C" void kernel_launch(int rank, int nranks, ncclComm_t comm,
                              hipStream_t stream,
                              void* const* d_in, const int* in_sizes, int n_in,
                              void* d_out, int out_size,
                              const int* in_dtypes, int out_dtype,
                              void* const* d_out_all,
                              void* const* d_ws_all, size_t ws_bytes) {
  static bool init_a[NR];
  if (nranks != NR || ws_bytes < WS_NEED || (unsigned)rank >= (unsigned)NR) { fprintf(stderr, "bad config\n"); return; }
  if (!init_a[rank]) {
    HIP_CHECK(hipFuncSetAttribute((const void*)ag_gemm_kernel, hipFuncAttributeMaxDynamicSharedMemorySize, LDS_BYTES));
    init_a[rank] = true;
  }
  Params p;
  p.x = (const unsigned char*)d_in[0]; p.w = (const unsigned char*)d_in[1];
  p.sx = (const float*)d_in[2]; p.sw = (const float*)d_in[3]; p.out = (float*)d_out;
  for (int i = 0; i < NR; ++i) p.ws[i] = (unsigned char*)d_ws_all[i];
  p.rank = rank;
  ag_gemm_kernel<<<NWG, NTHREADS, LDS_BYTES, stream>>>(p);
}
```
